```python
import numpy as np

kernel_name = "gpu_gemm_ar_m4096_k4096_n4096_bf16_reluqe4m3"


ABSMAX_RTOL = 0.09

COMPUTE_DTYPE = "bfloat16"
EPILOGUE = "relu_quant_e4m3"
IN_QUANT = None
OUT_QUANT = "e4m3"
IN_DTYPES = {"x": "bfloat16", "w_mat": "bfloat16"}
WORLD_SIZE = 8

def _to_bf16(x):
    u = np.ascontiguousarray(x, dtype=np.float32).view(np.uint32)
    return ((u + 0x7FFF + ((u >> 16) & 1)) & 0xFFFF0000).view(np.float32)

def setup_inputs():

    x = _to_bf16(np.random.randn(4096, 4096).astype(np.float32))
    w_mat = _to_bf16((np.random.randn(4096, 4096) / np.sqrt(4096)).astype(np.float32))
    return {"x": x, "w_mat": w_mat}

def _snap_fp8(x, mant_bits, max_abs, min_normal):
    a = np.abs(x.astype(np.float64))
    _m, _e = np.frexp(a)
    q = np.where(a >= min_normal, np.ldexp(1.0, _e - 1 - mant_bits),
                 min_normal * 2.0 ** -mant_bits)
    snapped = np.minimum(np.round(a / q) * q, max_abs)
    return (np.sign(x) * snapped).astype(np.float32)
def _quant_dequant(y, amax):
    scale = amax / 448.0
    q = _snap_fp8(y / scale, 3, 448.0, 2.0 ** -6)
    return (q * scale).astype(np.float32)
def _epilogue(y):
    y = np.maximum(y, 0.0)
    return _quant_dequant(y, np.abs(y).max())

def reference(x, w_mat):
    y = (x @ w_mat).astype(np.float32)
    return _epilogue(y)
```

```cpp
#include <hip/hip_runtime.h>
#include <rccl/rccl.h>
#include <cstdio>
#include <cstdint>

#define HIP_CHECK(expr) do { hipError_t _e = (expr); if (_e != hipSuccess) { \
    fprintf(stderr, "HIP error %s at %s:%d\n", hipGetErrorString(_e), __FILE__, __LINE__); \
    return; } } while (0)

typedef unsigned u32;
typedef unsigned long long u64;
typedef u32 u32x2 __attribute__((ext_vector_type(2)));
typedef u32 u32x4 __attribute__((ext_vector_type(4)));
typedef float f32x2 __attribute__((ext_vector_type(2)));
typedef float f32x4 __attribute__((ext_vector_type(4)));
typedef unsigned short u16x2 __attribute__((ext_vector_type(2)));
typedef __bf16 bf16x8 __attribute__((ext_vector_type(8)));
#define GAS __attribute__((address_space(1)))
#define LAS __attribute__((address_space(3)))
#define DI __device__ __forceinline__

constexpr int NR = 8;
constexpr int MM = 4096, NN = 4096, KS = 512;
constexpr int NCH = 16;
constexpr int NSLOT = 48;
constexpr int NSTEP = 64;
constexpr u32 UNIT_B = 3072, RAWU_B = 4096, FL_B = 128;
constexpr u32 CTR_OFF = 0;
constexpr u32 TMO_OFF = 32768;
constexpr u32 AMAX_OFF = 65536;
constexpr u32 F1_OFF = 131072;
constexpr u32 F2_OFF = 1u << 20;
constexpr u32 DBG_OFF = 1572864;
constexpr u32 P1_OFF = 2u << 20;
constexpr u32 RAW_OFF = 20u << 20;
constexpr u32 P2_OFF = 44u << 20;
constexpr u32 P2_B = 4608;
constexpr u32 WS_USED = 62u << 20;
static_assert(F1_OFF + NR * NCH * NSLOT * FL_B <= F2_OFF, "");
static_assert(F2_OFF + NR * 512 * FL_B <= DBG_OFF, "");
static_assert(P1_OFF + NR * NCH * NSLOT * UNIT_B <= RAW_OFF, "");
static_assert(RAW_OFF + NR * NCH * NSLOT * RAWU_B <= P2_OFF, "");
static_assert(P2_OFF + NR * 512 * P2_B <= WS_USED, "");
constexpr u64 DEADLINE = 50000000ull;
constexpr int MAXEXC = 8;
constexpr u32 HDR_RAW = 0x1000;

constexpr u32 L_STAGE = 0;
constexpr u32 L_COMP = 65536;
constexpr u32 L_MISC = 131072;
constexpr u32 L_DEC = 0;
constexpr u32 DEC_B = 4352;
constexpr u32 L_LUT = 40960;
constexpr u32 L_SCR = 49152;
constexpr u32 LDS_BYTES = 131072 + 512;

struct Params {
    const char* x;
    const char* w;
    char* out;
    char* ws[NR];
    char* peer[14];
    int rank;
    int pad;
};

#define STAMP(i) do {} while (0)

DI u32 perm(u32 hi, u32 lo, u32 sel) { return __builtin_amdgcn_perm(hi, lo, sel); }
DI u32 bfi(u32 m, u32 a, u32 b) { return (m & a) | (~m & b); }
DI u32 rotr(u32 v, u32 n) { return __builtin_amdgcn_alignbit(v, v, n); }
DI u32 rfl(u32 v) { return __builtin_amdgcn_readfirstlane(v); }
template <int CTRL, int RM> DI u32 dpp0(u32 v) { return __builtin_amdgcn_update_dpp(0u, v, CTRL, RM, 0xF, false); }
DI u32 wave_umax(u32 v) {
    v = max(v, dpp0<0x111, 0xF>(v)); v = max(v, dpp0<0x112, 0xF>(v));
    v = max(v, dpp0<0x114, 0xF>(v)); v = max(v, dpp0<0x118, 0xF>(v));
    v = max(v, dpp0<0x142, 0xA>(v)); v = max(v, dpp0<0x143, 0xC>(v));
    return __builtin_amdgcn_readlane(v, 63);
}
DI u32 wave_incl_scan(u32 v) {
    v += dpp0<0x111, 0xF>(v); v += dpp0<0x112, 0xF>(v);
    v += dpp0<0x114, 0xF>(v); v += dpp0<0x118, 0xF>(v);
    v += dpp0<0x142, 0xA>(v); v += dpp0<0x143, 0xC>(v);
    return v;
}
DI u64 ld_sys64(const char* p) {
    return __hip_atomic_load((GAS u64*)p, __ATOMIC_RELAXED, __HIP_MEMORY_SCOPE_SYSTEM);
}
DI void st_sys64(char* p, u64 v) {
    __hip_atomic_store((GAS u64*)p, v, __ATOMIC_RELAXED, __HIP_MEMORY_SCOPE_SYSTEM);
}
DI void st_wt128(char* p, u32x4 v) {
    asm volatile("global_store_dwordx4 %0, %1, off sc0 sc1\n\ts_nop 1" :: "v"(p), "v"(v) : "memory");
}
DI void st_wt128s(const char* sbase, u32 voff, u32x4 v) {
    asm volatile("global_store_dwordx4 %0, %1, %2 sc0 sc1\n\ts_nop 1" :: "v"(voff), "v"(v), "s"(sbase) : "memory");
}
#define WAIT_VM0() asm volatile("s_waitcnt vmcnt(0)" ::: "memory")
#define WAIT_LGKM0() asm volatile("s_waitcnt lgkmcnt(0)" ::: "memory")
#define BARRIER() do { WAIT_LGKM0(); __builtin_amdgcn_s_barrier(); asm volatile("" ::: "memory"); } while (0)

DI bool spin_expired(char* myws, u32 call, u64 t0, u32& it) {
    if ((++it & 15u) == 0) {
        GAS u32* tmo = (GAS u32*)(myws + TMO_OFF);
        u32 t = rfl(__hip_atomic_load(tmo, __ATOMIC_RELAXED, __HIP_MEMORY_SCOPE_AGENT));
        if (t == call || __builtin_amdgcn_s_memrealtime() - t0 > DEADLINE) {
            __hip_atomic_store(tmo, call, __ATOMIC_RELAXED, __HIP_MEMORY_SCOPE_AGENT);
            return true;
        }
    }
    __builtin_amdgcn_s_sleep(1);
    return false;
}
DI bool tmo_set(char* myws, u32 call) {
    return rfl(__hip_atomic_load((GAS u32*)(myws + TMO_OFF), __ATOMIC_RELAXED, __HIP_MEMORY_SCOPE_AGENT)) == call;
}

template <int ND>
DI u32 pack_unit(const u32x4 (&R)[4], u32x4& p0, u32x4& p1, u32x4& p2, LAS char* scr, int lane,
                 char* recp, char* const (&dst)[ND], u32 unit) {
    u32 L[8], H[8];
    u16x2 mx = {0, 0};
#pragma unroll
    for (int f = 0; f < 4; ++f) {
        L[2 * f] = perm(R[f][1], R[f][0], 0x06040200u); H[2 * f] = perm(R[f][1], R[f][0], 0x07050301u);
        L[2 * f + 1] = perm(R[f][3], R[f][2], 0x06040200u); H[2 * f + 1] = perm(R[f][3], R[f][2], 0x07050301u);
#pragma unroll
        for (int d = 0; d < 4; ++d)
            mx = __builtin_elementwise_max(mx, __builtin_bit_cast(u16x2, R[f][d] & 0x7FFF7FFFu));
    }
    u32 top = wave_umax(max((u32)mx[0], (u32)mx[1])) >> 8;
    u32 base = top > 7u ? top - 7u : 0u;
    u32 base4 = base * 0x01010101u;
    u32 T[8], all = 0x80808080u;
#pragma unroll
    for (int i = 0; i < 8; ++i) { T[i] = (H[i] | 0x80808080u) - base4; all &= T[i]; }
    u32 W[4];
#pragma unroll
    for (int f = 0; f < 4; ++f) {
        u32 na = bfi(0x80808080u, H[2 * f], T[2 * f]), nb = bfi(0x80808080u, H[2 * f + 1], T[2 * f + 1]);
        W[f] = bfi(0x87878787u, na, rotr(nb, 28));
    }
    p0 = u32x4{L[0], L[1], L[2], L[3]};
    p1 = u32x4{L[4], L[5], L[6], L[7]};
    p2 = u32x4{W[0], W[1], W[2], W[3]};
    u32 hdr = base;
    if (__builtin_amdgcn_ballot_w64(all != 0x80808080u) != 0) {
#pragma unroll
        for (int f = 0; f < 4; ++f) *(LAS u32x4*)(scr + f * 1024 + lane * 16) = R[f];
        u32 m = 0;
#pragma unroll
        for (int i = 0; i < 8; ++i) m |= (~T[i] & 0x80808080u) >> (7 - i);
        u32 cnt = 0;
        for (;;) {
            u64 bal = __builtin_amdgcn_ballot_w64(m != 0);
            if (bal == 0 || cnt > MAXEXC) break;
            if (m != 0) {
                u32 bp = __builtin_ctz(m), i = bp & 7, b = bp >> 3;
                u32 addr = (i >> 1) * 1024 + lane * 16 + ((i & 1) * 4 + b) * 2;
                u32 v = *(LAS unsigned short*)(scr + addr);
                u32 slot = cnt + __builtin_amdgcn_mbcnt_hi((u32)(bal >> 32), __builtin_amdgcn_mbcnt_lo((u32)bal, 0));
                if (slot < MAXEXC) *(LAS u32*)(scr + 4096 + slot * 4) = (addr << 16) | v;
                m &= m - 1;
            }
            cnt += __builtin_popcountll(bal);
        }
        if (cnt > MAXEXC) {
            hdr |= HDR_RAW;
#pragma unroll
            for (int d = 0; d < ND; ++d)
#pragma unroll
                for (int f = 0; f < 4; ++f)
                    st_wt128s(dst[d], RAW_OFF + unit * RAWU_B + f * 1024 + lane * 16, R[f]);
        } else {
            hdr |= cnt << 8;
            if (lane < 2 * ND) {
                u32x4 rec = *(LAS u32x4*)(scr + 4096 + (lane & 1) * 16);
                st_wt128(recp, rec);
            }
        }
    }
    return hdr;
}

DI void issue_x(u32x4 (&R)[4], const char* src, const u32 (&lo)[4], u32 so, u32x4 p0, u32x4 p1, u32x4 p2,
                char* const (&d)[4]) {
    asm volatile(
        "s_nop 4\n\t"
        "global_load_dwordx4 %0, %4, %8\n\t"
        "global_load_dwordx4 %1, %5, %8\n\t"
        "global_load_dwordx4 %2, %6, %8\n\t"
        "global_load_dwordx4 %3, %7, %8\n\t"
        "global_store_dwordx4 %9, %10, %13 sc0 sc1\n\t"
        "global_store_dwordx4 %9, %10, %14 sc0 sc1\n\t"
        "global_store_dwordx4 %9, %10, %15 sc0 sc1\n\t"
        "global_store_dwordx4 %9, %10, %16 sc0 sc1\n\t"
        "global_store_dwordx4 %9, %11, %13 offset:1024 sc0 sc1\n\t"
        "global_store_dwordx4 %9, %11, %14 offset:1024 sc0 sc1\n\t"
        "global_store_dwordx4 %9, %11, %15 offset:1024 sc0 sc1\n\t"
        "global_store_dwordx4 %9, %11, %16 offset:1024 sc0 sc1\n\t"
        "global_store_dwordx4 %9, %12, %13 offset:2048 sc0 sc1\n\t"
        "global_store_dwordx4 %9, %12, %14 offset:2048 sc0 sc1\n\t"
        "global_store_dwordx4 %9, %12, %15 offset:2048 sc0 sc1\n\t"
        "global_store_dwordx4 %9, %12, %16 offset:2048 sc0 sc1\n\t"
        "s_nop 1\n\t"
        "s_waitcnt vmcnt(12)"
        : "=&v"(R[0]), "=&v"(R[1]), "=&v"(R[2]), "=&v"(R[3])
        : "v"(lo[0]), "v"(lo[1]), "v"(lo[2]), "v"(lo[3]), "s"(src), "v"(so), "v"(p0), "v"(p1), "v"(p2),
          "s"(d[0]), "s"(d[1]), "s"(d[2]), "s"(d[3])
        : "memory");
}
DI void issue_w(u32x2 (&G)[8], const char* src, const u32 (&lo)[8], u32 so, u32x4 p0, u32x4 p1, u32x4 p2,
                char* const (&d)[2]) {
    asm volatile(
        "s_nop 4\n\t"
        "global_load_dwordx2 %0, %8, %16\n\t"
        "global_load_dwordx2 %1, %9, %16\n\t"
        "global_load_dwordx2 %2, %10, %16\n\t"
        "global_load_dwordx2 %3, %11, %16\n\t"
        "global_load_dwordx2 %4, %12, %16\n\t"
        "global_load_dwordx2 %5, %13, %16\n\t"
        "global_load_dwordx2 %6, %14, %16\n\t"
        "global_load_dwordx2 %7, %15, %16\n\t"
        "global_store_dwordx4 %17, %18, %21 sc0 sc1\n\t"
        "global_store_dwordx4 %17, %18, %22 sc0 sc1\n\t"
        "global_store_dwordx4 %17, %19, %21 offset:1024 sc0 sc1\n\t"
        "global_store_dwordx4 %17, %19, %22 offset:1024 sc0 sc1\n\t"
        "global_store_dwordx4 %17, %20, %21 offset:2048 sc0 sc1\n\t"
        "global_store_dwordx4 %17, %20, %22 offset:2048 sc0 sc1\n\t"
        "s_nop 1\n\t"
        "s_waitcnt vmcnt(6)"
        : "=&v"(G[0]), "=&v"(G[1]), "=&v"(G[2]), "=&v"(G[3]), "=&v"(G[4]), "=&v"(G[5]), "=&v"(G[6]), "=&v"(G[7])
        : "v"(lo[0]), "v"(lo[1]), "v"(lo[2]), "v"(lo[3]), "v"(lo[4]), "v"(lo[5]), "v"(lo[6]), "v"(lo[7]),
          "s"(src), "v"(so), "v"(p0), "v"(p1), "v"(p2), "s"(d[0]), "s"(d[1])
        : "memory");
}
DI void w_to_frags(const u32x2 (&G)[8], u32x4 (&R)[4]) {
#pragma unroll
    for (int c = 0; c < 4; ++c)
#pragma unroll
        for (int d = 0; d < 4; ++d)
            R[c][d] = perm(G[2 * d + 1][c >> 1], G[2 * d][c >> 1], (c & 1) ? 0x07060302u : 0x05040100u);
}

DI u32 push_x(const Params& p, int u, u32 ctr_old, LAS char* lds, int lane) {
    const int r0 = (u >> 5) * 4;
    char* const dst[4] = {p.ws[r0], p.ws[r0 + 1], p.ws[r0 + 2], p.ws[r0 + 3]};
    const int q = lane >> 4, i = lane & 15;
    u32 lo[4];
#pragma unroll
    for (int f = 0; f < 4; ++f) lo[f] = (u32)(64 * u + 16 * f + i) * (KS * 2) + q * 16;
    u32 unit = p.rank * (NCH * NSLOT) + (u & 31);
    char* fd = (lane & 2) ? ((lane & 1) ? dst[3] : dst[2]) : ((lane & 1) ? dst[1] : dst[0]);
    char* rd = (lane & 4) ? ((lane & 2) ? dst[3] : dst[2]) : ((lane & 2) ? dst[1] : dst[0]);
    u32x4 R[4], p0, p1, p2;
#pragma unroll
    for (int f = 0; f < 4; ++f) R[f] = *(const GAS u32x4*)(p.x + lo[f]);
    const u32 call = rfl(ctr_old) + 1u;
    BARRIER();
    u32 prev = 0;
    for (int c = 0; c < NCH; ++c) {
        u32 hdr = pack_unit<4>(R, p0, p1, p2, lds + L_SCR, lane,
                               rd + F1_OFF + unit * FL_B + 64 + (lane & 1) * 16, dst, unit);
        issue_x(R, p.x + min(c + 1, NCH - 1) * 64, lo, P1_OFF + unit * UNIT_B + lane * 16, p0, p1, p2, dst);
        if (c > 0 && lane < 4) st_sys64(fd + F1_OFF + (unit - NSLOT) * FL_B, ((u64)call << 32) | prev);
        prev = hdr;
        unit += NSLOT;
    }
    WAIT_VM0();
    if (lane < 4) st_sys64(fd + F1_OFF + (unit - NSLOT) * FL_B, ((u64)call << 32) | prev);
    return call;
}
DI u32 push_w(const Params& p, int u, u32 ctr_old, LAS char* lds, int lane) {
    const int rj = u >> 4;
    char* const dst[2] = {p.ws[rj], p.ws[rj + 4]};
    const int q = lane >> 4, i = lane & 15;
    u32 lo[8];
#pragma unroll
    for (int j = 0; j < 8; ++j) lo[j] = (u32)(8 * q + j) * (NN * 2) + u * 128 + i * 8;
    u32 unit = p.rank * (NCH * NSLOT) + 32 + (u & 15);
    char* fd = (lane & 1) ? dst[1] : dst[0];
    char* rd = (lane & 2) ? dst[1] : dst[0];
    u32x2 G[8];
    u32x4 R[4], p0, p1, p2;
#pragma unroll
    for (int j = 0; j < 8; ++j) G[j] = *(const GAS u32x2*)(p.w + lo[j]);
    const u32 call = rfl(ctr_old) + 1u;
    BARRIER();
    u32 prev = 0;
    for (int c = 0; c < NCH; ++c) {
        w_to_frags(G, R);
        u32 hdr = pack_unit<2>(R, p0, p1, p2, lds + L_SCR, lane,
                               rd + F1_OFF + unit * FL_B + 64 + (lane & 1) * 16, dst, unit);
        issue_w(G, p.w + (size_t)min(c + 1, NCH - 1) * (32 * NN * 2), lo, P1_OFF + unit * UNIT_B + lane * 16,
                p0, p1, p2, dst);
        if (c > 0 && lane < 2) st_sys64(fd + F1_OFF + (unit - NSLOT) * FL_B, ((u64)call << 32) | prev);
        prev = hdr;
        unit += NSLOT;
    }
    WAIT_VM0();
    if (lane < 2) st_sys64(fd + F1_OFF + (unit - NSLOT) * FL_B, ((u64)call << 32) | prev);
    return call;
}

struct PSet { u32x4 pay[2][3]; u32 rec[2]; u32 hdr[2]; u32 unit[2]; };

DI u32 unit_of(int rank, int t, int slot) {
    return (u32)((((rank + t) & 7) * NCH + (t >> 3)) * NSLOT + slot);
}
DI u64 load_flags(char* myws, int rank, int s, int slot, int lane) {
    int t = min(2 * s + (lane & 1), 2 * NSTEP - 1);
    return ld_sys64(myws + F1_OFF + unit_of(rank, t, slot) * FL_B);
}
DI bool wait_flags(u64& F, char* myws, int rank, int s, int slot, int lane, u32 call) {
    if (__builtin_amdgcn_ballot_w64((u32)(F >> 32) < call) == 0) return false;
    u64 t0 = __builtin_amdgcn_s_memrealtime();
    u32 it = 15;
    for (;;) {
        F = load_flags(myws, rank, s, slot, lane);
        if (__builtin_amdgcn_ballot_w64((u32)(F >> 32) < call) == 0) return false;
        if (spin_expired(myws, call, t0, it)) return true;
    }
}
DI void issue_set(PSet& S, u64 F, __amdgpu_buffer_rsrc_t rs, int rank, int s, int slot, int lane) {
    asm volatile("" ::: "memory");
#pragma unroll
    for (int h = 0; h < 2; ++h) {
        S.hdr[h] = __builtin_amdgcn_readlane((u32)F, h);
        u32 un = unit_of(rank, 2 * s + h, slot);
        S.unit[h] = un;
#pragma unroll
        for (int j = 0; j < 3; ++j)
            S.pay[h][j] = __builtin_amdgcn_raw_buffer_load_b128(rs, lane * 16 + j * 1024, P1_OFF + un * UNIT_B, 17);
        S.rec[h] = __builtin_amdgcn_raw_buffer_load_b32(rs, (lane & 7) * 4 + 64, F1_OFF + un * FL_B, 17);
    }
}
DI void unpack_set(const PSet& S, __amdgpu_buffer_rsrc_t rs, LAS char* dst, int lane) {
#pragma unroll
    for (int h = 0; h < 2; ++h) {
        LAS char* o = dst + h * 16384;
        const u32 hdr = S.hdr[h];
        if (hdr & HDR_RAW) {
#pragma unroll
            for (int f = 0; f < 4; ++f)
                *(LAS u32x4*)(o + f * 1024 + lane * 16) =
                    __builtin_amdgcn_raw_buffer_load_b128(rs, lane * 16 + f * 1024, RAW_OFF + S.unit[h] * RAWU_B, 17);
        } else {
            const u32 base4 = (hdr & 0x7F) * 0x01010101u;
#pragma unroll
            for (int f = 0; f < 4; ++f) {
                u32 w = S.pay[h][2][f];
                u32 h0 = (w & 0x87878787u) + base4, h1 = (rotr(w, 4) & 0x87878787u) + base4;
                u32 l0 = S.pay[h][f >> 1][(f & 1) * 2], l1 = S.pay[h][f >> 1][(f & 1) * 2 + 1];
                u32x4 r = {perm(h0, l0, 0x05010400u), perm(h0, l0, 0x07030602u),
                           perm(h1, l1, 0x05010400u), perm(h1, l1, 0x07030602u)};
                *(LAS u32x4*)(o + f * 1024 + lane * 16) = r;
            }
            if ((u32)lane < ((hdr >> 8) & 15u))
                *(LAS unsigned short*)(o + ((S.rec[h] >> 16) & 0xFFE)) = (unsigned short)S.rec[h];
        }
    }
}

__global__ __launch_bounds__(512, 2) void fused_kernel(Params p) {
    extern __shared__ __attribute__((aligned(16))) char smem_[];
    LAS char* lds = (LAS char*)smem_;
    const int lane = threadIdx.x & 63;
    const int wave = rfl(threadIdx.x >> 6);
    const int b = blockIdx.x, xcd = b & 7, idx = b >> 3;
    const int rank = p.rank;
    char* myws = p.ws[rank];
    STAMP(0);

    GAS u32* ctr = (GAS u32*)(myws + CTR_OFF + b * 128);
    const u32 ctr_old = __hip_atomic_load(ctr, __ATOMIC_RELAXED, __HIP_MEMORY_SCOPE_AGENT);

    if (idx < 16) {
        const int g = xcd * 16 + idx;
        if (threadIdx.x >= 256) {
            u32 m = threadIdx.x - 256, n = 0, s[2] = {0, 0};
#pragma unroll
            for (int e = 0; e < 8; ++e) {
                u32 bit = (m >> e) & 1;
                s[e >> 2] |= (bit ? n : 0x0Cu) << ((e & 3) * 8);
                n += bit;
            }
            *(LAS u32x2*)(lds + L_LUT + m * 8) = u32x2{s[0], s[1]};
        }
        u32 call;
        if (wave == 0) {
            const int u = xcd * 8 + (idx >> 1);
            if (idx & 1) call = push_w(p, u, ctr_old, lds, lane);
            else call = push_x(p, u, ctr_old, lds, lane);
        } else {
            call = rfl(ctr_old) + 1u;
            BARRIER();
            if (threadIdx.x == 64) __hip_atomic_store(ctr, call, __ATOMIC_RELAXED, __HIP_MEMORY_SCOPE_AGENT);
        }
        STAMP(1);
        const __amdgpu_buffer_rsrc_t rs = __builtin_amdgcn_make_buffer_rsrc(myws, 0, WS_USED, 0x00020000);
        const int uq = g * 4 + (wave & 3);
        const int rot = uq % 7;
        const int tr = (g >> 5) * 4 + ((g >> 2) & 3), tc = ((g >> 4) & 1) * 4 + (g & 3);
        LAS char* buf = lds + L_DEC + wave * DEC_B;
        for (int k = wave >> 2; k < 7; k += 2) {
            const int src = (rank + 7 - (k + rot) % 7) & 7;
            const u32 un = src * 512 + uq;
            bool bad = false;
            u64 F = ld_sys64(myws + F2_OFF + un * FL_B);
            if (rfl((u32)(F >> 32)) < call) {
                u64 t0 = __builtin_amdgcn_s_memrealtime();
    u32 it = 15;
                for (;;) {
                    F = ld_sys64(myws + F2_OFF + un * FL_B);
                    if (rfl((u32)(F >> 32)) >= call) break;
                    if (spin_expired(myws, call, t0, it)) { bad = true; break; }
                }
            }
            asm volatile("" ::: "memory");
            const u32 total = min(rfl((u32)F), 4096u);
            const u32 so = P2_OFF + un * P2_B;
            u32x2 mk = __builtin_amdgcn_raw_buffer_load_b64(rs, (lane & 31) * 16 + (lane >> 5) * 8, so, 17);
            u32x4 pl[4];
#pragma unroll
            for (int r = 0; r < 4; ++r)
                pl[r] = __builtin_amdgcn_raw_buffer_load_b128(
                    rs, (u32)(lane * 16 + r * 1024) < total ? (u32)(512 + lane * 16 + r * 1024) : 0x80000000u, so, 17);
            bad = bad || tmo_set(myws, call);
#pragma unroll
            for (int r = 0; r < 4; ++r) *(LAS u32x4*)(buf + r * 1024 + lane * 16) = pl[r];
            u32 cnt = __builtin_popcount(mk[0]) + __builtin_popcount(mk[1]);
            u32 o = wave_incl_scan(cnt) - cnt;
            char* orow = p.out + (size_t)((src >> 2) * 2048 + tr * 128 + ((wave >> 1) & 1) * 64 + (lane & 15)) * NN +
                         (src & 3) * 1024 + tc * 128 + (wave & 1) * 64 + (lane >> 4) * 16;
#pragma unroll
            for (int mt = 0; mt < 4; ++mt) {
                u32x4 ov;
#pragma unroll
                for (int hh = 0; hh < 2; ++hh) {
                    u32 mb = (mk[mt >> 1] >> ((mt & 1) * 16 + hh * 8)) & 0xFF;
                    u32x2 sel = *(LAS u32x2*)(lds + L_LUT + mb * 8);
                    LAS u32* dp = (LAS u32*)(buf + (o & ~3u));
                    u32 d0 = dp[0], d1 = dp[1], d2 = dp[2];
                    u32 a0 = __builtin_amdgcn_alignbyte(d1, d0, o), a1 = __builtin_amdgcn_alignbyte(d2, d1, o);
                    ov[2 * hh] = perm(a1, a0, sel[0]);
                    ov[2 * hh + 1] = perm(a1, a0, sel[1]);
                    o += __builtin_popcount(mb);
                }
                if (bad) ov = u32x4{0x7F7F7F7Fu, 0x7F7F7F7Fu, 0x7F7F7F7Fu, 0x7F7F7F7Fu};
                st_wt128(orow + (size_t)mt * 16 * NN, ov);
            }
            STAMP(2 + (k >> 1));
        }
        return;
    }

    const int g = xcd * 16 + (idx - 16);
    const int tr = (g >> 5) * 4 + ((g >> 2) & 3), tc = ((g >> 4) & 1) * 4 + (g & 3);
    const u32 call = rfl(ctr_old) + 1u;

    if (wave >= 4) {
        const int v = wave - 4;
        const int slot = v < 2 ? 2 * tr + v : 32 + 2 * tc + (v - 2);
        const __amdgpu_buffer_rsrc_t rs = __builtin_amdgcn_make_buffer_rsrc(myws, 0, WS_USED, 0x00020000);
        LAS char* o0 = lds + L_STAGE + v * 4096;
        LAS char* o1 = o0 + 32768;
        PSet A, B;
        bool bad = false;
        u64 F = load_flags(myws, rank, 0, slot, lane);
        bad |= wait_flags(F, myws, rank, 0, slot, lane, call);
        issue_set(A, F, rs, rank, 0, slot, lane);
        STAMP(1);
        F = load_flags(myws, rank, 1, slot, lane);
        bad |= wait_flags(F, myws, rank, 1, slot, lane, call);
        u64 Fn = load_flags(myws, rank, 2, slot, lane);
        issue_set(B, F, rs, rank, 1, slot, lane);
        for (int s = 0; s < NSTEP - 2; s += 2) {
            unpack_set(A, rs, o0, lane);
            F = Fn;
            bad |= wait_flags(F, myws, rank, s + 2, slot, lane, call);
            Fn = load_flags(myws, rank, s + 3, slot, lane);
            issue_set(A, F, rs, rank, s + 2, slot, lane);
            BARRIER();
            unpack_set(B, rs, o1, lane);
            F = Fn;
            bad |= wait_flags(F, myws, rank, s + 3, slot, lane, call);
            Fn = load_flags(myws, rank, s + 4, slot, lane);
            issue_set(B, F, rs, rank, s + 3, slot, lane);
            BARRIER();
        }
        STAMP(2);
        unpack_set(A, rs, o0, lane);
        BARRIER();
        unpack_set(B, rs, o1, lane);
        BARRIER();
        BARRIER();
        if (v == 0) {
            u32x4 wm = *(LAS u32x4*)(lds + L_MISC + 16);
            u32 tm = max(max(wm[0], wm[1]), max(wm[2], wm[3]));
            char* dp = p.ws[0];
#pragma unroll
            for (int r = 1; r < NR; ++r) dp = (lane & 7) == r ? p.ws[r] : dp;
            if (lane < NR) st_sys64(dp + AMAX_OFF + (rank * 128 + g) * 8, ((u64)call << 32) | tm);
        }
        STAMP(3);
        u32 gm;
        {
            const char* ap = myws + AMAX_OFF + (v * 256 + lane) * 8;
            u64 t0 = __builtin_amdgcn_s_memrealtime();
    u32 it = 15;
            for (;;) {
                u64 a0 = ld_sys64(ap), a1 = ld_sys64(ap + 512), a2 = ld_sys64(ap + 1024), a3 = ld_sys64(ap + 1536);
                gm = max(max((u32)a0, (u32)a1), max((u32)a2, (u32)a3));
                u32 tg = min(min((u32)(a0 >> 32), (u32)(a1 >> 32)), min((u32)(a2 >> 32), (u32)(a3 >> 32)));
                if (__builtin_amdgcn_ballot_w64(tg < call) == 0) break;
                if (spin_expired(myws, call, t0, it)) { bad = true; break; }
            }
        }
        gm = wave_umax(gm);
        bad = bad || tmo_set(myws, call);
        if (bad) gm = 0xFFFFFFFFu;
        if (lane == 0) *(LAS u32*)(lds + L_MISC + 32 + v * 4) = gm;
        STAMP(4);
        BARRIER();
        return;
    }

    const int wm = wave >> 1, wn = wave & 1;
    f32x4 acc[4][4];
#pragma unroll
    for (int i = 0; i < 4; ++i)
#pragma unroll
        for (int j = 0; j < 4; ++j) acc[i][j] = f32x4{0.f, 0.f, 0.f, 0.f};
    {
        LAS char* xb = lds + L_STAGE + wm * 4096 + lane * 16;
        LAS char* wb = lds + L_STAGE + (2 + wn) * 4096 + lane * 16;
        bf16x8 x0[4], w0[4], x1[4], w1[4];
#define RD(X, W, off) _Pragma("unroll") for (int f = 0; f < 4; ++f) { \
            X[f] = *(LAS bf16x8*)(xb + (off) + f * 1024); W[f] = *(LAS bf16x8*)(wb + (off) + f * 1024); }
#define MM16(X, W) _Pragma("unroll") for (int i = 0; i < 4; ++i) _Pragma("unroll") for (int j = 0; j < 4; ++j) \
            acc[i][j] = __builtin_amdgcn_mfma_f32_16x16x32_bf16(W[j], X[i], acc[i][j], 0, 0, 0);
        BARRIER();
        if (threadIdx.x == 0) __hip_atomic_store(ctr, call, __ATOMIC_RELAXED, __HIP_MEMORY_SCOPE_AGENT);
        RD(x0, w0, 0);
        for (int s = 0; s < NSTEP - 2; s += 2) {
            RD(x1, w1, 16384);
            MM16(x0, w0);
            BARRIER();
            RD(x0, w0, 32768);
            MM16(x1, w1);
            RD(x1, w1, 49152);
            MM16(x0, w0);
            BARRIER();
            RD(x0, w0, 0);
            MM16(x1, w1);
        }
        RD(x1, w1, 16384);
        MM16(x0, w0);
        BARRIER();
        RD(x0, w0, 32768);
        MM16(x1, w1);
        RD(x1, w1, 49152);
        MM16(x0, w0);
        MM16(x1, w1);
    }
    STAMP(2);
    u32 y[64];
    u32 mx = 0;
#pragma unroll
    for (int e = 0; e < 64; ++e) {
        y[e] = (u32)max((int)__float_as_uint(acc[e >> 4][e & 3][(e >> 2) & 3]), 0);
        mx = max(mx, y[e]);
    }
    mx = wave_umax(mx);
    if (lane == 0) *(LAS u32*)(lds + L_MISC + 16 + wave * 4) = mx;
    BARRIER();

    u32 mk0 = 0, mk1 = 0;
#pragma unroll
    for (int e = 31; e >= 0; --e) {
        mk0 = __builtin_amdgcn_alignbit(mk0, 0u - y[e], 31);
        mk1 = __builtin_amdgcn_alignbit(mk1, 0u - y[e + 32], 31);
    }
    const u32 cnt = __builtin_popcount(mk0) + __builtin_popcount(mk1);
    const u32 incl = wave_incl_scan(cnt);
    const u32 total = __builtin_amdgcn_readlane(incl, 63);
    const int uq = g * 4 + wave;
    const int rot = uq % 7;
    const u32 uoff = P2_OFF + (rank * 512 + uq) * P2_B;
    char* pd[7];
#pragma unroll
    for (int k = 0; k < 7; ++k) pd[k] = p.peer[rot + k];
    asm volatile("" : "+s"(pd[0]), "+s"(pd[1]), "+s"(pd[2]), "+s"(pd[3]), "+s"(pd[4]), "+s"(pd[5]), "+s"(pd[6]));
    {
        auto s0 = __builtin_amdgcn_permlane32_swap(mk0, mk0, false, false);
        auto s1 = __builtin_amdgcn_permlane32_swap(mk1, mk1, false, false);
        u32x4 mv = {s0[0], s1[0], s0[1], s1[1]};
        if (lane < 32) {
#pragma unroll
            for (int k = 0; k < 7; ++k) st_wt128s(pd[k], uoff + lane * 16, mv);
        }
    }
    LAS char* cb = lds + L_COMP + wave * 16384;
    {
        u32 pos = (incl - cnt) * 4;
        const u32 dummy = L_MISC + 256 + lane * 4 - (L_COMP + wave * 16384);
#pragma unroll
        for (int e = 0; e < 64; ++e) {
            bool nz = y[e] != 0;
            *(LAS u32*)(cb + (nz ? pos : dummy)) = y[e];
            pos += nz ? 4u : 0u;
        }
    }
    f32x4 cv[4][4];
#pragma unroll
    for (int r = 0; r < 4; ++r)
#pragma unroll
        for (int j = 0; j < 4; ++j) cv[r][j] = *(LAS f32x4*)(cb + r * 4096 + lane * 64 + j * 16);
    STAMP(3);
    BARRIER();
    float inv;
    {
        u32x4 gv = *(LAS u32x4*)(lds + L_MISC + 32);
        float amax = __uint_as_float(max(max(gv[0], gv[1]), max(gv[2], gv[3])));
        inv = 1.0f / (amax / 448.0f);
    }
    STAMP(4);
    u32x4 q[4];
#pragma unroll
    for (int r = 0; r < 4; ++r)
#pragma unroll
        for (int j = 0; j < 4; ++j) {
            f32x4 t = cv[r][j] * inv;
            int w = __builtin_amdgcn_cvt_pk_fp8_f32(t[0], t[1], 0, false);
            q[r][j] = (u32)__builtin_amdgcn_cvt_pk_fp8_f32(t[2], t[3], w, true);
        }
#pragma unroll
    for (int k = 0; k < 7; ++k) {
        char* d = pd[k];
#pragma unroll
        for (int r = 0; r < 4; ++r)
            if ((u32)(lane * 16 + r * 1024) < total) st_wt128s(d, uoff + 512 + r * 1024 + lane * 16, q[r]);
        if (k == 0) {
            char* orow = p.out + (size_t)((rank >> 2) * 2048 + tr * 128 + wm * 64 + (lane & 15)) * NN +
                         (rank & 3) * 1024 + tc * 128 + wn * 64 + (lane >> 4) * 16;
#pragma unroll
            for (int mt = 0; mt < 4; ++mt) {
                u32x4 ov;
#pragma unroll
                for (int j = 0; j < 4; ++j) {
                    const int e = mt * 16 + j * 4;
                    int w = __builtin_amdgcn_cvt_pk_fp8_f32(__uint_as_float(y[e]) * inv, __uint_as_float(y[e + 1]) * inv, 0, false);
                    ov[j] = (u32)__builtin_amdgcn_cvt_pk_fp8_f32(__uint_as_float(y[e + 2]) * inv, __uint_as_float(y[e + 3]) * inv, w, true);
                }
                st_wt128(orow + (size_t)mt * 16 * NN, ov);
            }
        }
        WAIT_VM0();
        if (lane == 0) st_sys64(d + F2_OFF + (rank * 512 + uq) * FL_B, ((u64)call << 32) | total);
    }
    STAMP(5);
}

extern "C" void kernel_launch(int rank, int nranks, ncclComm_t comm, hipStream_t stream,
                              void* const* d_in, const int* in_sizes, int n_in,
                              void* d_out, int out_size, const int* in_dtypes, int out_dtype,
                              void* const* d_out_all, void* const* d_ws_all, size_t ws_bytes) {
    (void)comm; (void)in_dtypes; (void)out_dtype; (void)d_out_all;
    if (nranks != NR || (unsigned)rank >= (unsigned)NR || n_in < 2 || ws_bytes < WS_USED ||
        in_sizes[0] != MM * KS || in_sizes[1] != KS * NN || out_size != MM * NN) {
        fprintf(stderr, "kernel_launch: unexpected problem shape\n");
        return;
    }
    HIP_CHECK(hipFuncSetAttribute((const void*)fused_kernel, hipFuncAttributeMaxDynamicSharedMemorySize, LDS_BYTES));
    Params p;
    p.x = (const char*)d_in[0];
    p.w = (const char*)d_in[1];
    p.out = (char*)d_out;
    for (int r = 0; r < NR; ++r) p.ws[r] = (char*)d_ws_all[r];
    for (int k = 0; k < 14; ++k) p.peer[k] = p.ws[(rank + 1 + k % 7) & 7];
    p.rank = rank;
    p.pad = 0;
    fused_kernel<<<dim3(256), dim3(512), LDS_BYTES, stream>>>(p);
    HIP_CHECK(hipGetLastError());
}
```

```cpp
#include <hip/hip_runtime.h>
#include <rccl/rccl.h>
#include <cstdio>
#include <cstdint>

#define HIP_CHECK(expr) do { hipError_t _e = (expr); if (_e != hipSuccess) { \
    fprintf(stderr, "HIP error %s at %s:%d\n", hipGetErrorString(_e), __FILE__, __LINE__); \
    return; } } while (0)

typedef unsigned u32;
typedef unsigned long long u64;
typedef u32 u32x2 __attribute__((ext_vector_type(2)));
typedef u32 u32x4 __attribute__((ext_vector_type(4)));
typedef float f32x2 __attribute__((ext_vector_type(2)));
typedef float f32x4 __attribute__((ext_vector_type(4)));
typedef float f32x16 __attribute__((ext_vector_type(16)));
typedef unsigned short u16x2 __attribute__((ext_vector_type(2)));
typedef __bf16 bf16x8 __attribute__((ext_vector_type(8)));
#define GAS __attribute__((address_space(1)))
#define LAS __attribute__((address_space(3)))
#define DI __device__ __forceinline__

constexpr int NR = 8;
constexpr int MM = 4096, NN = 4096, KS = 512;
constexpr int NCH = 16;
constexpr int NSLOT = 48;
constexpr u32 NITEM = NCH * 128;
constexpr int NSTEP = 64;
constexpr u32 UNIT_B = 3072, RAWU_B = 4096, FL_B = 128;
constexpr u32 CTR_OFF = 0;
constexpr u32 TMO_OFF = 32768;
constexpr u32 TKT_OFF = 33024;
constexpr u32 AMAX_OFF = 65536;
constexpr u32 F1_OFF = 131072;
constexpr u32 F2_OFF = 1u << 20;
constexpr u32 DBG_OFF = 1572864;
constexpr u32 P1_OFF = 2u << 20;
constexpr u32 RAW_OFF = 20u << 20;
constexpr u32 P2_OFF = 44u << 20;
constexpr u32 P2_B = 4608;
constexpr u32 WS_USED = 62u << 20;
static_assert(F1_OFF + NR * NCH * NSLOT * FL_B <= F2_OFF, "");
static_assert(F2_OFF + NR * 512 * FL_B <= DBG_OFF, "");
static_assert(P1_OFF + NR * NCH * NSLOT * UNIT_B <= RAW_OFF, "");
static_assert(RAW_OFF + NR * NCH * NSLOT * RAWU_B <= P2_OFF, "");
static_assert(P2_OFF + NR * 512 * P2_B <= WS_USED, "");
constexpr u64 DEADLINE = 50000000ull;
constexpr int MAXEXC = 8;
constexpr u32 HDR_RAW = 0x1000;

constexpr u32 L_STAGE = 0;
constexpr u32 L_COMP = 0;
constexpr u32 L_MISC = 131072;
constexpr u32 L_DEC = 0;
constexpr u32 DEC_B = 4352;
constexpr u32 L_LUT = 53248;
constexpr u32 L_SCR = 57344;
constexpr u32 LDS_BYTES = 131072 + 512;

struct Params {
    const char* x;
    const char* w;
    char* out;
    char* ws[NR];
    char* peer[14];
    int rank;
    int pad;
};

#define PAY_AUX 17
#define ACC(var) do {} while (0)
#define PUT(i, val) do {} while (0)
#define STAMP(i) do {} while (0)

#define PSTAMP() do {} while (0)
DI u32 perm(u32 hi, u32 lo, u32 sel) { return __builtin_amdgcn_perm(hi, lo, sel); }
DI u32 bfi(u32 m, u32 a, u32 b) { return (m & a) | (~m & b); }
DI u32 rotr(u32 v, u32 n) { return __builtin_amdgcn_alignbit(v, v, n); }
DI u32 rfl(u32 v) { return __builtin_amdgcn_readfirstlane(v); }
template <int CTRL, int RM> DI u32 dpp0(u32 v) { return __builtin_amdgcn_update_dpp(0u, v, CTRL, RM, 0xF, false); }
DI u32 wave_umax(u32 v) {
    v = max(v, dpp0<0x111, 0xF>(v)); v = max(v, dpp0<0x112, 0xF>(v));
    v = max(v, dpp0<0x114, 0xF>(v)); v = max(v, dpp0<0x118, 0xF>(v));
    v = max(v, dpp0<0x142, 0xA>(v)); v = max(v, dpp0<0x143, 0xC>(v));
    return __builtin_amdgcn_readlane(v, 63);
}
DI u32 wave_incl_scan(u32 v) {
    v += dpp0<0x111, 0xF>(v); v += dpp0<0x112, 0xF>(v);
    v += dpp0<0x114, 0xF>(v); v += dpp0<0x118, 0xF>(v);
    v += dpp0<0x142, 0xA>(v); v += dpp0<0x143, 0xC>(v);
    return v;
}
DI u64 ld_sys64(const char* p) {
    return __hip_atomic_load((GAS u64*)p, __ATOMIC_RELAXED, __HIP_MEMORY_SCOPE_SYSTEM);
}
DI void st_sys64(char* p, u64 v) {
    __hip_atomic_store((GAS u64*)p, v, __ATOMIC_RELAXED, __HIP_MEMORY_SCOPE_SYSTEM);
}
DI void st_wt128(char* p, u32x4 v) {
    asm volatile("global_store_dwordx4 %0, %1, off sc0 sc1\n\ts_nop 1" :: "v"(p), "v"(v) : "memory");
}
DI void st_wt128s(const char* sbase, u32 voff, u32x4 v) {
    asm volatile("global_store_dwordx4 %0, %1, %2 sc0 sc1\n\ts_nop 1" :: "v"(voff), "v"(v), "s"(sbase) : "memory");
}
#define WAIT_VM0() asm volatile("s_waitcnt vmcnt(0)" ::: "memory")
#define WAIT_LGKM0() asm volatile("s_waitcnt lgkmcnt(0)" ::: "memory")
#define BARRIER() do { WAIT_LGKM0(); __builtin_amdgcn_s_barrier(); asm volatile("" ::: "memory"); } while (0)

DI bool spin_expired(char* myws, u32 call, u64 t0, u32& it) {
    if ((++it & 15u) == 0) {
        GAS u32* tmo = (GAS u32*)(myws + TMO_OFF);
        u32 t = rfl(__hip_atomic_load(tmo, __ATOMIC_RELAXED, __HIP_MEMORY_SCOPE_AGENT));
        if (t == call || __builtin_amdgcn_s_memrealtime() - t0 > DEADLINE) {
            __hip_atomic_store(tmo, call, __ATOMIC_RELAXED, __HIP_MEMORY_SCOPE_AGENT);
            return true;
        }
    }
    __builtin_amdgcn_s_sleep(1);
    return false;
}
DI bool tmo_set(char* myws, u32 call) {
    return rfl(__hip_atomic_load((GAS u32*)(myws + TMO_OFF), __ATOMIC_RELAXED, __HIP_MEMORY_SCOPE_AGENT)) == call;
}

DI u32 pack_unit(const u32x4 (&R)[4], u32x4& p0, u32x4& p1, u32x4& p2, LAS char* scr, int lane,
                 char* d0, char* d1, char* d2, char* d3, char* fl, int nd, u32 unit) {
    u32 L[8], H[8];
    u16x2 mx = {0, 0};
#pragma unroll
    for (int f = 0; f < 4; ++f) {
        L[2 * f] = perm(R[f][1], R[f][0], 0x06040200u); H[2 * f] = perm(R[f][1], R[f][0], 0x07050301u);
        L[2 * f + 1] = perm(R[f][3], R[f][2], 0x06040200u); H[2 * f + 1] = perm(R[f][3], R[f][2], 0x07050301u);
#pragma unroll
        for (int d = 0; d < 4; ++d)
            mx = __builtin_elementwise_max(mx, __builtin_bit_cast(u16x2, R[f][d] & 0x7FFF7FFFu));
    }
    u32 top = wave_umax(max((u32)mx[0], (u32)mx[1])) >> 8;
    u32 base = top > 7u ? top - 7u : 0u;
    u32 base4 = base * 0x01010101u;
    u32 T[8], all = 0x80808080u;
#pragma unroll
    for (int i = 0; i < 8; ++i) { T[i] = (H[i] | 0x80808080u) - base4; all &= T[i]; }
    u32 W[4];
#pragma unroll
    for (int f = 0; f < 4; ++f) {
        u32 na = bfi(0x80808080u, H[2 * f], T[2 * f]), nb = bfi(0x80808080u, H[2 * f + 1], T[2 * f + 1]);
        W[f] = bfi(0x87878787u, na, rotr(nb, 28));
    }
    p0 = u32x4{L[0], L[1], L[2], L[3]};
    p1 = u32x4{L[4], L[5], L[6], L[7]};
    p2 = u32x4{W[0], W[1], W[2], W[3]};
    u32 hdr = base;
    if (__builtin_amdgcn_ballot_w64(all != 0x80808080u) != 0) {
#pragma unroll
        for (int f = 0; f < 4; ++f) *(LAS u32x4*)(scr + f * 1024 + lane * 16) = R[f];
        u32 m = 0;
#pragma unroll
        for (int i = 0; i < 8; ++i) m |= (~T[i] & 0x80808080u) >> (7 - i);
        u32 cnt = 0;
        for (;;) {
            u64 bal = __builtin_amdgcn_ballot_w64(m != 0);
            if (bal == 0 || cnt > MAXEXC) break;
            if (m != 0) {
                u32 bp = __builtin_ctz(m), i = bp & 7, b = bp >> 3;
                u32 addr = (i >> 1) * 1024 + lane * 16 + ((i & 1) * 4 + b) * 2;
                u32 v = *(LAS unsigned short*)(scr + addr);
                u32 slot = cnt + __builtin_amdgcn_mbcnt_hi((u32)(bal >> 32), __builtin_amdgcn_mbcnt_lo((u32)bal, 0));
                if (slot < MAXEXC) *(LAS u32*)(scr + 4096 + slot * 4) = (addr << 16) | v;
                m &= m - 1;
            }
            cnt += __builtin_popcountll(bal);
        }
        if (cnt > MAXEXC) {
            hdr |= HDR_RAW;
#pragma unroll
            for (int f = 0; f < 4; ++f) {
                const u32 o = RAW_OFF + unit * RAWU_B + f * 1024 + lane * 16;
                st_wt128s(d0, o, R[f]);
                st_wt128s(d1, o, R[f]);
                if (nd > 2) { st_wt128s(d2, o, R[f]); st_wt128s(d3, o, R[f]); }
            }
        } else {
            hdr |= cnt << 8;
            if (lane < 8 && (lane & 3) < nd) {
                u32x4 rec = *(LAS u32x4*)(scr + 4096 + (lane & 4) * 4);
                st_wt128(fl + 64 + (lane & 4) * 4, rec);
            }
        }
    }
    return hdr;
}

#define A_TICKET(tk, adr) "s_nop 4\n\ts_mov_b64 exec, 1\n\tglobal_atomic_add %" #tk ", %" #adr ", %" #tk ", off sc0\n\ts_mov_b64 exec, -1\n\t"
#define A_ST(so, v, d, off) "global_store_dwordx4 %" #so ", %" #v ", %" #d " offset:" #off " sc0 sc1\n\t"
#define A_ST4(so, a, b, c, d0, d1, d2, d3) \
    A_ST(so, a, d0, 0) A_ST(so, a, d1, 0) A_ST(so, a, d2, 0) A_ST(so, a, d3, 0) \
    A_ST(so, b, d0, 1024) A_ST(so, b, d1, 1024) A_ST(so, b, d2, 1024) A_ST(so, b, d3, 1024) \
    A_ST(so, c, d0, 2048) A_ST(so, c, d1, 2048) A_ST(so, c, d2, 2048) A_ST(so, c, d3, 2048) \
    "s_nop 1\n\ts_waitcnt vmcnt(12)"
#define A_ST2(so, a, b, c, d0, d1) \
    A_ST(so, a, d0, 0) A_ST(so, a, d1, 0) A_ST(so, b, d0, 1024) A_ST(so, b, d1, 1024) \
    A_ST(so, c, d0, 2048) A_ST(so, c, d1, 2048) "s_nop 1\n\ts_waitcnt vmcnt(6)"
#define A_LDX "global_load_dwordx4 %0, %5, %7\n\tglobal_load_dwordx4 %1, %6, %7\n\t" \
              "global_load_dwordx4 %2, %5, %7 offset:32\n\tglobal_load_dwordx4 %3, %6, %7 offset:32\n\t"
#define A_WL(i) "global_load_dword %" #i ", %17, %18\n\tv_add_u32 %17, 0x2000, %17\n\t"
#define A_LDW A_WL(0) A_WL(1) A_WL(2) A_WL(3) A_WL(4) A_WL(5) A_WL(6) \
              "global_load_dword %7, %17, %18\n\tv_add_u32 %17, 0x12000, %17\n\t" \
              A_WL(8) A_WL(9) A_WL(10) A_WL(11) A_WL(12) A_WL(13) A_WL(14) "global_load_dword %15, %17, %18\n\t"
#define X_OUT "=&v"(R[0]), "=&v"(R[1]), "=&v"(R[2]), "=&v"(R[3]), "+v"(tk)
#define X_IN "v"(lx0), "v"(lx1), "s"(src), "v"(tka), "v"(so), "v"(p0), "v"(p1), "v"(p2)
#define W_OUT "=&v"(G[0]), "=&v"(G[1]), "=&v"(G[2]), "=&v"(G[3]), "=&v"(G[4]), "=&v"(G[5]), "=&v"(G[6]), \
              "=&v"(G[7]), "=&v"(G[8]), "=&v"(G[9]), "=&v"(G[10]), "=&v"(G[11]), "=&v"(G[12]), "=&v"(G[13]), \
              "=&v"(G[14]), "=&v"(G[15]), "+v"(tk), "+v"(lw)
#define W_IN "s"(src), "v"(tka), "v"(so), "v"(p0), "v"(p1), "v"(p2)

DI void w_to_frags(const u32 (&G)[16], u32x4 (&R)[4]) {
#pragma unroll
    for (int f = 0; f < 4; ++f)
#pragma unroll
        for (int d = 0; d < 4; ++d)
            R[f][d] = perm(G[(f >> 1) * 8 + 2 * d + 1], G[(f >> 1) * 8 + 2 * d], (f & 1) ? 0x07060302u : 0x05040100u);
}

struct Item { bool isw; u32 unit; const char* src; char *d0, *d1, *d2, *d3; };
DI Item make_item(const Params& p, u32 i) {
    Item it;
    const u32 c = (i >> 7) & (NCH - 1), k = (i >> 1) & 63;
    it.isw = i & 1;
    const u32 r0 = it.isw ? (k & 3) : (k & 1) * 4, rs = it.isw ? 4 : 1;
    const u32 sl = it.isw ? k >> 2 : k >> 1;
    it.unit = (p.rank * NCH + c) * NSLOT + (it.isw ? 32 : 0) + sl;
    it.src = it.isw ? p.w + (size_t)c * (32 * NN * 2) + ((k & 3) * 16 + sl) * 128
                    : p.x + (size_t)((k & 1) * 32 + sl) * (64 * KS * 2) + c * 64;
    it.d0 = p.ws[r0]; it.d1 = p.ws[r0 + rs]; it.d2 = p.ws[(r0 + 2 * rs) & 7]; it.d3 = p.ws[(r0 + 3 * rs) & 7];
    return it;
}

DI u32 push_inputs(const Params& p, u32 g, u32 ctr_old, LAS char* lds, int lane) {
    const u32 lx0 = (u32)(lane & 31) * (KS * 2) + (lane >> 5) * 16, lx1 = lx0 + 32 * KS * 2;
    const u32 lw0 = (u32)(8 * (lane >> 5)) * (NN * 2) + ((lane & 19) | ((lane & 4) << 1) | ((lane & 8) >> 1)) * 4;
    char* const tka = p.ws[p.rank] + TKT_OFF;
    Item cur = make_item(p, g), nxt = make_item(p, 128 + g);
    u32 G[16];
    u32x4 R[4], p0, p1, p2;
    if (cur.isw) {
#pragma unroll
        for (int j = 0; j < 16; ++j) G[j] = *(const GAS u32*)(cur.src + lw0 + ((j >> 3) * 16 + (j & 7)) * (NN * 2));
        w_to_frags(G, R);
    } else {
#pragma unroll
        for (int f = 0; f < 4; ++f) R[f] = *(const GAS u32x4*)(cur.src + ((f & 1) ? lx1 : lx0) + (f >> 1) * 32);
    }
    const u32 call = rfl(ctr_old) + 1u;
    BARRIER();
    const u32 tbase = (call - 1u) * NITEM - 256u;
    bool more = true;
    char* fp = nullptr;
    u32 fv = 0;
    int fn = 0;
    for (;;) {
        const int nd = cur.isw ? 2 : 4;
        char* const dl = (lane & 1) ? cur.d1 : cur.d0;
        char* const dh = (lane & 1) ? cur.d3 : cur.d2;
        char* const fl = ((lane & 2) ? dh : dl) + F1_OFF + cur.unit * FL_B;
        const u32 hdr = pack_unit(R, p0, p1, p2, lds + L_SCR, lane, cur.d0, cur.d1, cur.d2, cur.d3, fl, nd, cur.unit);
        const u32 so = P1_OFF + cur.unit * UNIT_B + lane * 16;
        const char* src = nxt.src;
        u32 tk = 1;
        if (nxt.isw) {
            u32 lw = lw0;
            if (cur.isw)
                asm volatile(A_TICKET(16, 19) A_LDW A_ST2(20, 21, 22, 23, 24, 25)
                             : W_OUT : W_IN, "s"(cur.d0), "s"(cur.d1) : "memory");
            else
                asm volatile(A_TICKET(16, 19) A_LDW A_ST4(20, 21, 22, 23, 24, 25, 26, 27)
                             : W_OUT : W_IN, "s"(cur.d0), "s"(cur.d1), "s"(cur.d2), "s"(cur.d3) : "memory");
            w_to_frags(G, R);
        } else {
            if (cur.isw)
                asm volatile(A_TICKET(4, 8) A_LDX A_ST2(9, 10, 11, 12, 13, 14)
                             : X_OUT : X_IN, "s"(cur.d0), "s"(cur.d1) : "memory");
            else
                asm volatile(A_TICKET(4, 8) A_LDX A_ST4(9, 10, 11, 12, 13, 14, 15, 16)
                             : X_OUT : X_IN, "s"(cur.d0), "s"(cur.d1), "s"(cur.d2), "s"(cur.d3) : "memory");
        }
        if (lane < fn) st_sys64(fp, ((u64)call << 32) | fv);
        fp = fl;
        fv = hdr;
        fn = nd;
        if (!more) break;
        const u32 i = rfl(tk) - tbase;
        cur = nxt;
        more = i < NITEM;
        nxt = make_item(p, i);
    }
    WAIT_VM0();
    if (lane < fn) st_sys64(fp, ((u64)call << 32) | fv);
    return call;
}

typedef __amdgpu_buffer_rsrc_t rsrc_t;
struct PSet { u32x4 pay[6]; u32 rec; u32 hdr0, hdr1; u32 unit; };

DI u32x2 load_flags(rsrc_t rs, u32 voff, u32 unit) {
    return __builtin_amdgcn_raw_buffer_load_b64(rs, voff, F1_OFF + unit * FL_B, 17);
}
DI bool wait_flags(u32x2& F0, u32x2& Fn, rsrc_t rs, char* myws, u32 unit, u32 next, int lane, u32 call) {
    u64 t0 = __builtin_amdgcn_s_memrealtime();
    u32 it = 0;
    bool bad = false;
    const u32 voff = F1_OFF + (((lane & 2) ? next : unit) + (lane & 1)) * FL_B;
    u32x2 F;
    for (;;) {
        asm volatile("" ::: "memory");
        F = __builtin_amdgcn_raw_buffer_load_b64(rs, voff, 0, 17);
        if ((__builtin_amdgcn_ballot_w64(F[1] < call) & 3) == 0) break;
        if (spin_expired(myws, call, t0, it)) { bad = true; break; }
    }
    F0 = F;
    Fn[0] = __builtin_amdgcn_update_dpp(0u, F[0], 0xEE, 0xF, 0xF, false);
    Fn[1] = __builtin_amdgcn_update_dpp(0u, F[1], 0xEE, 0xF, 0xF, false);
    return bad;
}
DI void issue_set(PSet& S, u32x2 F, rsrc_t rs, u32 unit, int lane) {
    asm volatile("" ::: "memory");
    S.hdr0 = __builtin_amdgcn_readlane(F[0], 0);
    S.hdr1 = __builtin_amdgcn_readlane(F[0], 1);
    S.unit = unit;
#pragma unroll
    for (int j = 0; j < 6; ++j)
        S.pay[j] = __builtin_amdgcn_raw_buffer_load_b128(rs, lane * 16 + j * 1024, P1_OFF + unit * UNIT_B, PAY_AUX);
    S.rec = __builtin_amdgcn_raw_buffer_load_b32(rs, (lane & 7) * 4 + 64 + (lane & 8) * (FL_B / 8), F1_OFF + unit * FL_B, 17);
}
DI void unpack_set(const PSet& S, rsrc_t rs, LAS char* o, int lane) {
#pragma unroll
    for (int u = 0; u < 2; ++u) {
        const u32 base4 = ((u ? S.hdr1 : S.hdr0) & 0x7F) * 0x01010101u;
#pragma unroll
        for (int f = 0; f < 4; ++f) {
            u32 w = S.pay[3 * u + 2][f];
            u32 h0 = (w & 0x87878787u) + base4, h1 = (rotr(w, 4) & 0x87878787u) + base4;
            u32 l0 = S.pay[3 * u + (f >> 1)][(f & 1) * 2], l1 = S.pay[3 * u + (f >> 1)][(f & 1) * 2 + 1];
            u32x4 r = {perm(h0, l0, 0x05010400u), perm(h0, l0, 0x07030602u),
                       perm(h1, l1, 0x05010400u), perm(h1, l1, 0x07030602u)};
            *(LAS u32x4*)(o + u * 4096 + f * 1024 + lane * 16) = r;
            __builtin_amdgcn_sched_barrier(0);
        }
    }
    if (__builtin_expect((S.hdr0 | S.hdr1) >= 0x100u, 0)) {
        asm volatile("" ::: "memory");
        const u32 hdr = (lane & 8) ? S.hdr1 : S.hdr0;
        if (lane < 16 && (u32)(lane & 7) < ((hdr >> 8) & 15u))
            *(LAS unsigned short*)(o + (lane & 8) * 512 + ((S.rec >> 16) & 0xFFE)) = (unsigned short)S.rec;
        if (__builtin_expect(((S.hdr0 | S.hdr1) & HDR_RAW) != 0, 0)) {
#pragma unroll
            for (int u = 0; u < 2; ++u)
                if ((u ? S.hdr1 : S.hdr0) & HDR_RAW) {
#pragma unroll
                    for (int f = 0; f < 4; ++f)
                        *(LAS u32x4*)(o + u * 4096 + f * 1024 + lane * 16) = __builtin_amdgcn_raw_buffer_load_b128(
                            rs, lane * 16 + f * 1024, RAW_OFF + (S.unit + u) * RAWU_B, 17);
                }
        }
    }
}

DI void store_tile(char* o, const u32x4 (&v)[4], int lane) {
    o += (size_t)(lane & 15) * NN + ((lane >> 4) & 1) * 32 + (lane >> 5) * 16;
#pragma unroll
    for (int mi = 0; mi < 2; ++mi) {
        u32x4 a, b;
#pragma unroll
        for (int j = 0; j < 4; ++j) {
            auto r = __builtin_amdgcn_permlane16_swap(v[2 * mi][j], v[2 * mi + 1][j], false, false);
            a[j] = r[0]; b[j] = r[1];
        }
        st_wt128(o + (size_t)(32 * mi) * NN, a);
        st_wt128(o + (size_t)(32 * mi + 16) * NN, b);
    }
}

__global__ __launch_bounds__(768, 3) void fused_kernel(Params p) {
    extern __shared__ __attribute__((aligned(16))) char smem_[];
    LAS char* lds = (LAS char*)smem_;
    const int lane = threadIdx.x & 63;
    const int wave = rfl(threadIdx.x >> 6);
    const int b = blockIdx.x, xcd = b & 7, idx = b >> 3;
    const int rank = p.rank;
    char* myws = p.ws[rank];
    STAMP(0);

    GAS u32* ctr = (GAS u32*)(myws + CTR_OFF + b * 128);
    const u32 ctr_old = __hip_atomic_load(ctr, __ATOMIC_RELAXED, __HIP_MEMORY_SCOPE_AGENT);

    if (idx < 16) {
        const int g = xcd * 16 + idx;
        if (threadIdx.x >= 256 && threadIdx.x < 512) {
            u32 m = threadIdx.x - 256, n = 0, s[2] = {0, 0};
#pragma unroll
            for (int e = 0; e < 8; ++e) {
                u32 bit = (m >> e) & 1;
                s[e >> 2] |= (bit ? n : 0x0Cu) << ((e & 3) * 8);
                n += bit;
            }
            *(LAS u32x2*)(lds + L_LUT + m * 8) = u32x2{s[0], s[1]};
        }
        u32 call;
        if (wave == 0) {
            call = push_inputs(p, g, ctr_old, lds, lane);
        } else {
            call = rfl(ctr_old) + 1u;
            BARRIER();
            if (threadIdx.x == 64) __hip_atomic_store(ctr, call, __ATOMIC_RELAXED, __HIP_MEMORY_SCOPE_AGENT);
        }
        STAMP(1);
        const rsrc_t rs = __builtin_amdgcn_make_buffer_rsrc(myws, 0, WS_USED, 0x00020000);
        const int uq = g * 4 + (wave & 3);
        const int rot = uq % 7;
        const int tr = (g >> 5) * 4 + ((g >> 2) & 3), tc = ((g >> 4) & 1) * 4 + (g & 3);
        LAS char* buf = lds + L_DEC + wave * DEC_B;
        for (int k = wave >> 2; k < 7; k += 3) {
            const int src = (rank + 7 - (k + rot) % 7) & 7;
            const u32 un = src * 512 + uq;
            bool bad = false;
            u64 F = ld_sys64(myws + F2_OFF + un * FL_B);
            if (rfl((u32)(F >> 32)) < call) {
                u64 t0 = __builtin_amdgcn_s_memrealtime();
    u32 it = 0;
                for (;;) {
                    F = ld_sys64(myws + F2_OFF + un * FL_B);
                    if (rfl((u32)(F >> 32)) >= call) break;
                    if (spin_expired(myws, call, t0, it)) { bad = true; break; }
                }
            }
            asm volatile("" ::: "memory");
            const u32 total = min(rfl((u32)F), 4096u);
            const u32 so = P2_OFF + un * P2_B;
            u32x2 mk = __builtin_amdgcn_raw_buffer_load_b64(rs, (lane & 31) * 16 + (lane >> 5) * 8, so, 17);
            u32x4 pl[4];
#pragma unroll
            for (int r = 0; r < 4; ++r)
                pl[r] = __builtin_amdgcn_raw_buffer_load_b128(
                    rs, (u32)(lane * 16 + r * 1024) < total ? (u32)(512 + lane * 16 + r * 1024) : 0x80000000u, so, 17);
            bad = bad || tmo_set(myws, call);
#pragma unroll
            for (int r = 0; r < 4; ++r) *(LAS u32x4*)(buf + r * 1024 + lane * 16) = pl[r];
            u32 cnt = __builtin_popcount(mk[0]) + __builtin_popcount(mk[1]);
            u32 o = wave_incl_scan(cnt) - cnt;
            u32x4 ot[4];
#pragma unroll
            for (int mt = 0; mt < 4; ++mt) {
                u32x4 ov;
#pragma unroll
                for (int hh = 0; hh < 2; ++hh) {
                    u32 mb = (mk[mt >> 1] >> ((mt & 1) * 16 + hh * 8)) & 0xFF;
                    u32x2 sel = *(LAS u32x2*)(lds + L_LUT + mb * 8);
                    LAS u32* dp = (LAS u32*)(buf + (o & ~3u));
                    u32 d0 = dp[0], d1 = dp[1], d2 = dp[2];
                    u32 a0 = __builtin_amdgcn_alignbyte(d1, d0, o), a1 = __builtin_amdgcn_alignbyte(d2, d1, o);
                    ov[2 * hh] = perm(a1, a0, sel[0]);
                    ov[2 * hh + 1] = perm(a1, a0, sel[1]);
                    o += __builtin_popcount(mb);
                }
                if (bad) ov = u32x4{0x7F7F7F7Fu, 0x7F7F7F7Fu, 0x7F7F7F7Fu, 0x7F7F7F7Fu};
                ot[mt] = ov;
            }
            store_tile(p.out + (size_t)((src >> 2) * 2048 + tr * 128 + ((wave >> 1) & 1) * 64) * NN +
                       (src & 3) * 1024 + tc * 128 + (wave & 1) * 64, ot, lane);
            STAMP(2 + k / 3);
        }
        return;
    }

    const int g = xcd * 16 + (idx - 16);
    const int tr = (g >> 5) * 4 + ((g >> 2) & 3), tc = ((g >> 4) & 1) * 4 + (g & 3);
    const u32 call = rfl(ctr_old) + 1u;

    if (wave >= 4) {
        const int v = wave - 4, ph = v >> 1;
        const int slot = (v & 1) ? 32 + 2 * tc : 2 * tr;
        const rsrc_t rs = __builtin_amdgcn_make_buffer_rsrc(myws, 0, WS_USED, 0x00020000);
        LAS char* oa = lds + L_STAGE + ph * 16384 + (v & 1) * 8192;
        volatile LAS unsigned short* prog = (volatile LAS unsigned short*)(lds + L_MISC + 64 + v * 2);
        volatile LAS u32x4* done = (volatile LAS u32x4*)(lds + L_MISC + 96);
        const u32 fo = (lane & 1) * FL_B;
        u32 ua = ((rank + ph) & 7) * (NCH * NSLOT) + slot, ub = ((rank + ph + 4) & 7) * (NCH * NSLOT) + slot;
        PSet A, B;
        bool bad = false;
        u32x2 F0, Fn;
#define PAYWAIT() do {} while (0)
#define ACC2(x) do {} while (0)
#define STAGE(S, o, t) do { \
            if (__builtin_expect((int)rfl(min(min(D[0], D[1]), min(D[2], D[3]))) < (t) - 7, 0)) { \
                do { __builtin_amdgcn_s_sleep(1); D = *done; } while ((int)rfl(min(min(D[0], D[1]), min(D[2], D[3]))) < (t) - 7); } \
            ACC2(a_bar); PAYWAIT(); \
            unpack_set(S, rs, o, lane); \
            asm volatile("" ::: "memory"); \
            *prog = (unsigned short)(((t) >> 2) + 1); \
            D = *done; ACC2(a_unp); } while (0)
#define TRYFETCH(S, pend, un, nx, valid) do { \
            const bool up_ = (valid) && __builtin_amdgcn_ballot_w64(Fn[1] < call) == 0; \
            pend = (valid) && !up_; \
            F0 = Fn; \
            Fn = load_flags(rs, fo, nx); \
            issue_set(S, F0, __builtin_amdgcn_make_buffer_rsrc(myws, 0, up_ ? WS_USED : 0u, 0x00020000), un, lane); \
            ACC2(a_iss); } while (0)
#define REPAIR(S, pend, un, nx) do { \
            if (pend) { bad |= wait_flags(F0, Fn, rs, myws, un, nx, lane, call); issue_set(S, F0, rs, un, lane); LASTREP(un); } \
            ACC2(a_flag); } while (0)
#define LASTREP(un) do {} while (0)
        bool pa = true, pb;
        REPAIR(A, pa, ua, ub);
        STAMP(1);
        TRYFETCH(B, pb, ub, ua + NSLOT, true);
        BARRIER();
        u32x4 D = {0, 0, 0, 0};
        for (int t = ph; t < 2 * NSTEP; t += 8) {
            const bool more = t < 2 * NSTEP - 8;
            const u32 st = more ? NSLOT : 0;
            STAGE(A, oa, t);
            ua += st;
            REPAIR(B, pb, ub, ua);
            TRYFETCH(A, pa, ua, ub + st, more);
            STAGE(B, oa + 65536, t + 4);
            ub += st;
            REPAIR(A, pa, ua, ub);
            TRYFETCH(B, pb, ub, ua + (t < 2 * NSTEP - 16 ? NSLOT : 0), more);
        }
        STAMP(2);
        PUT(6, lr_t); PUT(13, lr_u * 100); PUT(14, lr_n * 100);
        BARRIER();
        STAMP(3);
        u32 gm;
        {
            const char* ap = myws + AMAX_OFF + (v * 128 + lane) * 8;
            u64 t0 = __builtin_amdgcn_s_memrealtime();
    u32 it = 0;
            for (;;) {
                u64 a0 = ld_sys64(ap), a1 = ld_sys64(ap + 512);
                gm = max((u32)a0, (u32)a1);
                u32 tg = min((u32)(a0 >> 32), (u32)(a1 >> 32));
                if (__builtin_amdgcn_ballot_w64(tg < call) == 0) break;
                if (spin_expired(myws, call, t0, it)) { bad = true; break; }
            }
        }
        gm = wave_umax(gm);
        bad = bad || tmo_set(myws, call);
        if (bad) gm = 0xFFFFFFFFu;
        if (lane == 0) *(LAS u32*)(lds + L_MISC + 32 + v * 4) = gm;
        STAMP(4);
        BARRIER();
        return;
    }

    const int wm = wave >> 1, wn = wave & 1;
    char* amp = p.ws[0];
#pragma unroll
    for (int r = 1; r < NR; ++r) amp = (lane & 7) == r ? p.ws[r] : amp;
    amp += AMAX_OFF + (rank * 128 + g) * 8;
    asm volatile("" : "+v"(amp));
    f32x16 acc[2][2];
#pragma unroll
    for (int i = 0; i < 4; ++i)
#pragma unroll
        for (int j = 0; j < 16; ++j) acc[i >> 1][i & 1][j] = 0.f;
    {
        u32 xb = L_STAGE + wm * 4096 + lane * 16, wb = L_STAGE + (2 + wn) * 4096 + lane * 16;
        volatile LAS u32* prog = (volatile LAS u32*)(lds + L_MISC + 64);
        volatile LAS u32* done = (volatile LAS u32*)(lds + L_MISC + 96 + wave * 4);
        bf16x8 x0[4], w0[4], x1[4], w1[4];
#define SB() __builtin_amdgcn_sched_barrier(0)
#define RD1(X, W, off, f) do { W[f] = *(LAS bf16x8*)(lds + wb + (off) + (f) * 1024); X[f] = *(LAS bf16x8*)(lds + xb + (off) + (f) * 1024); SB(); } while (0)
#define MM1(X, W, i) do { acc[((i) >> 1) & 1][(i) & 1] = __builtin_amdgcn_mfma_f32_32x32x16_bf16( \
            W[((i) >> 2) * 2 + ((i) & 1)], X[(i) >> 1], acc[((i) >> 1) & 1][(i) & 1], 0, 0, 0); \
            asm volatile("" : "+v"(acc[((i) >> 1) & 1][(i) & 1])); SB(); } while (0)
#define LACKS(need) (((rfl(P) - (u32)(need) * 0x10001u) & 0x80008000u) != 0)
#define AWAIT(j, need) do { \
            if (__builtin_expect(LACKS(need), 0)) { do { P = prog[(j) & 3]; } while (LACKS(need)); } \
            asm volatile("" ::: "memory"); SB(); } while (0)
#define RELEASE(j, t) do { asm volatile("" ::: "memory"); *done = (t) + 1; P = prog[((j) + 1) & 3]; SB(); } while (0)
#define STEP(X, W, XN, WN, j) do { constexpr u32 o_ = ((j) & 3) * 16384; \
            MM1(X, W, 0); AWAIT(j, (t4 >> 2) + ((j) >> 2) + 1); MM1(X, W, 1); \
            if ((j) == 4) { xb ^= 65536u; wb ^= 65536u; } \
            RD1(XN, WN, o_, 0); RD1(XN, WN, o_, 1); MM1(X, W, 2); \
            RD1(XN, WN, o_, 2); RD1(XN, WN, o_, 3); MM1(X, W, 3); RELEASE(j, t4 + (j)); \
            MM1(X, W, 4); MM1(X, W, 5); MM1(X, W, 6); MM1(X, W, 7); } while (0)
        if (threadIdx.x < 12) *(LAS u32*)(lds + L_MISC + 64 + threadIdx.x * 4) = 0;
        BARRIER();
        if (threadIdx.x == 0 && call) __hip_atomic_store(ctr, call, __ATOMIC_RELAXED, __HIP_MEMORY_SCOPE_AGENT);
        u32 P = 0;
        AWAIT(0, 1);
        RD1(x0, w0, 0, 0); RD1(x0, w0, 0, 1); RD1(x0, w0, 0, 2); RD1(x0, w0, 0, 3);
        RELEASE(0, 0);
        int t4 = 0;
        for (; t4 < 2 * NSTEP - 4; t4 += 4) {
            STEP(x0, w0, x1, w1, 1); STEP(x1, w1, x0, w0, 2); STEP(x0, w0, x1, w1, 3); STEP(x1, w1, x0, w0, 4);
        }
        STEP(x0, w0, x1, w1, 1); STEP(x1, w1, x0, w0, 2); STEP(x0, w0, x1, w1, 3);
#pragma unroll
        for (int i = 0; i < 8; ++i) MM1(x1, w1, i);
    }
    STAMP(2);
    u32 y[64];
    u32 mx = 0;
#pragma unroll
    for (int e = 0; e < 64; ++e) {
        y[e] = (u32)max((int)__float_as_uint(acc[e >> 5][e & 1][(e >> 1) & 15]), 0);
        mx = max(mx, y[e]);
    }
    mx = wave_umax(mx);
    if (lane == 0) *(LAS u32*)(lds + L_MISC + 16 + wave * 4) = mx;
    BARRIER();
    if (wave == 0) {
        u32x4 wx = *(LAS u32x4*)(lds + L_MISC + 16);
        u32 tm = max(max(wx[0], wx[1]), max(wx[2], wx[3]));
        if (lane < NR) st_sys64(amp, ((u64)call << 32) | tm);
        STAMP(6);
    }

    u32 mk0 = 0, mk1 = 0;
#pragma unroll
    for (int e = 31; e >= 0; --e) {
        mk0 = __builtin_amdgcn_alignbit(mk0, 0u - y[e], 31);
        mk1 = __builtin_amdgcn_alignbit(mk1, 0u - y[e + 32], 31);
    }
    const u32 cnt = __builtin_popcount(mk0) + __builtin_popcount(mk1);
    const u32 incl = wave_incl_scan(cnt);
    const u32 total = __builtin_amdgcn_readlane(incl, 63);
    const int uq = g * 4 + wave;
    const int rot = uq % 7;
    const u32 uoff = P2_OFF + (rank * 512 + uq) * P2_B;
    char* pd[7];
#pragma unroll
    for (int k = 0; k < 7; ++k) pd[k] = p.peer[rot + k];
    asm volatile("" : "+s"(pd[0]), "+s"(pd[1]), "+s"(pd[2]), "+s"(pd[3]), "+s"(pd[4]), "+s"(pd[5]), "+s"(pd[6]));
    {
        auto s0 = __builtin_amdgcn_permlane32_swap(mk0, mk0, false, false);
        auto s1 = __builtin_amdgcn_permlane32_swap(mk1, mk1, false, false);
        u32x4 mv = {s0[0], s1[0], s0[1], s1[1]};
        if (lane < 32) {
#pragma unroll
            for (int k = 0; k < 7; ++k) st_wt128s(pd[k], uoff + lane * 16, mv);
        }
    }
    LAS char* cb = lds + L_COMP + wave * 16384;
    {
        u32 pos = L_COMP + wave * 16384 + (incl - cnt) * 4;
#define C1(n) "v_cmpx_lt_i32_e32 vcc, 0, %" #n "\n\tds_write_b32 %0, %" #n "\n\tv_add_u32 %0, 4, %0\n\ts_mov_b64 exec, -1\n\t"
#pragma unroll
        for (int e = 0; e < 64; e += 8)
            asm volatile(C1(1) C1(2) C1(3) C1(4) C1(5) C1(6) C1(7) C1(8)
                         : "+v"(pos)
                         : "v"(y[e]), "v"(y[e + 1]), "v"(y[e + 2]), "v"(y[e + 3]), "v"(y[e + 4]), "v"(y[e + 5]),
                           "v"(y[e + 6]), "v"(y[e + 7])
                         : "memory", "vcc");
    }
    f32x4 cv[4][4];
#pragma unroll
    for (int r = 0; r < 4; ++r)
#pragma unroll
        for (int j = 0; j < 4; ++j) cv[r][j] = *(LAS f32x4*)(cb + r * 4096 + lane * 64 + j * 16);
    STAMP(3);
    BARRIER();
    float inv;
    {
        u32x4 gv = *(LAS u32x4*)(lds + L_MISC + 32), gw = *(LAS u32x4*)(lds + L_MISC + 48);
        float amax = __uint_as_float(max(max(max(gv[0], gv[1]), max(gv[2], gv[3])), max(max(gw[0], gw[1]), max(gw[2], gw[3]))));
        inv = 1.0f / (amax / 448.0f);
    }
    STAMP(4);
    u32x4 q[4];
#pragma unroll
    for (int r = 0; r < 4; ++r)
#pragma unroll
        for (int j = 0; j < 4; ++j) {
            f32x4 t = cv[r][j] * inv;
            int w = __builtin_amdgcn_cvt_pk_fp8_f32(t[0], t[1], 0, false);
            q[r][j] = (u32)__builtin_amdgcn_cvt_pk_fp8_f32(t[2], t[3], w, true);
        }
#pragma unroll
    for (int k = 0; k < 7; ++k) {
        char* d = pd[k];
#pragma unroll
        for (int r = 0; r < 4; ++r)
            if ((u32)(lane * 16 + r * 1024) < total) st_wt128s(d, uoff + 512 + r * 1024 + lane * 16, q[r]);
        if (k == 0) {
            u32x4 ot[4];
#pragma unroll
            for (int mt = 0; mt < 4; ++mt) {
                u32x4 ov;
#pragma unroll
                for (int j = 0; j < 4; ++j) {
                    const int e = mt * 16 + j * 4;
                    int w = __builtin_amdgcn_cvt_pk_fp8_f32(__uint_as_float(y[e]) * inv, __uint_as_float(y[e + 1]) * inv, 0, false);
                    ov[j] = (u32)__builtin_amdgcn_cvt_pk_fp8_f32(__uint_as_float(y[e + 2]) * inv, __uint_as_float(y[e + 3]) * inv, w, true);
                }
                ot[mt] = ov;
            }
            store_tile(p.out + (size_t)((rank >> 2) * 2048 + tr * 128 + wm * 64) * NN + (rank & 3) * 1024 + tc * 128 + wn * 64,
                       ot, lane);
        }
        WAIT_VM0();
        if (lane == 0) st_sys64(d + F2_OFF + (rank * 512 + uq) * FL_B, ((u64)call << 32) | total);
    }
    STAMP(5);
}

extern "C" void kernel_launch(int rank, int nranks, ncclComm_t comm, hipStream_t stream,
                              void* const* d_in, const int* in_sizes, int n_in,
                              void* d_out, int out_size, const int* in_dtypes, int out_dtype,
                              void* const* d_out_all, void* const* d_ws_all, size_t ws_bytes) {
    (void)comm; (void)in_dtypes; (void)out_dtype; (void)d_out_all;
    if (nranks != NR || (unsigned)rank >= (unsigned)NR || n_in < 2 || ws_bytes < WS_USED ||
        in_sizes[0] != MM * KS || in_sizes[1] != KS * NN || out_size != MM * NN) {
        fprintf(stderr, "kernel_launch: unexpected problem shape\n");
        return;
    }
    HIP_CHECK(hipFuncSetAttribute((const void*)fused_kernel, hipFuncAttributeMaxDynamicSharedMemorySize, LDS_BYTES));
    Params p;
    p.x = (const char*)d_in[0];
    p.w = (const char*)d_in[1];
    p.out = (char*)d_out;
    for (int r = 0; r < NR; ++r) p.ws[r] = (char*)d_ws_all[r];
    for (int k = 0; k < 14; ++k) p.peer[k] = p.ws[(rank + 1 + k % 7) & 7];
    p.rank = rank;
    p.pad = 0;
    fused_kernel<<<dim3(256), dim3(768), LDS_BYTES, stream>>>(p);
    HIP_CHECK(hipGetLastError());
}
```

```cpp
#include <hip/hip_runtime.h>
#include <rccl/rccl.h>
#include <cstdio>
#include <cstdint>

#define HIP_CHECK(expr) do { hipError_t _e = (expr); if (_e != hipSuccess) { \
    fprintf(stderr, "HIP error %s at %s:%d\n", hipGetErrorString(_e), __FILE__, __LINE__); \
    return; } } while (0)

typedef unsigned u32;
typedef unsigned long long u64;
typedef u32 u32x2 __attribute__((ext_vector_type(2)));
typedef u32 u32x3 __attribute__((ext_vector_type(3)));
typedef u32 u32x4 __attribute__((ext_vector_type(4)));
typedef float f32x2 __attribute__((ext_vector_type(2)));
typedef float f32x4 __attribute__((ext_vector_type(4)));
typedef float f32x16 __attribute__((ext_vector_type(16)));
typedef unsigned short u16x2 __attribute__((ext_vector_type(2)));
typedef __bf16 bf16x8 __attribute__((ext_vector_type(8)));
#define GAS __attribute__((address_space(1)))
#define LAS __attribute__((address_space(3)))
#define DI __device__ __forceinline__

constexpr int NR = 8;
constexpr int MM = 4096, NN = 4096, KS = 512;
constexpr int NCH = 16;
constexpr int NSLOT = 48;
constexpr u32 NITEM = NCH * 128;
#define NEARLY 7
#define NPUSH 64
constexpr int NSTEP = 64;
constexpr u32 UNIT_B = 3072, RAWU_B = 4096, FL_B = 128;
constexpr u32 CTR_OFF = 0;
constexpr u32 TMO_OFF = 32768;
constexpr u32 TKT_OFF = 33024;
constexpr u32 AMAX_OFF = 65536;
constexpr u32 F1_OFF = 131072;
constexpr u32 F2_OFF = 1u << 20;
constexpr u32 DBG_OFF = 1572864;
constexpr u32 P1_OFF = 2u << 20;
constexpr u32 RAW_OFF = 20u << 20;
constexpr u32 P2_OFF = 44u << 20;
constexpr u32 P2_B = 4608;
constexpr u32 WS_USED = 62u << 20;
constexpr u32 SPARE_OFF = 64u << 20;
static_assert(F1_OFF + NR * NCH * NSLOT * FL_B <= F2_OFF, "");
static_assert(F2_OFF + NR * 512 * FL_B <= DBG_OFF, "");
static_assert(P1_OFF + NR * NCH * NSLOT * UNIT_B <= RAW_OFF, "");
static_assert(RAW_OFF + NR * NCH * NSLOT * RAWU_B <= P2_OFF, "");
static_assert(P2_OFF + NR * 512 * P2_B <= WS_USED, "");
constexpr u64 DEADLINE = 50000000ull;
constexpr int MAXEXC = 8;
constexpr u32 HDR_RAW = 0x1000;
constexpr int MAXE2 = 128;
constexpr u32 H2_RAW = 1u << 28;

constexpr u32 L_STAGE = 0;
constexpr u32 L_COMP = 0;
constexpr u32 L_MISC = 131072;
constexpr u32 L_DEC = 0;
constexpr u32 DEC_B = 4352;
constexpr u32 L_LUT = 53248;
constexpr u32 L_SCR = 57344;
constexpr u32 LDS_BYTES = 131072 + 512;

struct Params {
    const char* x;
    const char* w;
    char* out;
    char* ws[NR];
    char* peer[14];
    char* wsd[NR];
    int rank;
    int pad;
};

#define PAY_AUX 17
#define ACC(var) do {} while (0)
#define PUT(i, val) do {} while (0)
#define STAMP(i) do {} while (0)

#define PSTAMP() do {} while (0)
DI u32 perm(u32 hi, u32 lo, u32 sel) { return __builtin_amdgcn_perm(hi, lo, sel); }
DI u32 bfi(u32 m, u32 a, u32 b) { return (m & a) | (~m & b); }
DI u32 rotr(u32 v, u32 n) { return __builtin_amdgcn_alignbit(v, v, n); }
DI u32 rfl(u32 v) { return __builtin_amdgcn_readfirstlane(v); }
template <int CTRL, int RM> DI u32 dpp0(u32 v) { return __builtin_amdgcn_update_dpp(0u, v, CTRL, RM, 0xF, false); }
DI u32 wave_umax(u32 v) {
    v = max(v, dpp0<0x111, 0xF>(v)); v = max(v, dpp0<0x112, 0xF>(v));
    v = max(v, dpp0<0x114, 0xF>(v)); v = max(v, dpp0<0x118, 0xF>(v));
    v = max(v, dpp0<0x142, 0xA>(v)); v = max(v, dpp0<0x143, 0xC>(v));
    return __builtin_amdgcn_readlane(v, 63);
}
DI u32 wave_incl_scan(u32 v) {
    v += dpp0<0x111, 0xF>(v); v += dpp0<0x112, 0xF>(v);
    v += dpp0<0x114, 0xF>(v); v += dpp0<0x118, 0xF>(v);
    v += dpp0<0x142, 0xA>(v); v += dpp0<0x143, 0xC>(v);
    return v;
}
DI u64 ld_sys64(const char* p) {
    return __hip_atomic_load((GAS u64*)p, __ATOMIC_RELAXED, __HIP_MEMORY_SCOPE_SYSTEM);
}
DI void st_sys64(char* p, u64 v) {
    __hip_atomic_store((GAS u64*)p, v, __ATOMIC_RELAXED, __HIP_MEMORY_SCOPE_SYSTEM);
}
DI void st_wt128(char* p, u32x4 v) {
    asm volatile("global_store_dwordx4 %0, %1, off sc0 sc1\n\ts_nop 1" :: "v"(p), "v"(v) : "memory");
}
DI void st_wt128s(const char* sbase, u32 voff, u32x4 v) {
    asm volatile("global_store_dwordx4 %0, %1, %2 sc0 sc1\n\ts_nop 1" :: "v"(voff), "v"(v), "s"(sbase) : "memory");
}
#define WAIT_VM0() asm volatile("s_waitcnt vmcnt(0)" ::: "memory")
#define WAIT_LGKM0() asm volatile("s_waitcnt lgkmcnt(0)" ::: "memory")
#define BARRIER() do { WAIT_LGKM0(); __builtin_amdgcn_s_barrier(); asm volatile("" ::: "memory"); } while (0)

DI bool spin_expired(char* myws, u32 call, u64 t0, u32& it) {
    if ((++it & 15u) == 0) {
        GAS u32* tmo = (GAS u32*)(myws + TMO_OFF);
        u32 t = rfl(__hip_atomic_load(tmo, __ATOMIC_RELAXED, __HIP_MEMORY_SCOPE_AGENT));
        if (t == call || __builtin_amdgcn_s_memrealtime() - t0 > DEADLINE) {
            __hip_atomic_store(tmo, call, __ATOMIC_RELAXED, __HIP_MEMORY_SCOPE_AGENT);
            return true;
        }
    }
    __builtin_amdgcn_s_sleep(1);
    return false;
}
DI bool tmo_set(char* myws, u32 call) {
    return rfl(__hip_atomic_load((GAS u32*)(myws + TMO_OFF), __ATOMIC_RELAXED, __HIP_MEMORY_SCOPE_AGENT)) == call;
}

DI u32 pack_unit(const u32x4 (&R)[4], u32x4& p0, u32x4& p1, u32x4& p2, LAS char* scr, int lane,
                 char* d0, char* d1, char* d2, char* d3, char* fl, int nd, u32 unit) {
    u32 L[8], H[8];
    u16x2 mx = {0, 0};
#pragma unroll
    for (int f = 0; f < 4; ++f) {
        L[2 * f] = perm(R[f][1], R[f][0], 0x06040200u); H[2 * f] = perm(R[f][1], R[f][0], 0x07050301u);
        L[2 * f + 1] = perm(R[f][3], R[f][2], 0x06040200u); H[2 * f + 1] = perm(R[f][3], R[f][2], 0x07050301u);
#pragma unroll
        for (int d = 0; d < 4; ++d)
            mx = __builtin_elementwise_max(mx, __builtin_bit_cast(u16x2, R[f][d] & 0x7FFF7FFFu));
    }
    u32 top = wave_umax(max((u32)mx[0], (u32)mx[1])) >> 8;
    u32 base = top > 7u ? top - 7u : 0u;
    u32 base4 = base * 0x01010101u;
    u32 T[8], all = 0x80808080u;
#pragma unroll
    for (int i = 0; i < 8; ++i) { T[i] = (H[i] | 0x80808080u) - base4; all &= T[i]; }
    u32 W[4];
#pragma unroll
    for (int f = 0; f < 4; ++f) {
        u32 na = bfi(0x80808080u, H[2 * f], T[2 * f]), nb = bfi(0x80808080u, H[2 * f + 1], T[2 * f + 1]);
        W[f] = bfi(0x87878787u, na, rotr(nb, 28));
    }
    p0 = u32x4{L[0], L[1], L[2], L[3]};
    p1 = u32x4{L[4], L[5], L[6], L[7]};
    p2 = u32x4{W[0], W[1], W[2], W[3]};
    u32 hdr = base;
    if (__builtin_amdgcn_ballot_w64(all != 0x80808080u) != 0) {
#pragma unroll
        for (int f = 0; f < 4; ++f) *(LAS u32x4*)(scr + f * 1024 + lane * 16) = R[f];
        u32 m = 0;
#pragma unroll
        for (int i = 0; i < 8; ++i) m |= (~T[i] & 0x80808080u) >> (7 - i);
        u32 cnt = 0;
        for (;;) {
            u64 bal = __builtin_amdgcn_ballot_w64(m != 0);
            if (bal == 0 || cnt > MAXEXC) break;
            if (m != 0) {
                u32 bp = __builtin_ctz(m), i = bp & 7, b = bp >> 3;
                u32 addr = (i >> 1) * 1024 + lane * 16 + ((i & 1) * 4 + b) * 2;
                u32 v = *(LAS unsigned short*)(scr + addr);
                u32 slot = cnt + __builtin_amdgcn_mbcnt_hi((u32)(bal >> 32), __builtin_amdgcn_mbcnt_lo((u32)bal, 0));
                if (slot < MAXEXC) *(LAS u32*)(scr + 4096 + slot * 4) = (addr << 16) | v;
                m &= m - 1;
            }
            cnt += __builtin_popcountll(bal);
        }
        if (cnt > MAXEXC) {
            hdr |= HDR_RAW;
#pragma unroll
            for (int f = 0; f < 4; ++f) {
                const u32 o = RAW_OFF + unit * RAWU_B + f * 1024 + lane * 16;
                st_wt128s(d0, o, R[f]);
                st_wt128s(d1, o, R[f]);
                if (nd > 2) { st_wt128s(d2, o, R[f]); st_wt128s(d3, o, R[f]); }
            }
        } else {
            hdr |= cnt << 8;
            if (lane < 8 && (lane & 3) < nd) {
                u32x4 rec = *(LAS u32x4*)(scr + 4096 + (lane & 4) * 4);
                st_wt128(fl + 64 + (lane & 4) * 4, rec);
            }
        }
    }
    return hdr;
}

#define A_TICKET(tk, adr) "s_nop 4\n\ts_mov_b64 exec, 1\n\tglobal_atomic_add %" #tk ", %" #adr ", %" #tk ", off sc0\n\ts_mov_b64 exec, -1\n\t"
#define A_ST(so, v, d, off) "global_store_dwordx4 %" #so ", %" #v ", %" #d " offset:" #off " sc0 sc1\n\t"
#define A_ST4(so, a, b, c, d0, d1, d2, d3) \
    A_ST(so, a, d0, 0) A_ST(so, a, d1, 0) A_ST(so, a, d2, 0) A_ST(so, a, d3, 0) \
    A_ST(so, b, d0, 1024) A_ST(so, b, d1, 1024) A_ST(so, b, d2, 1024) A_ST(so, b, d3, 1024) \
    A_ST(so, c, d0, 2048) A_ST(so, c, d1, 2048) A_ST(so, c, d2, 2048) A_ST(so, c, d3, 2048) \
    "s_nop 1\n\ts_waitcnt vmcnt(12)"
#define A_ST2(so, a, b, c, d0, d1) \
    A_ST(so, a, d0, 0) A_ST(so, a, d1, 0) A_ST(so, b, d0, 1024) A_ST(so, b, d1, 1024) \
    A_ST(so, c, d0, 2048) A_ST(so, c, d1, 2048) "s_nop 1\n\ts_waitcnt vmcnt(6)"
#define A_LDX "global_load_dwordx4 %0, %5, %7\n\tglobal_load_dwordx4 %1, %6, %7\n\t" \
              "global_load_dwordx4 %2, %5, %7 offset:32\n\tglobal_load_dwordx4 %3, %6, %7 offset:32\n\t"
#define A_WL(i) "global_load_dword %" #i ", %17, %18\n\tv_add_u32 %17, 0x2000, %17\n\t"
#define A_LDW A_WL(0) A_WL(1) A_WL(2) A_WL(3) A_WL(4) A_WL(5) A_WL(6) \
              "global_load_dword %7, %17, %18\n\tv_add_u32 %17, 0x12000, %17\n\t" \
              A_WL(8) A_WL(9) A_WL(10) A_WL(11) A_WL(12) A_WL(13) A_WL(14) "global_load_dword %15, %17, %18\n\t"
#define X_OUT "=&v"(R[0]), "=&v"(R[1]), "=&v"(R[2]), "=&v"(R[3]), "+v"(tk)
#define X_IN "v"(lx0), "v"(lx1), "s"(src), "v"(tka), "v"(so), "v"(p0), "v"(p1), "v"(p2)
#define W_OUT "=&v"(G[0]), "=&v"(G[1]), "=&v"(G[2]), "=&v"(G[3]), "=&v"(G[4]), "=&v"(G[5]), "=&v"(G[6]), \
              "=&v"(G[7]), "=&v"(G[8]), "=&v"(G[9]), "=&v"(G[10]), "=&v"(G[11]), "=&v"(G[12]), "=&v"(G[13]), \
              "=&v"(G[14]), "=&v"(G[15]), "+v"(tk), "+v"(lw)
#define W_IN "s"(src), "v"(tka), "v"(so), "v"(p0), "v"(p1), "v"(p2)

DI void w_to_frags(const u32 (&G)[16], u32x4 (&R)[4]) {
#pragma unroll
    for (int f = 0; f < 4; ++f)
#pragma unroll
        for (int d = 0; d < 4; ++d)
            R[f][d] = perm(G[(f >> 1) * 8 + 2 * d + 1], G[(f >> 1) * 8 + 2 * d], (f & 1) ? 0x07060302u : 0x05040100u);
}

struct Item { bool isw; u32 unit; const char* src; char *d0, *d1, *d2, *d3; };
DI Item make_item(const Params& p, u32 i) {
    Item it;
    const u32 c = (i >> 7) & (NCH - 1), k = (i >> 1) & 63;
    it.isw = i & 1;
    const u32 r0 = it.isw ? (k & 3) : (k & 1) * 4, rs = it.isw ? 4 : 1;
    const u32 sl = it.isw ? k >> 2 : k >> 1;
    it.unit = (p.rank * NCH + c) * NSLOT + (it.isw ? 32 : 0) + sl;
    it.src = it.isw ? p.w + (size_t)c * (32 * NN * 2) + ((k & 3) * 16 + sl) * 128
                    : p.x + (size_t)((k & 1) * 32 + sl) * (64 * KS * 2) + c * 64;
    it.d0 = p.wsd[r0]; it.d1 = p.wsd[r0 + rs]; it.d2 = p.wsd[(r0 + 2 * rs) & 7]; it.d3 = p.wsd[(r0 + 3 * rs) & 7];
    return it;
}

DI u32 lane_x(int lane) { return (u32)(lane & 31) * (KS * 2) + (lane >> 5) * 16; }
DI u32 lane_w(int lane) {
    return (u32)(8 * (lane >> 5)) * (NN * 2) + ((lane & 19) | ((lane & 4) << 1) | ((lane & 8) >> 1)) * 4;
}

DI void push_inputs(const Params& p, u32 g, u32 ctr_old, LAS char* lds, int lane) {
    const u32 lx0 = lane_x(lane), lx1 = lx0 + 32 * KS * 2, lw0 = lane_w(lane);
    char* const tka = p.ws[p.rank] + TKT_OFF;
    Item cur = make_item(p, g), nxt = make_item(p, NPUSH + g);
    u32 G[16];
    u32x4 R[4], p0, p1, p2;
    if (cur.isw) {
#pragma unroll
        for (int j = 0; j < 16; ++j) G[j] = *(const GAS u32*)(cur.src + lw0 + ((j >> 3) * 16 + (j & 7)) * (NN * 2));
        w_to_frags(G, R);
    } else {
#pragma unroll
        for (int f = 0; f < 4; ++f) R[f] = *(const GAS u32x4*)(cur.src + ((f & 1) ? lx1 : lx0) + (f >> 1) * 32);
    }
    const u32 call = rfl(ctr_old) + 1u;
    const u32 tbase = (call - 1u) * NITEM - 2u * NPUSH;
    bool more = true;
    char* fp = nullptr;
    u32 fv = 0;
    int fn = 0;
    for (;;) {
        const int nd = cur.isw ? 2 : 4;
        char* const dl = (lane & 1) ? cur.d1 : cur.d0;
        char* const dh = (lane & 1) ? cur.d3 : cur.d2;
        char* const fl = ((lane & 2) ? dh : dl) + F1_OFF + cur.unit * FL_B;
        const u32 hdr = pack_unit(R, p0, p1, p2, lds + L_SCR, lane, cur.d0, cur.d1, cur.d2, cur.d3, fl, nd, cur.unit);
        const u32 so = P1_OFF + cur.unit * UNIT_B + lane * 16;
        const char* src = nxt.src;
        u32 tk = 1;
        if (nxt.isw) {
            u32 lw = lw0;
            if (cur.isw)
                asm volatile(A_TICKET(16, 19) A_LDW A_ST2(20, 21, 22, 23, 24, 25)
                             : W_OUT : W_IN, "s"(cur.d0), "s"(cur.d1) : "memory");
            else
                asm volatile(A_TICKET(16, 19) A_LDW A_ST4(20, 21, 22, 23, 24, 25, 26, 27)
                             : W_OUT : W_IN, "s"(cur.d0), "s"(cur.d1), "s"(cur.d2), "s"(cur.d3) : "memory");
            w_to_frags(G, R);
        } else {
            if (cur.isw)
                asm volatile(A_TICKET(4, 8) A_LDX A_ST2(9, 10, 11, 12, 13, 14)
                             : X_OUT : X_IN, "s"(cur.d0), "s"(cur.d1) : "memory");
            else
                asm volatile(A_TICKET(4, 8) A_LDX A_ST4(9, 10, 11, 12, 13, 14, 15, 16)
                             : X_OUT : X_IN, "s"(cur.d0), "s"(cur.d1), "s"(cur.d2), "s"(cur.d3) : "memory");
        }
        if (lane < fn) st_sys64(fp, ((u64)call << 32) | fv);
        fp = fl;
        fv = hdr;
        fn = nd;
        if (!more) break;
        const u32 i = rfl(tk) - tbase;
        cur = nxt;
        more = i < NITEM;
        nxt = make_item(p, i);
    }
    WAIT_VM0();
    if (lane < fn) st_sys64(fp, ((u64)call << 32) | fv);
}

DI void pack_own(const Params& p, u32 s, u32 ctr_old, LAS char* scr, int lane) {
    const u32 c = s / NSLOT, sl = s % NSLOT;
    u32x4 R[4], p0, p1, p2;
    if (sl >= 32) {
        const char* src = p.w + (size_t)c * (32 * NN * 2) + ((p.rank & 3) * 16 + sl - 32) * 128 + lane_w(lane);
        u32 G[16];
#pragma unroll
        for (int j = 0; j < 16; ++j) G[j] = *(const GAS u32*)(src + ((j >> 3) * 16 + (j & 7)) * (NN * 2));
        w_to_frags(G, R);
    } else {
        const char* src = p.x + (size_t)((p.rank >> 2) * 32 + sl) * (64 * KS * 2) + c * 64 + lane_x(lane);
#pragma unroll
        for (int f = 0; f < 4; ++f) R[f] = *(const GAS u32x4*)(src + (f & 1) * (32 * KS * 2) + (f >> 1) * 32);
    }
    char* const me = p.ws[p.rank];
    const u32 unit = p.rank * (NCH * NSLOT) + s;
    char* const fl = me + F1_OFF + unit * FL_B;
    const u32 hdr = pack_unit(R, p0, p1, p2, scr, lane, me, me, me, me, fl, 1, unit);
    const u32 so = P1_OFF + unit * UNIT_B + lane * 16;
    st_wt128s(me, so, p0); st_wt128s(me, so + 1024, p1); st_wt128s(me, so + 2048, p2);
    WAIT_VM0();
    if (lane == 0) st_sys64(fl, ((u64)(ctr_old + 1u) << 32) | hdr);
}

typedef __amdgpu_buffer_rsrc_t rsrc_t;
struct PSet { u32x4 pay[6]; u32 rec; u32 hdr0, hdr1; u32 unit; };

DI u32x2 load_flags(rsrc_t rs, u32 voff, u32 unit) {
    return __builtin_amdgcn_raw_buffer_load_b64(rs, voff, F1_OFF + unit * FL_B, 17);
}
DI bool wait_flags(u32x2& F0, u32x2& Fn, rsrc_t rs, char* myws, u32 unit, u32 next, int lane, u32 call) {
    u64 t0 = __builtin_amdgcn_s_memrealtime();
    u32 it = 0;
    bool bad = false;
    const u32 voff = F1_OFF + (((lane & 2) ? next : unit) + (lane & 1)) * FL_B;
    u32x2 F;
    for (;;) {
        asm volatile("" ::: "memory");
        F = __builtin_amdgcn_raw_buffer_load_b64(rs, voff, 0, 17);
        if ((__builtin_amdgcn_ballot_w64(F[1] < call) & 3) == 0) break;
        if (spin_expired(myws, call, t0, it)) { bad = true; break; }
    }
    F0 = F;
    Fn[0] = __builtin_amdgcn_update_dpp(0u, F[0], 0xEE, 0xF, 0xF, false);
    Fn[1] = __builtin_amdgcn_update_dpp(0u, F[1], 0xEE, 0xF, 0xF, false);
    return bad;
}
DI void issue_set(PSet& S, u32x2 F, rsrc_t rs, u32 unit, int lane) {
    asm volatile("" ::: "memory");
    S.hdr0 = __builtin_amdgcn_readlane(F[0], 0);
    S.hdr1 = __builtin_amdgcn_readlane(F[0], 1);
    S.unit = unit;
#pragma unroll
    for (int j = 0; j < 6; ++j)
        S.pay[j] = __builtin_amdgcn_raw_buffer_load_b128(rs, lane * 16 + j * 1024, P1_OFF + unit * UNIT_B, PAY_AUX);
    S.rec = __builtin_amdgcn_raw_buffer_load_b32(rs, (lane & 7) * 4 + 64 + (lane & 8) * (FL_B / 8), F1_OFF + unit * FL_B, 17);
}
DI void unpack_set(const PSet& S, rsrc_t rs, LAS char* o, int lane) {
#pragma unroll
    for (int u = 0; u < 2; ++u) {
        const u32 base4 = ((u ? S.hdr1 : S.hdr0) & 0x7F) * 0x01010101u;
#pragma unroll
        for (int f = 0; f < 4; ++f) {
            u32 w = S.pay[3 * u + 2][f];
            u32 h0 = (w & 0x87878787u) + base4, h1 = (rotr(w, 4) & 0x87878787u) + base4;
            u32 l0 = S.pay[3 * u + (f >> 1)][(f & 1) * 2], l1 = S.pay[3 * u + (f >> 1)][(f & 1) * 2 + 1];
            u32x4 r = {perm(h0, l0, 0x05010400u), perm(h0, l0, 0x07030602u),
                       perm(h1, l1, 0x05010400u), perm(h1, l1, 0x07030602u)};
            *(LAS u32x4*)(o + u * 4096 + f * 1024 + lane * 16) = r;
            __builtin_amdgcn_sched_barrier(0);
        }
    }
    if (__builtin_expect((S.hdr0 | S.hdr1) >= 0x100u, 0)) {
        asm volatile("" ::: "memory");
        const u32 hdr = (lane & 8) ? S.hdr1 : S.hdr0;
        if (lane < 16 && (u32)(lane & 7) < ((hdr >> 8) & 15u))
            *(LAS unsigned short*)(o + (lane & 8) * 512 + ((S.rec >> 16) & 0xFFE)) = (unsigned short)S.rec;
        if (__builtin_expect(((S.hdr0 | S.hdr1) & HDR_RAW) != 0, 0)) {
#pragma unroll
            for (int u = 0; u < 2; ++u)
                if ((u ? S.hdr1 : S.hdr0) & HDR_RAW) {
#pragma unroll
                    for (int f = 0; f < 4; ++f)
                        *(LAS u32x4*)(o + u * 4096 + f * 1024 + lane * 16) = __builtin_amdgcn_raw_buffer_load_b128(
                            rs, lane * 16 + f * 1024, RAW_OFF + (S.unit + u) * RAWU_B, 17);
                }
        }
    }
}

DI void store_tile(char* o, const u32x4 (&v)[4], int lane) {
    o += (size_t)(lane & 15) * NN + ((lane >> 4) & 1) * 32 + (lane >> 5) * 16;
#pragma unroll
    for (int mi = 0; mi < 2; ++mi) {
        u32x4 a, b;
#pragma unroll
        for (int j = 0; j < 4; ++j) {
            auto r = __builtin_amdgcn_permlane16_swap(v[2 * mi][j], v[2 * mi + 1][j], false, false);
            a[j] = r[0]; b[j] = r[1];
        }
        st_wt128(o + (size_t)(32 * mi) * NN, a);
        st_wt128(o + (size_t)(32 * mi + 16) * NN, b);
    }
}

__global__ __launch_bounds__(768, 3) void fused_kernel(Params p) {
    extern __shared__ __attribute__((aligned(16))) char smem_[];
    LAS char* lds = (LAS char*)smem_;
    const int lane = threadIdx.x & 63;
    const int wave = rfl(threadIdx.x >> 6);
    const int b = blockIdx.x, xcd = b & 7, idx = b >> 3;
    const int rank = p.rank;
    char* myws = p.ws[rank];
    STAMP(0);

    GAS u32* ctr = (GAS u32*)(myws + CTR_OFF + b * 128);
    const u32 ctr_old = __hip_atomic_load(ctr, __ATOMIC_RELAXED, __HIP_MEMORY_SCOPE_AGENT);

    if (idx < 16) {
        const int g = xcd * 16 + idx;
        if (threadIdx.x >= 256 && threadIdx.x < 512) {
            u32 m = threadIdx.x - 256, n = 0, s[2] = {0, 0};
#pragma unroll
            for (int e = 0; e < 8; ++e) {
                u32 bit = (m >> e) & 1;
                s[e >> 2] |= (bit ? n : 0x0Cu) << ((e & 3) * 8);
                n += bit;
            }
            *(LAS u32x2*)(lds + L_LUT + m * 8) = u32x2{s[0], s[1]};
        }
        if (wave == 0) { if (idx < NPUSH / 8) push_inputs(p, idx * 8 + xcd, ctr_old, lds, lane); }
        else if (wave <= 6) pack_own(p, (wave - 1) * 128 + g, ctr_old, lds + L_SCR + wave * DEC_B, lane);
        const u32 call = rfl(ctr_old) + 1u;
        STAMP(1);
        BARRIER();
        if (threadIdx.x == 64) __hip_atomic_store(ctr, call, __ATOMIC_RELAXED, __HIP_MEMORY_SCOPE_AGENT);
        const rsrc_t rs = __builtin_amdgcn_make_buffer_rsrc(myws, 0, WS_USED, 0x00020000);
        const int uq = g * 4 + (wave & 3);
        const int rot = uq % 7;
        const int tr = (g >> 5) * 4 + ((g >> 2) & 3), tc = ((g >> 4) & 1) * 4 + (g & 3);
        LAS char* buf = lds + L_DEC + wave * DEC_B;
        for (int k = wave >> 2; k < 7; k += 3) {
            const int src = (rank + 7 - (k + rot) % 7) & 7;
            const u32 un = src * 512 + uq;
            bool bad = false;
            u64 F = ld_sys64(myws + F2_OFF + un * FL_B);
            if (rfl((u32)(F >> 32)) < call) {
                u64 t0 = __builtin_amdgcn_s_memrealtime();
    u32 it = 0;
                for (;;) {
                    F = ld_sys64(myws + F2_OFF + un * FL_B);
                    if (rfl((u32)(F >> 32)) >= call) break;
                    if (spin_expired(myws, call, t0, it)) { bad = true; break; }
                }
            }
            asm volatile("" ::: "memory");
            const u32 h2 = rfl((u32)F);
            const u32 total = min(h2 & 0x1FFFu, 4096u), ng = (total + 15) >> 4;
            const u32 so = P2_OFF + un * P2_B;
            const u32x2 mk = __builtin_amdgcn_raw_buffer_load_b64(rs, (lane & 31) * 16 + (lane >> 5) * 8, so, 17);
            if (__builtin_expect((h2 & H2_RAW) != 0, 0)) {
                u32x4 pl[4];
#pragma unroll
                for (int r = 0; r < 4; ++r)
                    pl[r] = __builtin_amdgcn_raw_buffer_load_b128(
                        rs, (u32)(lane + r * 64) < ng ? (u32)(512 + lane * 16 + r * 1024) : 0x80000000u, so, 17);
                bad = bad || tmo_set(myws, call);
#pragma unroll
                for (int r = 0; r < 4; ++r) *(LAS u32x4*)(buf + r * 1024 + lane * 16) = pl[r];
            } else {
                const u32 ne = min((h2 >> 13) & 0xFFu, (u32)MAXE2), w4 = ((h2 >> 21) & 0x3Fu) * 0x01010101u;
                u32x3 pl[4];
#pragma unroll
                for (int r = 0; r < 4; ++r)
                    pl[r] = __builtin_amdgcn_raw_buffer_load_b96(
                        rs, (u32)(lane + r * 64) < ng ? (u32)(512 + lane * 12 + r * 768) : 0x80000000u, so, 17);
                u32 ex[2];
#pragma unroll
                for (int r = 0; r < 2; ++r)
                    ex[r] = __builtin_amdgcn_raw_buffer_load_b16(
                        rs, (u32)(lane + r * 64) < ne ? 512 + ng * 12 + lane * 2 + r * 128 : 0x80000000u, so, 17);
                bad = bad || tmo_set(myws, call);
#pragma unroll
                for (int r = 0; r < 4; ++r)
                    if (r < 2 || (u32)(r * 64) < ng) {
                        const u32 d3 = bfi(0x03030303u, pl[r][0] >> 6, bfi(0x0C0C0C0Cu, pl[r][1] >> 4, (pl[r][2] >> 2) & 0x30303030u));
                        *(LAS u32x4*)(buf + r * 1024 + lane * 16) =
                            u32x4{(pl[r][0] & 0x3F3F3F3Fu) + w4, (pl[r][1] & 0x3F3F3F3Fu) + w4, (pl[r][2] & 0x3F3F3F3Fu) + w4, d3 + w4};
                    }
#pragma unroll
                for (int r = 0; r < 2; ++r)
                    if ((u32)(lane + r * 64) < ne)
                        __hip_atomic_fetch_and((LAS u32*)(buf + (ex[r] & 0xFFCu)), ~(0x40u << ((ex[r] & 3u) * 8)),
                                               __ATOMIC_RELAXED, __HIP_MEMORY_SCOPE_WAVEFRONT);
                asm volatile("" ::: "memory");
            }
            u32 cnt = __builtin_popcount(mk[0]) + __builtin_popcount(mk[1]);
            u32 o = wave_incl_scan(cnt) - cnt;
            u32x4 ot[4];
#pragma unroll
            for (int mt = 0; mt < 4; ++mt) {
                u32x4 ov;
#pragma unroll
                for (int hh = 0; hh < 2; ++hh) {
                    u32 mb = (mk[mt >> 1] >> ((mt & 1) * 16 + hh * 8)) & 0xFF;
                    u32x2 sel = *(LAS u32x2*)(lds + L_LUT + mb * 8);
                    LAS u32* dp = (LAS u32*)(buf + (o & ~3u));
                    u32 d0 = dp[0], d1 = dp[1], d2 = dp[2];
                    u32 a0 = __builtin_amdgcn_alignbyte(d1, d0, o), a1 = __builtin_amdgcn_alignbyte(d2, d1, o);
                    ov[2 * hh] = perm(a1, a0, sel[0]);
                    ov[2 * hh + 1] = perm(a1, a0, sel[1]);
                    o += __builtin_popcount(mb);
                }
                if (bad) ov = u32x4{0x7F7F7F7Fu, 0x7F7F7F7Fu, 0x7F7F7F7Fu, 0x7F7F7F7Fu};
                ot[mt] = ov;
            }
            store_tile(p.out + (size_t)((src >> 2) * 2048 + tr * 128 + ((wave >> 1) & 1) * 64) * NN +
                       (src & 3) * 1024 + tc * 128 + (wave & 1) * 64, ot, lane);
            STAMP(2 + k / 3);
        }
        return;
    }

    const int g = xcd * 16 + (idx - 16);
    const int tr = (g >> 5) * 4 + ((g >> 2) & 3), tc = ((g >> 4) & 1) * 4 + (g & 3);
    const u32 call = rfl(ctr_old) + 1u;

    if (wave >= 4) {
        const int v = wave - 4, ph = v >> 1;
        const int slot = (v & 1) ? 32 + 2 * tc : 2 * tr;
        const rsrc_t rs = __builtin_amdgcn_make_buffer_rsrc(myws, 0, WS_USED, 0x00020000);
        LAS char* oa = lds + L_STAGE + ph * 16384 + (v & 1) * 8192;
        volatile LAS unsigned short* prog = (volatile LAS unsigned short*)(lds + L_MISC + 64 + v * 2);
        volatile LAS u32x4* done = (volatile LAS u32x4*)(lds + L_MISC + 96);
        const u32 fo = (lane & 1) * FL_B;
        u32 utab;
        {
            const u32 t = min(4 * lane + ph, 2 * NSTEP - 1), r = t - NCH;
            utab = (t < NCH ? rank * NCH + t : ((rank + 1 + r % 7) & 7) * NCH + r / 7) * NSLOT + slot;
        }
#define UNIT(i) ((u32)__builtin_amdgcn_readlane(utab, (i)))
        PSet A, B;
        bool bad = false;
        u32x2 F0, Fn;
#define PAYWAIT() do {} while (0)
#define ACC2(x) do {} while (0)
#define STAGE(S, o, t) do { \
            if (__builtin_expect((int)rfl(min(min(D[0], D[1]), min(D[2], D[3]))) < (t) - 7, 0)) { \
                do { __builtin_amdgcn_s_sleep(1); D = *done; } while ((int)rfl(min(min(D[0], D[1]), min(D[2], D[3]))) < (t) - 7); } \
            ACC2(a_bar); PAYWAIT(); \
            unpack_set(S, rs, o, lane); \
            asm volatile("" ::: "memory"); \
            *prog = (unsigned short)(((t) >> 2) + 1); \
            D = *done; ACC2(a_unp); } while (0)
#define TRYFETCH(S, pend, un, nx, valid) do { \
            const bool up_ = (valid) && __builtin_amdgcn_ballot_w64(Fn[1] < call) == 0; \
            pend = (valid) && !up_; \
            F0 = Fn; \
            Fn = load_flags(rs, fo, nx); \
            issue_set(S, F0, __builtin_amdgcn_make_buffer_rsrc(myws, 0, up_ ? WS_USED : 0u, 0x00020000), un, lane); \
            ACC2(a_iss); } while (0)
#define REPAIR(S, pend, un, nx) do { \
            if (pend) { bad |= wait_flags(F0, Fn, rs, myws, un, nx, lane, call); issue_set(S, F0, rs, un, lane); LASTREP(un); } \
            ACC2(a_flag); } while (0)
#define LASTREP(un) do {} while (0)
        bool pa = true, pb;
        u32 ua = UNIT(0), ub = UNIT(1);
        REPAIR(A, pa, ua, ub);
        STAMP(1);
        TRYFETCH(B, pb, ub, UNIT(2), true);
        BARRIER();
        u32x4 D = {0, 0, 0, 0};
        for (int i = 0; i < NSTEP / 2; i += 2) {
            const bool more = i < NSTEP / 2 - 2;
            const int t = 4 * i + ph;
            STAGE(A, oa, t);
            ua = UNIT(i + 2);
            REPAIR(B, pb, ub, ua);
            TRYFETCH(A, pa, ua, UNIT(i + 3), more);
            STAGE(B, oa + 65536, t + 4);
            ub = UNIT(i + 3);
            REPAIR(A, pa, ua, ub);
            TRYFETCH(B, pb, ub, UNIT(i + 4), more);
        }
        STAMP(2);
        PUT(6, lr_t); PUT(13, lr_u * 100); PUT(14, lr_n * 100);
        BARRIER();
        STAMP(3);
        u32 gm;
        {
            const char* ap = myws + AMAX_OFF + (v * 128 + lane) * 8;
            u64 t0 = __builtin_amdgcn_s_memrealtime();
    u32 it = 0;
            for (;;) {
                u64 a0 = ld_sys64(ap), a1 = ld_sys64(ap + 512);
                gm = max((u32)a0, (u32)a1);
                u32 tg = min((u32)(a0 >> 32), (u32)(a1 >> 32));
                if (__builtin_amdgcn_ballot_w64(tg < call) == 0) break;
                if (spin_expired(myws, call, t0, it)) { bad = true; break; }
            }
        }
        gm = wave_umax(gm);
        bad = bad || tmo_set(myws, call);
        if (bad) gm = 0xFFFFFFFFu;
        if (lane == 0) *(LAS u32*)(lds + L_MISC + 32 + v * 4) = gm;
        STAMP(4);
        BARRIER();
        return;
    }

    const int wm = wave >> 1, wn = wave & 1;
    char* amp = p.ws[0];
#pragma unroll
    for (int r = 1; r < NR; ++r) amp = (lane & 7) == r ? p.ws[r] : amp;
    amp += AMAX_OFF + (rank * 128 + g) * 8;
    asm volatile("" : "+v"(amp));
    f32x16 acc[2][2];
#pragma unroll
    for (int i = 0; i < 4; ++i)
#pragma unroll
        for (int j = 0; j < 16; ++j) acc[i >> 1][i & 1][j] = 0.f;
    {
        u32 xb = L_STAGE + wm * 4096 + lane * 16, wb = L_STAGE + (2 + wn) * 4096 + lane * 16;
        volatile LAS u32* prog = (volatile LAS u32*)(lds + L_MISC + 64);
        volatile LAS u32* done = (volatile LAS u32*)(lds + L_MISC + 96 + wave * 4);
        bf16x8 x0[4], w0[4], x1[4], w1[4];
#define SB() __builtin_amdgcn_sched_barrier(0)
#define RD1(X, W, off, f) do { W[f] = *(LAS bf16x8*)(lds + wb + (off) + (f) * 1024); X[f] = *(LAS bf16x8*)(lds + xb + (off) + (f) * 1024); SB(); } while (0)
#define MM1(X, W, i) do { acc[((i) >> 1) & 1][(i) & 1] = __builtin_amdgcn_mfma_f32_32x32x16_bf16( \
            W[((i) >> 2) * 2 + ((i) & 1)], X[(i) >> 1], acc[((i) >> 1) & 1][(i) & 1], 0, 0, 0); \
            asm volatile("" : "+v"(acc[((i) >> 1) & 1][(i) & 1])); SB(); } while (0)
#define LACKS(need) (((rfl(P) - (u32)(need) * 0x10001u) & 0x80008000u) != 0)
#define AWAIT(j, need) do { \
            if (__builtin_expect(LACKS(need), 0)) { do { P = prog[(j) & 3]; } while (LACKS(need)); } \
            asm volatile("" ::: "memory"); SB(); } while (0)
#define RELEASE(j, t) do { asm volatile("" ::: "memory"); *done = (t) + 1; P = prog[((j) + 1) & 3]; SB(); } while (0)
#define STEP(X, W, XN, WN, j) do { constexpr u32 o_ = ((j) & 3) * 16384; \
            MM1(X, W, 0); AWAIT(j, (t4 >> 2) + ((j) >> 2) + 1); MM1(X, W, 1); \
            if ((j) == 4) { xb ^= 65536u; wb ^= 65536u; } \
            RD1(XN, WN, o_, 0); RD1(XN, WN, o_, 1); MM1(X, W, 2); \
            RD1(XN, WN, o_, 2); RD1(XN, WN, o_, 3); MM1(X, W, 3); RELEASE(j, t4 + (j)); \
            MM1(X, W, 4); MM1(X, W, 5); MM1(X, W, 6); MM1(X, W, 7); } while (0)
        if (threadIdx.x < 12) *(LAS u32*)(lds + L_MISC + 64 + threadIdx.x * 4) = 0;
        BARRIER();
        if (threadIdx.x == 0 && call) __hip_atomic_store(ctr, call, __ATOMIC_RELAXED, __HIP_MEMORY_SCOPE_AGENT);
        u32 P = 0;
        AWAIT(0, 1);
        RD1(x0, w0, 0, 0); RD1(x0, w0, 0, 1); RD1(x0, w0, 0, 2); RD1(x0, w0, 0, 3);
        RELEASE(0, 0);
        int t4 = 0;
        for (; t4 < 2 * NSTEP - 4; t4 += 4) {
            STEP(x0, w0, x1, w1, 1); STEP(x1, w1, x0, w0, 2); STEP(x0, w0, x1, w1, 3); STEP(x1, w1, x0, w0, 4);
        }
        STEP(x0, w0, x1, w1, 1); STEP(x1, w1, x0, w0, 2); STEP(x0, w0, x1, w1, 3);
#pragma unroll
        for (int i = 0; i < 8; ++i) MM1(x1, w1, i);
    }
    STAMP(2);
    u32 y[64];
    u32 mx = 0;
#pragma unroll
    for (int e = 0; e < 64; ++e) {
        y[e] = (u32)max((int)__float_as_uint(acc[e >> 5][e & 1][(e >> 1) & 15]), 0);
        mx = max(mx, y[e]);
    }
    mx = wave_umax(mx);
    if (lane == 0) *(LAS u32*)(lds + L_MISC + 16 + wave * 4) = mx;
    BARRIER();
    if (wave == 0) {
        u32x4 wx = *(LAS u32x4*)(lds + L_MISC + 16);
        u32 tm = max(max(wx[0], wx[1]), max(wx[2], wx[3]));
        if (lane < NR) st_sys64(amp, ((u64)call << 32) | tm);
        STAMP(6);
    }

    u32 mk0 = 0, mk1 = 0;
#pragma unroll
    for (int e = 31; e >= 0; --e) {
        mk0 = __builtin_amdgcn_alignbit(mk0, 0u - y[e], 31);
        mk1 = __builtin_amdgcn_alignbit(mk1, 0u - y[e + 32], 31);
    }
    const u32 cnt = __builtin_popcount(mk0) + __builtin_popcount(mk1);
    const u32 incl = wave_incl_scan(cnt);
    const u32 total = __builtin_amdgcn_readlane(incl, 63);
    const int uq = g * 4 + wave;
    const int rot = uq % 7;
    const u32 uoff = P2_OFF + (rank * 512 + uq) * P2_B;
    char* pd[7];
#pragma unroll
    for (int k = 0; k < 7; ++k) pd[k] = p.peer[rot + k];
    asm volatile("" : "+s"(pd[0]), "+s"(pd[1]), "+s"(pd[2]), "+s"(pd[3]), "+s"(pd[4]), "+s"(pd[5]), "+s"(pd[6]));
    u32x4 mv;
    {
        auto s0 = __builtin_amdgcn_permlane32_swap(mk0, mk0, false, false);
        auto s1 = __builtin_amdgcn_permlane32_swap(mk1, mk1, false, false);
        mv = u32x4{s0[0], s1[0], s0[1], s1[1]};
        if (lane < 32) {
#pragma unroll
            for (int k = 0; k < NEARLY; ++k) st_wt128s(pd[k], uoff + lane * 16, mv);
        }
    }
    LAS char* cb = lds + L_COMP + wave * 16384;
    {
        u32 pos = L_COMP + wave * 16384 + (incl - cnt) * 4;
#define C1(n) "v_cmpx_lt_i32_e32 vcc, 0, %" #n "\n\tds_write_b32 %0, %" #n "\n\tv_add_u32 %0, 4, %0\n\ts_mov_b64 exec, -1\n\t"
#pragma unroll
        for (int e = 0; e < 64; e += 8)
            asm volatile(C1(1) C1(2) C1(3) C1(4) C1(5) C1(6) C1(7) C1(8)
                         : "+v"(pos)
                         : "v"(y[e]), "v"(y[e + 1]), "v"(y[e + 2]), "v"(y[e + 3]), "v"(y[e + 4]), "v"(y[e + 5]),
                           "v"(y[e + 6]), "v"(y[e + 7])
                         : "memory", "vcc");
        asm volatile("s_nop 3");
        if (lane < 16 && total + lane < 4096u) *(LAS u32*)(cb + (total + lane) * 4) = mx;
    }
    const u32 ng = (total + 15) >> 4;
    f32x4 cv[4][4];
#pragma unroll
    for (int r = 0; r < 4; ++r)
#pragma unroll
        for (int j = 0; j < 4; ++j) cv[r][j] = *(LAS f32x4*)(cb + r * 4096 + lane * 64 + j * 16);
    STAMP(3);
    BARRIER();
    float inv;
    {
        u32x4 gv = *(LAS u32x4*)(lds + L_MISC + 32), gw = *(LAS u32x4*)(lds + L_MISC + 48);
        float amax = __uint_as_float(max(max(max(gv[0], gv[1]), max(gv[2], gv[3])), max(max(gw[0], gw[1]), max(gw[2], gw[3]))));
        inv = 1.0f / (amax / 448.0f);
    }
    STAMP(4);
    u32x4 q[4];
#pragma unroll
    for (int r = 0; r < 4; ++r) {
        q[r] = u32x4{0x7F7F7F7Fu, 0x7F7F7F7Fu, 0x7F7F7F7Fu, 0x7F7F7F7Fu};
        if (r < 2 || (u32)(r * 64) < ng) {
#pragma unroll
            for (int j = 0; j < 4; ++j) {
                f32x4 t = cv[r][j] * inv;
                int w = __builtin_amdgcn_cvt_pk_fp8_f32(t[0], t[1], 0, false);
                q[r][j] = (u32)__builtin_amdgcn_cvt_pk_fp8_f32(t[2], t[3], w, true);
            }
        }
    }
    u32 h2, nb;
    {
        const u32 cmax = rfl((u32)__builtin_amdgcn_cvt_pk_fp8_f32(__uint_as_float(mx) * inv, 0.f, 0, false)) & 0xFFu;
        const u32 wl = max(cmax, 63u) - 63u, k4 = (0x80u - wl) * 0x01010101u;
        u32 m[2] = {0, 0};
#pragma unroll
        for (int r = 0; r < 4; ++r) {
            u32 t[4];
#pragma unroll
            for (int j = 0; j < 4; ++j) { t[j] = q[r][j] + k4; m[r >> 1] = bfi(0x80808080u, t[j], m[r >> 1] >> 1); }
            if (r < 2 || (u32)(r * 64) < ng) {
                LAS u32* o = (LAS u32*)(cb + lane * 12 + r * 768);
                o[0] = bfi(0x3F3F3F3Fu, t[0], t[3] << 6);
                o[1] = bfi(0x3F3F3F3Fu, t[1], t[3] << 4);
                o[2] = bfi(0x3F3F3F3Fu, t[2], t[3] << 2);
            }
        }
#pragma unroll
        for (int h = 0; h < 2; ++h)
            m[h] = ~m[h] & (((u32)(lane + 128 * h) < ng ? 0x0F0F0F0Fu : 0u) | ((u32)(lane + 128 * h + 64) < ng ? 0xF0F0F0F0u : 0u));
        const u32 ec = __builtin_popcount(m[0]) + __builtin_popcount(m[1]);
        const u32 ei = wave_incl_scan(ec);
        const u32 ne = __builtin_amdgcn_readlane(ei, 63);
        if (__builtin_expect(ne > MAXE2 || cmax > 126u, 0)) {
            h2 = total | H2_RAW;
            nb = ng * 16;
        } else {
            h2 = total | (ne << 13) | (wl << 21);
            nb = ng * 12 + ne * 2;
            LAS unsigned short* el = (LAS unsigned short*)(cb + ng * 12) + (ei - ec);
            u64 mm = ((u64)m[1] << 32) | m[0];
            while (mm != 0) {
                const u32 bp = __builtin_ctzll(mm);
                *el++ = (unsigned short)(lane * 16 + (bp >> 5) * 2048 + (bp & 4) * 256 + (bp & 3) * 4 + ((bp >> 3) & 3));
                mm &= mm - 1;
            }
#pragma unroll
            for (int r = 0; r < 4; ++r) q[r] = *(LAS u32x4*)(cb + r * 1024 + lane * 16);
        }
    }
#pragma unroll
    for (int k = 0; k < 7; ++k) {
        char* d = pd[k];
        if (k >= NEARLY && lane < 32) st_wt128s(d, uoff + lane * 16, mv);
#pragma unroll
        for (int r = 0; r < 4; ++r)
            if ((u32)(lane * 16 + r * 1024) < nb) st_wt128s(d, uoff + 512 + r * 1024 + lane * 16, q[r]);
        if (k == 0) {
            u32x4 ot[4];
#pragma unroll
            for (int mt = 0; mt < 4; ++mt) {
                u32x4 ov;
#pragma unroll
                for (int j = 0; j < 4; ++j) {
                    const int e = mt * 16 + j * 4;
                    int w = __builtin_amdgcn_cvt_pk_fp8_f32(__uint_as_float(y[e]) * inv, __uint_as_float(y[e + 1]) * inv, 0, false);
                    ov[j] = (u32)__builtin_amdgcn_cvt_pk_fp8_f32(__uint_as_float(y[e + 2]) * inv, __uint_as_float(y[e + 3]) * inv, w, true);
                }
                ot[mt] = ov;
            }
            store_tile(p.out + (size_t)((rank >> 2) * 2048 + tr * 128 + wm * 64) * NN + (rank & 3) * 1024 + tc * 128 + wn * 64,
                       ot, lane);
        }
        WAIT_VM0();
        if (lane == 0) st_sys64(d + F2_OFF + (rank * 512 + uq) * FL_B, ((u64)call << 32) | h2);
    }
    STAMP(5);
}

extern "C" void kernel_launch(int rank, int nranks, ncclComm_t comm, hipStream_t stream,
                              void* const* d_in, const int* in_sizes, int n_in,
                              void* d_out, int out_size, const int* in_dtypes, int out_dtype,
                              void* const* d_out_all, void* const* d_ws_all, size_t ws_bytes) {
    (void)comm; (void)in_dtypes; (void)out_dtype; (void)d_out_all;
    if (nranks != NR || (unsigned)rank >= (unsigned)NR || n_in < 2 || ws_bytes < (size_t)SPARE_OFF + WS_USED ||
        in_sizes[0] != MM * KS || in_sizes[1] != KS * NN || out_size != MM * NN) {
        fprintf(stderr, "kernel_launch: unexpected problem shape\n");
        return;
    }
    HIP_CHECK(hipFuncSetAttribute((const void*)fused_kernel, hipFuncAttributeMaxDynamicSharedMemorySize, LDS_BYTES));
    Params p;
    p.x = (const char*)d_in[0];
    p.w = (const char*)d_in[1];
    p.out = (char*)d_out;
    for (int r = 0; r < NR; ++r) p.ws[r] = (char*)d_ws_all[r];
    for (int r = 0; r < NR; ++r) p.wsd[r] = p.ws[r] + (r == rank ? SPARE_OFF : 0u);
    for (int k = 0; k < 14; ++k) p.peer[k] = p.ws[(rank + 1 + k % 7) & 7];
    p.rank = rank;
    p.pad = 0;
    fused_kernel<<<dim3(256), dim3(768), LDS_BYTES, stream>>>(p);
    HIP_CHECK(hipGetLastError());
}
```

```cpp
#include <hip/hip_runtime.h>
#include <rccl/rccl.h>
#include <cstdio>
#include <cstdint>

#define HIP_CHECK(expr) do { hipError_t _e = (expr); if (_e != hipSuccess) { \
    fprintf(stderr, "HIP error %s at %s:%d\n", hipGetErrorString(_e), __FILE__, __LINE__); \
    return; } } while (0)

typedef unsigned u32;
typedef unsigned long long u64;
typedef u32 u32x2 __attribute__((ext_vector_type(2)));
typedef u32 u32x3 __attribute__((ext_vector_type(3)));
typedef u32 u32x4 __attribute__((ext_vector_type(4)));
typedef float f32x2 __attribute__((ext_vector_type(2)));
typedef float f32x4 __attribute__((ext_vector_type(4)));
typedef float f32x16 __attribute__((ext_vector_type(16)));
typedef unsigned short u16x2 __attribute__((ext_vector_type(2)));
typedef __bf16 bf16x8 __attribute__((ext_vector_type(8)));
#define GAS __attribute__((address_space(1)))
#define LAS __attribute__((address_space(3)))
#define DI __device__ __forceinline__

constexpr int NR = 8;
constexpr int MM = 4096, NN = 4096, KS = 512;
constexpr int NCH = 16;
constexpr int NSLOT = 48;
constexpr u32 NITEM = NCH * 128;
#define NEARLY 7
#define NPUSH 112
constexpr int NSTEP = 64;
constexpr u32 UNIT_B = 3072, RAWU_B = 4096, FL_B = 128;
constexpr u32 CTR_OFF = 0;
constexpr u32 TMO_OFF = 32768;
constexpr u32 TKT_OFF = 33024;
constexpr u32 AMAX_OFF = 65536;
constexpr u32 F1_OFF = 131072;
constexpr u32 F2_OFF = 1u << 20;
constexpr u32 DBG_OFF = 1572864;
constexpr u32 P1_OFF = 2u << 20;
constexpr u32 RAW_OFF = 20u << 20;
constexpr u32 P2_OFF = 44u << 20;
constexpr u32 P2_B = 4608;
constexpr u32 WS_USED = 62u << 20;
constexpr u32 SPARE_OFF = 64u << 20;
static_assert(F1_OFF + NR * NCH * NSLOT * FL_B <= F2_OFF, "");
static_assert(F2_OFF + NR * 512 * FL_B <= DBG_OFF, "");
static_assert(P1_OFF + NR * NCH * NSLOT * UNIT_B <= RAW_OFF, "");
static_assert(RAW_OFF + NR * NCH * NSLOT * RAWU_B <= P2_OFF, "");
static_assert(P2_OFF + NR * 512 * P2_B <= WS_USED, "");
constexpr u64 DEADLINE = 50000000ull;
constexpr int MAXEXC = 8;
constexpr u32 MAXL1 = 64;
constexpr u32 HDR_RAW = 0x1000;
constexpr int MAXE2 = 128;
constexpr u32 H2_RAW = 1u << 28;

constexpr u32 L_STAGE = 0;
constexpr u32 L_COMP = 0;
constexpr u32 L_MISC = 131072;
constexpr u32 L_DEC = 0;
constexpr u32 DEC_B = 4352;
constexpr u32 L_LUT = 53248;
constexpr u32 L_SCR = 57344;
constexpr u32 SCR_B = 5248;
constexpr u32 LDS_BYTES = 131072 + 512;

struct Params {
    const char* x;
    const char* w;
    char* out;
    char* ws[NR];
    char* peer[14];
    char* wsd[NR];
    int rank;
    int pad;
};

#define PAY_AUX 17
#define ACC(var) do {} while (0)
#define PUT(i, val) do {} while (0)
#define STAMP(i) do {} while (0)

#define PSTAMP() do {} while (0)
DI u32 perm(u32 hi, u32 lo, u32 sel) { return __builtin_amdgcn_perm(hi, lo, sel); }
DI u32 bfi(u32 m, u32 a, u32 b) { return (m & a) | (~m & b); }
DI u32 rotr(u32 v, u32 n) { return __builtin_amdgcn_alignbit(v, v, n); }
DI u32 rfl(u32 v) { return __builtin_amdgcn_readfirstlane(v); }
template <int CTRL, int RM> DI u32 dpp0(u32 v) { return __builtin_amdgcn_update_dpp(0u, v, CTRL, RM, 0xF, false); }
DI u32 wave_umax(u32 v) {
    v = max(v, dpp0<0x111, 0xF>(v)); v = max(v, dpp0<0x112, 0xF>(v));
    v = max(v, dpp0<0x114, 0xF>(v)); v = max(v, dpp0<0x118, 0xF>(v));
    v = max(v, dpp0<0x142, 0xA>(v)); v = max(v, dpp0<0x143, 0xC>(v));
    return __builtin_amdgcn_readlane(v, 63);
}
DI u32 wave_incl_scan(u32 v) {
    v += dpp0<0x111, 0xF>(v); v += dpp0<0x112, 0xF>(v);
    v += dpp0<0x114, 0xF>(v); v += dpp0<0x118, 0xF>(v);
    v += dpp0<0x142, 0xA>(v); v += dpp0<0x143, 0xC>(v);
    return v;
}
DI u64 ld_sys64(const char* p) {
    return __hip_atomic_load((GAS u64*)p, __ATOMIC_RELAXED, __HIP_MEMORY_SCOPE_SYSTEM);
}
DI void st_sys64(char* p, u64 v) {
    __hip_atomic_store((GAS u64*)p, v, __ATOMIC_RELAXED, __HIP_MEMORY_SCOPE_SYSTEM);
}
DI void st_wt128(char* p, u32x4 v) {
    asm volatile("global_store_dwordx4 %0, %1, off sc0 sc1\n\ts_nop 1" :: "v"(p), "v"(v) : "memory");
}
DI void st_wt128s(const char* sbase, u32 voff, u32x4 v) {
    asm volatile("global_store_dwordx4 %0, %1, %2 sc0 sc1\n\ts_nop 1" :: "v"(voff), "v"(v), "s"(sbase) : "memory");
}
#define WAIT_VM0() asm volatile("s_waitcnt vmcnt(0)" ::: "memory")
#define WAIT_LGKM0() asm volatile("s_waitcnt lgkmcnt(0)" ::: "memory")
#define BARRIER() do { WAIT_LGKM0(); __builtin_amdgcn_s_barrier(); asm volatile("" ::: "memory"); } while (0)

DI bool spin_expired(char* myws, u32 call, u64 t0, u32& it) {
    if ((++it & 15u) == 0) {
        GAS u32* tmo = (GAS u32*)(myws + TMO_OFF);
        u32 t = rfl(__hip_atomic_load(tmo, __ATOMIC_RELAXED, __HIP_MEMORY_SCOPE_AGENT));
        if (t == call || __builtin_amdgcn_s_memrealtime() - t0 > DEADLINE) {
            __hip_atomic_store(tmo, call, __ATOMIC_RELAXED, __HIP_MEMORY_SCOPE_AGENT);
            return true;
        }
    }
    __builtin_amdgcn_s_sleep(1);
    return false;
}
DI bool tmo_set(char* myws, u32 call) {
    return rfl(__hip_atomic_load((GAS u32*)(myws + TMO_OFF), __ATOMIC_RELAXED, __HIP_MEMORY_SCOPE_AGENT)) == call;
}

DI u32 twice(u32 v) {
    const u32 m = v & 0x7FFFu;
    return (v & 0x8000u) | (m < 0x80u ? m << 1 : m >= 0x7F00u ? max(m, 0x7F80u) : m + 0x80u);
}
DI u32 pack_unit(const u32x4 (&R)[4], u32x4& p0, u32x4& p1, u32x4& p2, u64& act, LAS char* scr, int lane,
                 char* d0, char* d1, char* d2, char* d3, char* fl, int nd, u32 unit) {
    u32 L[8], H[8];
    u16x2 mx = {0, 0};
#pragma unroll
    for (int f = 0; f < 4; ++f) {
        const u32x4 r = R[f] + 0x00800080u;
        L[2 * f] = perm(r[1], r[0], 0x06040200u); H[2 * f] = perm(r[1], r[0], 0x07050301u);
        L[2 * f + 1] = perm(r[3], r[2], 0x06040200u); H[2 * f + 1] = perm(r[3], r[2], 0x07050301u);
#pragma unroll
        for (int d = 0; d < 4; ++d)
            mx = __builtin_elementwise_max(mx, __builtin_bit_cast(u16x2, R[f][d] & 0x7FFF7FFFu));
    }
    const u32 big = wave_umax(max((u32)mx[0], (u32)mx[1]));
    u32 top = (big + 0x80u) >> 8;
    bool raw = big >= 0x7F00u || top < 10u;
    u32 ct = 0;
#pragma unroll
    for (int i = 0; i < 8; ++i) ct += __builtin_popcount(((H[i] | 0x80808080u) - top * 0x01010101u) & 0x80808080u);
    if (__builtin_amdgcn_readlane(wave_incl_scan(ct), 63) <= 3u) top -= 1u;
    const u32 base = (top - 3u) & 0x7Fu, base4 = base * 0x01010101u;
    u32 T[8], N[8], far = 0, m1 = 0;
#pragma unroll
    for (int i = 0; i < 8; ++i) {
        T[i] = (H[i] | 0x80808080u) - base4;
        far |= (T[i] + 0x04040404u) ^ 0x80808080u;
        m1 = bfi(0x80808080u, T[i], m1 >> 1);
        N[i] = bfi(0x80808080u, H[i], T[i]);
    }
    LAS char* stg = scr + 4096 + 64;
    *(LAS u32*)(stg + 768 + (lane & 31) * 4) = 0;
    {
        LAS u32* o = (LAS u32*)(stg + lane * 12);
        o[0] = bfi(0x60606060u, N[6] << 5, bfi(0x83838383u, N[0], rotr(N[1], 29)));
        o[1] = bfi(0x60606060u, N[7] << 5, bfi(0x83838383u, N[2], rotr(N[3], 29)));
        o[2] = bfi(0x60606060u, bfi(0x20202020u, N[6] >> 2, N[7] >> 1), bfi(0x83838383u, N[4], rotr(N[5], 29)));
    }
    m1 = ~m1;
    const u32 c1 = __builtin_popcount(m1), i1 = wave_incl_scan(c1);
    u32 n1 = __builtin_amdgcn_readlane(i1, 63);
    raw = raw || n1 > MAXL1;
    if (!raw) {
        LAS unsigned short* el = (LAS unsigned short*)(stg + 768) + (i1 - c1);
        while (m1 != 0) {
            const u32 bp = __builtin_ctz(m1);
            *el++ = (unsigned short)((bp & 6) * 512 + lane * 16 + (bp & 1) * 8 + (bp >> 3) * 2 + 1);
            m1 &= m1 - 1;
        }
    }
    p0 = u32x4{L[0], L[1], L[2], L[3]};
    p1 = u32x4{L[4], L[5], L[6], L[7]};
    p2 = *(LAS u32x4*)(stg + lane * 16);
    u32 hdr = base;
    if (raw || __builtin_amdgcn_ballot_w64((far & 0xF8F8F8F8u) != 0) != 0) {
        u32 cnt = 0;
        if (!raw) {
#pragma unroll
            for (int f = 0; f < 4; ++f) *(LAS u32x4*)(scr + f * 1024 + lane * 16) = R[f];
            u32 m = 0;
#pragma unroll
            for (int i = 0; i < 8; ++i) {
                const u32 z = ((T[i] + 0x04040404u) ^ 0x80808080u) & 0xF8F8F8F8u;
                m |= ((((z & 0x7F7F7F7Fu) + 0x7F7F7F7Fu) | z) & 0x80808080u) >> (7 - i);
            }
            for (;;) {
                u64 bal = __builtin_amdgcn_ballot_w64(m != 0);
                if (bal == 0 || cnt > MAXEXC) break;
                if (m != 0) {
                    u32 bp = __builtin_ctz(m);
                    u32 addr = (bp & 6) * 512 + lane * 16 + (bp & 1) * 8 + (bp >> 3) * 2;
                    u32 v = twice(*(LAS unsigned short*)(scr + addr));
                    u32 slot = cnt + __builtin_amdgcn_mbcnt_hi((u32)(bal >> 32), __builtin_amdgcn_mbcnt_lo((u32)bal, 0));
                    if (slot < MAXEXC) *(LAS u32*)(scr + 4096 + slot * 4) = (addr << 16) | v;
                    m &= m - 1;
                }
                cnt += __builtin_popcountll(bal);
            }
        }
        if (raw || cnt > MAXEXC) {
            hdr = HDR_RAW;
            n1 = 0;
#pragma unroll
            for (int f = 0; f < 4; ++f) {
                u32x4 r;
#pragma unroll
                for (int d = 0; d < 4; ++d) r[d] = twice(R[f][d] & 0xFFFFu) | (twice(R[f][d] >> 16) << 16);
                const u32 o = RAW_OFF + unit * RAWU_B + f * 1024 + lane * 16;
                st_wt128s(d0, o, r);
                st_wt128s(d1, o, r);
                if (nd > 2) { st_wt128s(d2, o, r); st_wt128s(d3, o, r); }
            }
        } else {
            hdr |= cnt << 8;
            if (lane < 8 && (lane & 3) < nd) {
                u32x4 rec = *(LAS u32x4*)(scr + 4096 + (lane & 4) * 4);
                st_wt128(fl + 64 + (lane & 4) * 4, rec);
            }
        }
    }
    act = (1ull << (48 + ((n1 + 31) >> 5) * 4)) - 1;
    return hdr | (n1 << 16);
}

#define A_TICKET(tk, adr) "s_mov_b64 exec, 1\n\tglobal_atomic_add %" #tk ", %" #adr ", %" #tk ", off sc0\n\ts_mov_b64 exec, -1\n\t"
#define A_ST(so, v, d, off) "global_store_dwordx4 %" #so ", %" #v ", %" #d " offset:" #off " sc0 sc1\n\t"
#define A_ST4(so, a, b, c, m, d0, d1, d2, d3) \
    A_ST(so, a, d0, 0) A_ST(so, a, d1, 0) A_ST(so, a, d2, 0) A_ST(so, a, d3, 0) \
    A_ST(so, b, d0, 1024) A_ST(so, b, d1, 1024) A_ST(so, b, d2, 1024) A_ST(so, b, d3, 1024) \
    "s_mov_b64 exec, %" #m "\n\t" \
    A_ST(so, c, d0, 2048) A_ST(so, c, d1, 2048) A_ST(so, c, d2, 2048) A_ST(so, c, d3, 2048)

#define A_ST2(so, a, b, c, m, d0, d1) \
    A_ST(so, a, d0, 0) A_ST(so, a, d1, 0) A_ST(so, b, d0, 1024) A_ST(so, b, d1, 1024) \
    "s_mov_b64 exec, %" #m "\n\t" A_ST(so, c, d0, 2048) A_ST(so, c, d1, 2048)
#define A_LDX "global_load_dwordx4 %0, %5, %7\n\tglobal_load_dwordx4 %1, %6, %7\n\t" \
              "global_load_dwordx4 %2, %5, %7 offset:32\n\tglobal_load_dwordx4 %3, %6, %7 offset:32\n\t"
#define A_WL(i) "global_load_dword %" #i ", %17, %18\n\tv_add_u32 %17, 0x2000, %17\n\t"
#define A_LDW A_WL(0) A_WL(1) A_WL(2) A_WL(3) A_WL(4) A_WL(5) A_WL(6) \
              "global_load_dword %7, %17, %18\n\tv_add_u32 %17, 0x12000, %17\n\t" \
              A_WL(8) A_WL(9) A_WL(10) A_WL(11) A_WL(12) A_WL(13) A_WL(14) "global_load_dword %15, %17, %18\n\t"
#define X_OUT "=&v"(R[0]), "=&v"(R[1]), "=&v"(R[2]), "=&v"(R[3]), "+v"(tk)
#define X_IN "v"(lx0), "v"(lx1), "s"(src), "v"(tka), "v"(so), "v"(p0), "v"(p1), "v"(p2), "s"(act)
#define W_OUT "=&v"(G[0]), "=&v"(G[1]), "=&v"(G[2]), "=&v"(G[3]), "=&v"(G[4]), "=&v"(G[5]), "=&v"(G[6]), \
              "=&v"(G[7]), "=&v"(G[8]), "=&v"(G[9]), "=&v"(G[10]), "=&v"(G[11]), "=&v"(G[12]), "=&v"(G[13]), \
              "=&v"(G[14]), "=&v"(G[15]), "+v"(tk), "+v"(lw)
#define W_IN "s"(src), "v"(tka), "v"(so), "v"(p0), "v"(p1), "v"(p2), "s"(act)

DI void w_to_frags(const u32 (&G)[16], u32x4 (&R)[4]) {
#pragma unroll
    for (int f = 0; f < 4; ++f)
#pragma unroll
        for (int d = 0; d < 4; ++d)
            R[f][d] = perm(G[(f >> 1) * 8 + 2 * d + 1], G[(f >> 1) * 8 + 2 * d], (f & 1) ? 0x07060302u : 0x05040100u);
}

struct Item { bool isw; u32 unit; const char* src; char *d0, *d1, *d2, *d3; };
DI Item make_item(const Params& p, u32 i) {
    Item it;
    const u32 c = (i >> 7) & (NCH - 1), k = (i >> 1) & 63;
    it.isw = i & 1;
    const u32 r0 = it.isw ? (k & 3) : (k & 1) * 4, rs = it.isw ? 4 : 1;
    const u32 sl = it.isw ? k >> 2 : k >> 1;
    it.unit = (p.rank * NCH + c) * NSLOT + (it.isw ? 32 : 0) + sl;
    it.src = it.isw ? p.w + (size_t)c * (32 * NN * 2) + ((k & 3) * 16 + sl) * 128
                    : p.x + (size_t)((k & 1) * 32 + sl) * (64 * KS * 2) + c * 64;
    it.d0 = p.wsd[r0]; it.d1 = p.wsd[r0 + rs]; it.d2 = p.wsd[(r0 + 2 * rs) & 7]; it.d3 = p.wsd[(r0 + 3 * rs) & 7];
    return it;
}

DI u32 lane_x(int lane) { return (u32)(lane & 31) * (KS * 2) + (lane >> 5) * 16; }
DI u32 lane_w(int lane) {
    return (u32)(8 * (lane >> 5)) * (NN * 2) + ((lane & 19) | ((lane & 4) << 1) | ((lane & 8) >> 1)) * 4;
}

DI void push_inputs(const Params& p, u32 g, u32 ctr_old, LAS char* scr, int lane) {
    const u32 lx0 = lane_x(lane), lx1 = lx0 + 32 * KS * 2, lw0 = lane_w(lane);
    char* const tka = p.ws[p.rank] + TKT_OFF;
    Item cur = make_item(p, 2 * g), nxt = make_item(p, 2 * g + 1);
    u32 G[16];
    u32x4 R[4], p0, p1, p2;
    if (cur.isw) {
#pragma unroll
        for (int j = 0; j < 16; ++j) G[j] = *(const GAS u32*)(cur.src + lw0 + ((j >> 3) * 16 + (j & 7)) * (NN * 2));
        w_to_frags(G, R);
    } else {
#pragma unroll
        for (int f = 0; f < 4; ++f) R[f] = *(const GAS u32x4*)(cur.src + ((f & 1) ? lx1 : lx0) + (f >> 1) * 32);
    }
    const u32 call = rfl(ctr_old) + 1u;
    const u32 tbase = (call - 1u) * NITEM - 2u * NPUSH;
    bool more = true;
    char* fp = nullptr;
    u32 fv = 0, tk = 0;
    int fn = 0;
    for (;;) {
        const int nd = cur.isw ? 2 : 4;
        char* const dl = (lane & 1) ? cur.d1 : cur.d0;
        char* const dh = (lane & 1) ? cur.d3 : cur.d2;
        char* const fl = ((lane & 2) ? dh : dl) + F1_OFF + cur.unit * FL_B;
        u64 act;
        const u32 hdr = pack_unit(R, p0, p1, p2, act, scr, lane, cur.d0, cur.d1, cur.d2, cur.d3, fl, nd, cur.unit);
        const u32 so = P1_OFF + cur.unit * UNIT_B + lane * 16;
        asm volatile("s_waitcnt vmcnt(0)" : "+v"(tk), "+v"(p0), "+v"(p1), "+v"(p2) : "v"(so), "s"(act) : "memory");
        if (lane < fn) st_sys64(fp, ((u64)call << 32) | fv);
        if (fn != 0) {
            const u32 i = rfl(tk) - tbase;
            more = i < NITEM;
            nxt = make_item(p, i);
        }
        const char* src = nxt.src;
        tk = 1;
        if (nxt.isw) {
            u32 lw = lw0;
            if (cur.isw)
                asm volatile("s_nop 4\n\t" A_LDW A_ST2(20, 21, 22, 23, 24, 25, 26) A_TICKET(16, 19) "s_waitcnt vmcnt(7)"
                             : W_OUT : W_IN, "s"(cur.d0), "s"(cur.d1) : "memory");
            else
                asm volatile("s_nop 4\n\t" A_LDW A_ST4(20, 21, 22, 23, 24, 25, 26, 27, 28) A_TICKET(16, 19) "s_waitcnt vmcnt(13)"
                             : W_OUT : W_IN, "s"(cur.d0), "s"(cur.d1), "s"(cur.d2), "s"(cur.d3) : "memory");
            w_to_frags(G, R);
        } else {
            if (cur.isw)
                asm volatile("s_nop 4\n\t" A_LDX A_ST2(9, 10, 11, 12, 13, 14, 15) A_TICKET(4, 8) "s_waitcnt vmcnt(7)"
                             : X_OUT : X_IN, "s"(cur.d0), "s"(cur.d1) : "memory");
            else
                asm volatile("s_nop 4\n\t" A_LDX A_ST4(9, 10, 11, 12, 13, 14, 15, 16, 17) A_TICKET(4, 8) "s_waitcnt vmcnt(13)"
                             : X_OUT : X_IN, "s"(cur.d0), "s"(cur.d1), "s"(cur.d2), "s"(cur.d3) : "memory");
        }
        fp = fl;
        fv = hdr;
        fn = nd;
        if (!more) break;
        cur = nxt;
    }
    WAIT_VM0();
    if (lane < fn) st_sys64(fp, ((u64)call << 32) | fv);
}

DI void pack_own(const Params& p, u32 s, u32 call, LAS char* scr, int lane) {
    const u32 c = s / NSLOT, sl = s % NSLOT;
    u32x4 R[4], p0, p1, p2;
    if (sl >= 32) {
        const char* src = p.w + (size_t)c * (32 * NN * 2) + ((p.rank & 3) * 16 + sl - 32) * 128 + lane_w(lane);
        u32 G[16];
#pragma unroll
        for (int j = 0; j < 16; ++j) G[j] = *(const GAS u32*)(src + ((j >> 3) * 16 + (j & 7)) * (NN * 2));
        w_to_frags(G, R);
    } else {
        const char* src = p.x + (size_t)((p.rank >> 2) * 32 + sl) * (64 * KS * 2) + c * 64 + lane_x(lane);
#pragma unroll
        for (int f = 0; f < 4; ++f) R[f] = *(const GAS u32x4*)(src + (f & 1) * (32 * KS * 2) + (f >> 1) * 32);
    }
    char* const me = p.ws[p.rank];
    const u32 unit = p.rank * (NCH * NSLOT) + s;
    char* const fl = me + F1_OFF + unit * FL_B;
    u64 act;
    const u32 hdr = pack_unit(R, p0, p1, p2, act, scr, lane, me, me, me, me, fl, 1, unit);
    const u32 so = P1_OFF + unit * UNIT_B + lane * 16;
    st_wt128s(me, so, p0); st_wt128s(me, so + 1024, p1); st_wt128s(me, so + 2048, p2);
    WAIT_VM0();
    if (lane == 0) st_sys64(fl, ((u64)call << 32) | hdr);
}

typedef __amdgpu_buffer_rsrc_t rsrc_t;
struct PSet { u32x4 lo[4]; u32x3 hi[2]; u32 lst, rec; u32 hdr0, hdr1; u32 unit; };

DI u32x2 load_flags(rsrc_t rs, u32 voff, u32 unit) {
    return __builtin_amdgcn_raw_buffer_load_b64(rs, voff, F1_OFF + unit * FL_B, 17);
}
DI bool wait_flags(u32x2& F0, u32x2& Fn, rsrc_t rs, char* myws, u32 unit, u32 next, int lane, u32 call) {
    u64 t0 = __builtin_amdgcn_s_memrealtime();
    u32 it = 0;
    bool bad = false;
    const u32 voff = F1_OFF + (((lane & 2) ? next : unit) + (lane & 1)) * FL_B;
    u32x2 F;
    for (;;) {
        asm volatile("" ::: "memory");
        F = __builtin_amdgcn_raw_buffer_load_b64(rs, voff, 0, 17);
        if ((__builtin_amdgcn_ballot_w64(F[1] < call) & 3) == 0) break;
        if (spin_expired(myws, call, t0, it)) { bad = true; break; }
    }
    F0 = F;
    Fn[0] = __builtin_amdgcn_update_dpp(0u, F[0], 0xEE, 0xF, 0xF, false);
    Fn[1] = __builtin_amdgcn_update_dpp(0u, F[1], 0xEE, 0xF, 0xF, false);
    return bad;
}
DI void issue_set(PSet& S, u32x2 F, rsrc_t rs, u32 unit, int lane) {
    asm volatile("" ::: "memory");
    S.hdr0 = __builtin_amdgcn_readlane(F[0], 0);
    S.hdr1 = __builtin_amdgcn_readlane(F[0], 1);
    S.unit = unit;
#pragma unroll
    for (int u = 0; u < 2; ++u) {
        S.hi[u] = __builtin_amdgcn_raw_buffer_load_b96(rs, lane * 12 + u * UNIT_B + 2048, P1_OFF + unit * UNIT_B, 17);
        S.lo[2 * u] = __builtin_amdgcn_raw_buffer_load_b128(rs, lane * 16 + u * UNIT_B, P1_OFF + unit * UNIT_B, 17);
        S.lo[2 * u + 1] = __builtin_amdgcn_raw_buffer_load_b128(rs, lane * 16 + u * UNIT_B + 1024, P1_OFF + unit * UNIT_B, 17);
    }
    const u32 n1 = ((lane & 32) ? S.hdr1 : S.hdr0) >> 16;
    S.lst = __builtin_amdgcn_raw_buffer_load_b32(
        rs, (u32)(lane & 31) * 2 < n1 ? (lane >> 5) * UNIT_B + 2816 + (lane & 31) * 4 : 0x80000000u, P1_OFF + unit * UNIT_B, 17);
    S.rec = __builtin_amdgcn_raw_buffer_load_b32(rs, (lane & 7) * 4 + 64 + (lane & 8) * (FL_B / 8), F1_OFF + unit * FL_B, 17);
}
DI void unpack_set(const PSet& S, rsrc_t rs, LAS char* lds, u32 oo, int lane) {
    LAS char* o = lds + oo;
#pragma unroll
    for (int u = 0; u < 2; ++u) {
        const u32 base4 = ((u ? S.hdr1 : S.hdr0) & 0x7F) * 0x01010101u;
#pragma unroll
        for (int f = 0; f < 4; ++f) {
            u32 h0, h1;
            if (f < 3) {
                const u32 w = S.hi[u][f];
                h0 = (w & 0x83838383u) + base4;
                h1 = (rotr(w, 3) & 0x83838383u) + base4;
            } else {
                h0 = ((S.hi[u][2] << 2) & 0x80808080u) | (((S.hi[u][0] >> 5) & 0x03030303u) + base4);
                h1 = ((S.hi[u][2] << 1) & 0x80808080u) | (((S.hi[u][1] >> 5) & 0x03030303u) + base4);
            }
            u32 l0 = S.lo[2 * u + (f >> 1)][(f & 1) * 2], l1 = S.lo[2 * u + (f >> 1)][(f & 1) * 2 + 1];
            u32x4 r = {perm(h0, l0, 0x05010400u), perm(h0, l0, 0x07030602u),
                       perm(h1, l1, 0x05010400u), perm(h1, l1, 0x07030602u)};
            *(LAS u32x4*)(o + u * 4096 + f * 1024 + lane * 16) = r;
            __builtin_amdgcn_sched_barrier(0);
        }
    }
    {
        const u32 ob = oo + (lane >> 5) * 4096;
        const u32 e0 = S.lst & 0xFFFFu, e1 = S.lst >> 16;
        const u32 a0 = (e0 & 0xFFCu) | ob, a1 = (e1 & 0xFFCu) | ob;
        const u32 v0 = 4u << ((e0 << 3) & 31u), v1 = 4u << ((e1 << 3) & 31u);
        asm volatile("v_cmpx_ne_u32_e32 vcc, 0, %0\n\tds_sub_u32 %2, %4\n\t"
                     "v_cmpx_ne_u32_e32 vcc, 0, %1\n\tds_sub_u32 %3, %5\n\ts_mov_b64 exec, -1\n\ts_nop 3"
                     :: "v"(e0), "v"(e1), "v"(a0), "v"(a1), "v"(v0), "v"(v1) : "memory", "vcc");
    }
    if (__builtin_expect(((S.hdr0 | S.hdr1) & 0xFF00u) != 0, 0)) {
        asm volatile("" ::: "memory");
        const u32 hdr = (lane & 8) ? S.hdr1 : S.hdr0;
        if (lane < 16 && (u32)(lane & 7) < ((hdr >> 8) & 15u))
            *(LAS unsigned short*)(o + (lane & 8) * 512 + ((S.rec >> 16) & 0xFFE)) = (unsigned short)S.rec;
        if (__builtin_expect(((S.hdr0 | S.hdr1) & HDR_RAW) != 0, 0)) {
#pragma unroll
            for (int u = 0; u < 2; ++u)
                if ((u ? S.hdr1 : S.hdr0) & HDR_RAW) {
#pragma unroll
                    for (int f = 0; f < 4; ++f)
                        *(LAS u32x4*)(o + u * 4096 + f * 1024 + lane * 16) = __builtin_amdgcn_raw_buffer_load_b128(
                            rs, lane * 16 + f * 1024, RAW_OFF + (S.unit + u) * RAWU_B, 17);
                }
        }
    }
}

DI void store_tile(char* o, const u32x4 (&v)[4], int lane) {
    o += (size_t)(lane & 15) * NN + ((lane >> 4) & 1) * 32 + (lane >> 5) * 16;
#pragma unroll
    for (int mi = 0; mi < 2; ++mi) {
        u32x4 a, b;
#pragma unroll
        for (int j = 0; j < 4; ++j) {
            auto r = __builtin_amdgcn_permlane16_swap(v[2 * mi][j], v[2 * mi + 1][j], false, false);
            a[j] = r[0]; b[j] = r[1];
        }
        st_wt128(o + (size_t)(32 * mi) * NN, a);
        st_wt128(o + (size_t)(32 * mi + 16) * NN, b);
    }
}

__global__ __launch_bounds__(768, 3) void fused_kernel(Params p) {
    extern __shared__ __attribute__((aligned(16))) char smem_[];
    LAS char* lds = (LAS char*)smem_;
    const int lane = threadIdx.x & 63;
    const int wave = rfl(threadIdx.x >> 6);
    const int b = blockIdx.x, xcd = b & 7, idx = b >> 3;
    const int rank = p.rank;
    char* myws = p.ws[rank];
    STAMP(0);

    GAS u32* ctr = (GAS u32*)(myws + CTR_OFF + b * 128);
    const u32 ctr_old = __hip_atomic_load(ctr, __ATOMIC_RELAXED, __HIP_MEMORY_SCOPE_AGENT);

    if (idx < 16) {
        const int g = xcd * 16 + idx;
        if (threadIdx.x >= 256 && threadIdx.x < 512) {
            u32 m = threadIdx.x - 256, n = 0, s[2] = {0, 0};
#pragma unroll
            for (int e = 0; e < 8; ++e) {
                u32 bit = (m >> e) & 1;
                s[e >> 2] |= (bit ? n : 0x0Cu) << ((e & 3) * 8);
                n += bit;
            }
            *(LAS u32x2*)(lds + L_LUT + m * 8) = u32x2{s[0], s[1]};
        }
        if (wave == 0 && idx < NPUSH / 8) push_inputs(p, idx * 8 + xcd, ctr_old, lds + L_SCR, lane);
        const u32 call = rfl(ctr_old) + 1u;
        STAMP(1);
        BARRIER();
        if (threadIdx.x == 64) __hip_atomic_store(ctr, call, __ATOMIC_RELAXED, __HIP_MEMORY_SCOPE_AGENT);
        const rsrc_t rs = __builtin_amdgcn_make_buffer_rsrc(myws, 0, WS_USED, 0x00020000);
        const int uq = g * 4 + (wave & 3);
        const int rot = uq % 7;
        const int tr = (g >> 5) * 4 + ((g >> 2) & 3), tc = ((g >> 4) & 1) * 4 + (g & 3);
        LAS char* buf = lds + L_DEC + wave * DEC_B;
        for (int k = wave >> 2; k < 7; k += 3) {
            const int src = (rank + 7 - (k + rot) % 7) & 7;
            const u32 un = src * 512 + uq;
            bool bad = false;
            u64 F = ld_sys64(myws + F2_OFF + un * FL_B);
            if (rfl((u32)(F >> 32)) < call) {
                u64 t0 = __builtin_amdgcn_s_memrealtime();
    u32 it = 0;
                for (;;) {
                    F = ld_sys64(myws + F2_OFF + un * FL_B);
                    if (rfl((u32)(F >> 32)) >= call) break;
                    if (spin_expired(myws, call, t0, it)) { bad = true; break; }
                }
            }
            asm volatile("" ::: "memory");
            const u32 h2 = rfl((u32)F);
            const u32 total = min(h2 & 0x1FFFu, 4096u), ng = (total + 15) >> 4;
            const u32 so = P2_OFF + un * P2_B;
            const u32x2 mk = __builtin_amdgcn_raw_buffer_load_b64(rs, (lane & 31) * 16 + (lane >> 5) * 8, so, 17);
            if (__builtin_expect((h2 & H2_RAW) != 0, 0)) {
                u32x4 pl[4];
#pragma unroll
                for (int r = 0; r < 4; ++r)
                    pl[r] = __builtin_amdgcn_raw_buffer_load_b128(
                        rs, (u32)(lane + r * 64) < ng ? (u32)(512 + lane * 16 + r * 1024) : 0x80000000u, so, 17);
                bad = bad || tmo_set(myws, call);
#pragma unroll
                for (int r = 0; r < 4; ++r) *(LAS u32x4*)(buf + r * 1024 + lane * 16) = pl[r];
            } else {
                const u32 ne = min((h2 >> 13) & 0xFFu, (u32)MAXE2), w4 = ((h2 >> 21) & 0x3Fu) * 0x01010101u;
                u32x3 pl[4];
#pragma unroll
                for (int r = 0; r < 4; ++r)
                    pl[r] = __builtin_amdgcn_raw_buffer_load_b96(
                        rs, (u32)(lane + r * 64) < ng ? (u32)(512 + lane * 12 + r * 768) : 0x80000000u, so, 17);
                u32 ex[2];
#pragma unroll
                for (int r = 0; r < 2; ++r)
                    ex[r] = __builtin_amdgcn_raw_buffer_load_b16(
                        rs, (u32)(lane + r * 64) < ne ? 512 + ng * 12 + lane * 2 + r * 128 : 0x80000000u, so, 17);
                bad = bad || tmo_set(myws, call);
#pragma unroll
                for (int r = 0; r < 4; ++r)
                    if (r < 2 || (u32)(r * 64) < ng) {
                        const u32 d3 = bfi(0x03030303u, pl[r][0] >> 6, bfi(0x0C0C0C0Cu, pl[r][1] >> 4, (pl[r][2] >> 2) & 0x30303030u));
                        *(LAS u32x4*)(buf + r * 1024 + lane * 16) =
                            u32x4{(pl[r][0] & 0x3F3F3F3Fu) + w4, (pl[r][1] & 0x3F3F3F3Fu) + w4, (pl[r][2] & 0x3F3F3F3Fu) + w4, d3 + w4};
                    }
#pragma unroll
                for (int r = 0; r < 2; ++r)
                    if ((u32)(lane + r * 64) < ne)
                        __hip_atomic_fetch_and((LAS u32*)(buf + (ex[r] & 0xFFCu)), ~(0x40u << ((ex[r] & 3u) * 8)),
                                               __ATOMIC_RELAXED, __HIP_MEMORY_SCOPE_WAVEFRONT);
                asm volatile("" ::: "memory");
            }
            u32 cnt = __builtin_popcount(mk[0]) + __builtin_popcount(mk[1]);
            u32 o = wave_incl_scan(cnt) - cnt;
            u32x4 ot[4];
#pragma unroll
            for (int mt = 0; mt < 4; ++mt) {
                u32x4 ov;
#pragma unroll
                for (int hh = 0; hh < 2; ++hh) {
                    u32 mb = (mk[mt >> 1] >> ((mt & 1) * 16 + hh * 8)) & 0xFF;
                    u32x2 sel = *(LAS u32x2*)(lds + L_LUT + mb * 8);
                    LAS u32* dp = (LAS u32*)(buf + (o & ~3u));
                    u32 d0 = dp[0], d1 = dp[1], d2 = dp[2];
                    u32 a0 = __builtin_amdgcn_alignbyte(d1, d0, o), a1 = __builtin_amdgcn_alignbyte(d2, d1, o);
                    ov[2 * hh] = perm(a1, a0, sel[0]);
                    ov[2 * hh + 1] = perm(a1, a0, sel[1]);
                    o += __builtin_popcount(mb);
                }
                if (bad) ov = u32x4{0x7F7F7F7Fu, 0x7F7F7F7Fu, 0x7F7F7F7Fu, 0x7F7F7F7Fu};
                ot[mt] = ov;
            }
            store_tile(p.out + (size_t)((src >> 2) * 2048 + tr * 128 + ((wave >> 1) & 1) * 64) * NN +
                       (src & 3) * 1024 + tc * 128 + (wave & 1) * 64, ot, lane);
            STAMP(2 + k / 3);
        }
        return;
    }

    const int g = xcd * 16 + (idx - 16);
    const int tr = (g >> 5) * 4 + ((g >> 2) & 3), tc = ((g >> 4) & 1) * 4 + (g & 3);
    const u32 call = rfl(ctr_old) + 1u;

    if (wave >= 4) {
        const int v = wave - 4, ph = v >> 1;
        const int slot = (v & 1) ? 32 + 2 * tc : 2 * tr;
        const rsrc_t rs = __builtin_amdgcn_make_buffer_rsrc(myws, 0, WS_USED, 0x00020000);
        const u32 oa = L_STAGE + ph * 16384 + (v & 1) * 8192;
        volatile LAS unsigned short* prog = (volatile LAS unsigned short*)(lds + L_MISC + 64 + v * 2);
        volatile LAS u32x4* done = (volatile LAS u32x4*)(lds + L_MISC + 96);
        const u32 fo = (lane & 1) * FL_B;
        u32 utab;
        {
            const u32 t = min(4 * lane + ph, 2 * NSTEP - 1), r = t - NCH;
            utab = (t < NCH ? rank * NCH + t : ((rank + 1 + r % 7) & 7) * NCH + r / 7) * NSLOT + slot;
        }
#define UNIT(i) ((u32)__builtin_amdgcn_readlane(utab, (i)))
        PSet A, B;
        bool bad = false;
        u32x2 F0, Fn;
#define PAYWAIT() do {} while (0)
#define ACC2(x) do {} while (0)
#define STAGE(S, o, t) do { \
            if (__builtin_expect((int)rfl(min(min(D[0], D[1]), min(D[2], D[3]))) < (t) - 7, 0)) { \
                do { __builtin_amdgcn_s_sleep(1); D = *done; } while ((int)rfl(min(min(D[0], D[1]), min(D[2], D[3]))) < (t) - 7); } \
            ACC2(a_bar); PAYWAIT(); \
            unpack_set(S, rs, lds, o, lane); \
            asm volatile("" ::: "memory"); \
            *prog = (unsigned short)(((t) >> 2) + 1); \
            D = *done; ACC2(a_unp); } while (0)
#define TRYFETCH(S, pend, un, nx, valid) do { \
            const bool up_ = (valid) && __builtin_amdgcn_ballot_w64(Fn[1] < call) == 0; \
            pend = (valid) && !up_; \
            F0 = Fn; \
            Fn = load_flags(rs, fo, nx); \
            issue_set(S, F0, __builtin_amdgcn_make_buffer_rsrc(myws, 0, up_ ? WS_USED : 0u, 0x00020000), un, lane); \
            ACC2(a_iss); } while (0)
#define REPAIR(S, pend, un, nx) do { \
            if (pend) { bad |= wait_flags(F0, Fn, rs, myws, un, nx, lane, call); issue_set(S, F0, rs, un, lane); LASTREP(un); } \
            ACC2(a_flag); } while (0)
#define LASTREP(un) do {} while (0)
        if (v < 6) pack_own(p, v * 128 + g, call, lds + L_STAGE + 65536 + v * SCR_B, lane);
        bool pa = true, pb;
        u32 ua = UNIT(0), ub = UNIT(1);
        REPAIR(A, pa, ua, ub);
        STAMP(1);
        TRYFETCH(B, pb, ub, UNIT(2), true);
        BARRIER();
        u32x4 D = {0, 0, 0, 0};
        for (int i = 0; i < NSTEP / 2; i += 2) {
            const bool more = i < NSTEP / 2 - 2;
            const int t = 4 * i + ph;
            STAGE(A, oa, t);
            ua = UNIT(i + 2);
            REPAIR(B, pb, ub, ua);
            TRYFETCH(A, pa, ua, UNIT(i + 3), more);
            STAGE(B, oa + 65536, t + 4);
            ub = UNIT(i + 3);
            REPAIR(A, pa, ua, ub);
            TRYFETCH(B, pb, ub, UNIT(i + 4), more);
        }
        STAMP(2);
        PUT(6, lr_t); PUT(13, lr_u * 100); PUT(14, lr_n * 100);
        BARRIER();
        STAMP(3);
        u32 gm;
        {
            const char* ap = myws + AMAX_OFF + (v * 128 + lane) * 8;
            u64 t0 = __builtin_amdgcn_s_memrealtime();
    u32 it = 0;
            for (;;) {
                const u64 a0 = ld_sys64(ap), a1 = ld_sys64(ap + 512);
                const u32 tm = __hip_atomic_load((GAS u32*)(myws + TMO_OFF), __ATOMIC_RELAXED, __HIP_MEMORY_SCOPE_AGENT);
                gm = max((u32)a0, (u32)a1);
                const u32 tg = min((u32)(a0 >> 32), (u32)(a1 >> 32));
                if (rfl(tm) == call) { bad = true; break; }
                if (__builtin_amdgcn_ballot_w64(tg < call) == 0) break;
                if (spin_expired(myws, call, t0, it)) { bad = true; break; }
            }
        }
        gm = wave_umax(gm);
        if (bad) gm = 0xFFFFFFFFu;
        if (lane == 0) *(LAS u32*)(lds + L_MISC + 32 + v * 4) = gm;
        STAMP(4);
        BARRIER();
        return;
    }

    const int wm = wave >> 1, wn = wave & 1;
    char* amp = p.ws[0];
#pragma unroll
    for (int r = 1; r < NR; ++r) amp = (lane & 7) == r ? p.ws[r] : amp;
    amp += AMAX_OFF + (rank * 128 + g) * 8;
    asm volatile("" : "+v"(amp));
    f32x16 acc[2][2];
#pragma unroll
    for (int i = 0; i < 4; ++i)
#pragma unroll
        for (int j = 0; j < 16; ++j) acc[i >> 1][i & 1][j] = 0.f;
    {
        u32 xb = L_STAGE + wm * 4096 + lane * 16, wb = L_STAGE + (2 + wn) * 4096 + lane * 16;
        volatile LAS u32* prog = (volatile LAS u32*)(lds + L_MISC + 64);
        volatile LAS u32* done = (volatile LAS u32*)(lds + L_MISC + 96 + wave * 4);
        bf16x8 x0[4], w0[4], x1[4], w1[4];
#define SB() __builtin_amdgcn_sched_barrier(0)
#define RD1(X, W, off, f) do { W[f] = *(LAS bf16x8*)(lds + wb + (off) + (f) * 1024); X[f] = *(LAS bf16x8*)(lds + xb + (off) + (f) * 1024); SB(); } while (0)
#define MM1(X, W, i) do { acc[((i) >> 1) & 1][(i) & 1] = __builtin_amdgcn_mfma_f32_32x32x16_bf16( \
            W[((i) >> 2) * 2 + ((i) & 1)], X[(i) >> 1], acc[((i) >> 1) & 1][(i) & 1], 0, 0, 0); \
            asm volatile("" : "+v"(acc[((i) >> 1) & 1][(i) & 1])); SB(); } while (0)
#define LACKS(need) (((rfl(P) - (u32)(need) * 0x10001u) & 0x80008000u) != 0)
#define AWAIT(j, need) do { \
            if (__builtin_expect(LACKS(need), 0)) { do { P = prog[(j) & 3]; } while (LACKS(need)); } \
            asm volatile("" ::: "memory"); SB(); } while (0)
#define RELEASE(j, t) do { asm volatile("" ::: "memory"); *done = (t) + 1; P = prog[((j) + 1) & 3]; SB(); } while (0)
#define STEP(X, W, XN, WN, j) do { constexpr u32 o_ = ((j) & 3) * 16384; \
            MM1(X, W, 0); AWAIT(j, (t4 >> 2) + ((j) >> 2) + 1); MM1(X, W, 1); \
            if ((j) == 4) { xb ^= 65536u; wb ^= 65536u; } \
            RD1(XN, WN, o_, 0); RD1(XN, WN, o_, 1); MM1(X, W, 2); \
            RD1(XN, WN, o_, 2); RD1(XN, WN, o_, 3); MM1(X, W, 3); RELEASE(j, t4 + (j)); \
            MM1(X, W, 4); MM1(X, W, 5); MM1(X, W, 6); MM1(X, W, 7); } while (0)
        if (threadIdx.x < 12) *(LAS u32*)(lds + L_MISC + 64 + threadIdx.x * 4) = 0;
        BARRIER();
        if (threadIdx.x == 0 && call) __hip_atomic_store(ctr, call, __ATOMIC_RELAXED, __HIP_MEMORY_SCOPE_AGENT);
        u32 P = 0;
        AWAIT(0, 1);
        RD1(x0, w0, 0, 0); RD1(x0, w0, 0, 1); RD1(x0, w0, 0, 2); RD1(x0, w0, 0, 3);
        RELEASE(0, 0);
        int t4 = 0;
        for (; t4 < 2 * NSTEP - 4; t4 += 4) {
            STEP(x0, w0, x1, w1, 1); STEP(x1, w1, x0, w0, 2); STEP(x0, w0, x1, w1, 3); STEP(x1, w1, x0, w0, 4);
        }
        STEP(x0, w0, x1, w1, 1); STEP(x1, w1, x0, w0, 2); STEP(x0, w0, x1, w1, 3);
#pragma unroll
        for (int i = 0; i < 8; ++i) MM1(x1, w1, i);
    }
    STAMP(2);
    u32 y[64];
    u32 mx = 0;
#pragma unroll
    for (int e = 0; e < 64; ++e) {
        y[e] = (u32)max((int)__float_as_uint(acc[e >> 5][e & 1][(e >> 1) & 15]), 0);
        mx = max(mx, y[e]);
    }
    mx = wave_umax(mx);
    if (lane == 0) *(LAS u32*)(lds + L_MISC + 16 + wave * 4) = mx;
    BARRIER();
    if (wave == 0) {
        u32x4 wx = *(LAS u32x4*)(lds + L_MISC + 16);
        u32 tm = max(max(wx[0], wx[1]), max(wx[2], wx[3]));
        if (lane < NR) st_sys64(amp, ((u64)call << 32) | tm);
        STAMP(6);
    }

    u32 mk0 = 0, mk1 = 0;
#pragma unroll
    for (int e = 31; e >= 0; --e) {
        mk0 = __builtin_amdgcn_alignbit(mk0, 0u - y[e], 31);
        mk1 = __builtin_amdgcn_alignbit(mk1, 0u - y[e + 32], 31);
    }
    const u32 cnt = __builtin_popcount(mk0) + __builtin_popcount(mk1);
    const u32 incl = wave_incl_scan(cnt);
    const u32 total = __builtin_amdgcn_readlane(incl, 63);
    const int uq = g * 4 + wave;
    const int rot = uq % 7;
    const u32 uoff = P2_OFF + (rank * 512 + uq) * P2_B;
    char* pd[7];
#pragma unroll
    for (int k = 0; k < 7; ++k) pd[k] = p.peer[rot + k];
    asm volatile("" : "+s"(pd[0]), "+s"(pd[1]), "+s"(pd[2]), "+s"(pd[3]), "+s"(pd[4]), "+s"(pd[5]), "+s"(pd[6]));
    u32x4 mv;
    {
        auto s0 = __builtin_amdgcn_permlane32_swap(mk0, mk0, false, false);
        auto s1 = __builtin_amdgcn_permlane32_swap(mk1, mk1, false, false);
        mv = u32x4{s0[0], s1[0], s0[1], s1[1]};
    }
#define MA 3
#define MB 5
#define MASKS(k0, k1) do { if (lane < 32) { _Pragma("unroll") for (int k = (k0); k < (k1); ++k) st_wt128s(pd[k], uoff + lane * 16, mv); } } while (0)
    MASKS(0, MA);
    LAS char* cb = lds + L_COMP + wave * 16384;
    {
        u32 pos = L_COMP + wave * 16384 + (incl - cnt) * 4;
#define C1(n) "v_cmpx_lt_i32_e32 vcc, 0, %" #n "\n\tds_write_b32 %0, %" #n "\n\tv_add_u32 %0, 4, %0\n\ts_mov_b64 exec, -1\n\t"
#pragma unroll
        for (int e = 0; e < 64; e += 8) {
            if (e == 32 && MA < MB) { asm volatile("s_waitcnt vmcnt(1)" ::: "memory"); MASKS(MA, MB); }
            asm volatile(C1(1) C1(2) C1(3) C1(4) C1(5) C1(6) C1(7) C1(8)
                         : "+v"(pos)
                         : "v"(y[e]), "v"(y[e + 1]), "v"(y[e + 2]), "v"(y[e + 3]), "v"(y[e + 4]), "v"(y[e + 5]),
                           "v"(y[e + 6]), "v"(y[e + 7])
                         : "memory", "vcc");
        }
        asm volatile("s_nop 3");
        if (lane < 16 && total + lane < 4096u) *(LAS u32*)(cb + (total + lane) * 4) = mx;
    }
    const u32 ng = (total + 15) >> 4;
    f32x4 cv[4][4];
#pragma unroll
    for (int r = 0; r < 4; ++r)
#pragma unroll
        for (int j = 0; j < 4; ++j) cv[r][j] = *(LAS f32x4*)(cb + r * 4096 + lane * 64 + j * 16);
    STAMP(3);
    if (MB < NEARLY) { asm volatile("s_waitcnt vmcnt(1)" ::: "memory"); MASKS(MB, NEARLY); }
    BARRIER();
    float inv;
    {
        u32x4 gv = *(LAS u32x4*)(lds + L_MISC + 32), gw = *(LAS u32x4*)(lds + L_MISC + 48);
        float amax = __uint_as_float(max(max(max(gv[0], gv[1]), max(gv[2], gv[3])), max(max(gw[0], gw[1]), max(gw[2], gw[3]))));
        inv = 1.0f / (amax / 448.0f);
    }
    STAMP(4);
    u32x4 q[4];
#pragma unroll
    for (int r = 0; r < 4; ++r) {
        q[r] = u32x4{0x7F7F7F7Fu, 0x7F7F7F7Fu, 0x7F7F7F7Fu, 0x7F7F7F7Fu};
        if (r < 2 || (u32)(r * 64) < ng) {
#pragma unroll
            for (int j = 0; j < 4; ++j) {
                f32x4 t = cv[r][j] * inv;
                int w = __builtin_amdgcn_cvt_pk_fp8_f32(t[0], t[1], 0, false);
                q[r][j] = (u32)__builtin_amdgcn_cvt_pk_fp8_f32(t[2], t[3], w, true);
            }
        }
    }
    u32 h2, nb;
    {
        const u32 cmax = rfl((u32)__builtin_amdgcn_cvt_pk_fp8_f32(__uint_as_float(mx) * inv, 0.f, 0, false)) & 0xFFu;
        const u32 wl = max(cmax, 63u) - 63u, k4 = (0x80u - wl) * 0x01010101u;
        u32 m[2] = {0, 0};
#pragma unroll
        for (int r = 0; r < 4; ++r) {
            u32 t[4];
#pragma unroll
            for (int j = 0; j < 4; ++j) { t[j] = q[r][j] + k4; m[r >> 1] = bfi(0x80808080u, t[j], m[r >> 1] >> 1); }
            if (r < 2 || (u32)(r * 64) < ng) {
                LAS u32* o = (LAS u32*)(cb + lane * 12 + r * 768);
                o[0] = bfi(0x3F3F3F3Fu, t[0], t[3] << 6);
                o[1] = bfi(0x3F3F3F3Fu, t[1], t[3] << 4);
                o[2] = bfi(0x3F3F3F3Fu, t[2], t[3] << 2);
            }
        }
#pragma unroll
        for (int h = 0; h < 2; ++h)
            m[h] = ~m[h] & (((u32)(lane + 128 * h) < ng ? 0x0F0F0F0Fu : 0u) | ((u32)(lane + 128 * h + 64) < ng ? 0xF0F0F0F0u : 0u));
        const u32 ec = __builtin_popcount(m[0]) + __builtin_popcount(m[1]);
        const u32 ei = wave_incl_scan(ec);
        const u32 ne = __builtin_amdgcn_readlane(ei, 63);
        if (__builtin_expect(ne > MAXE2 || cmax > 126u, 0)) {
            h2 = total | H2_RAW;
            nb = ng * 16;
        } else {
            h2 = total | (ne << 13) | (wl << 21);
            nb = (ng * 12 + ne * 2 + 63) & ~63u;
            LAS unsigned short* el = (LAS unsigned short*)(cb + ng * 12) + (ei - ec);
            u64 mm = ((u64)m[1] << 32) | m[0];
            while (mm != 0) {
                const u32 bp = __builtin_ctzll(mm);
                *el++ = (unsigned short)(lane * 16 + (bp >> 5) * 2048 + (bp & 4) * 256 + (bp & 3) * 4 + ((bp >> 3) & 3));
                mm &= mm - 1;
            }
#pragma unroll
            for (int r = 0; r < 4; ++r) q[r] = *(LAS u32x4*)(cb + r * 1024 + lane * 16);
        }
    }
#pragma unroll
    for (int k = 0; k < 7; ++k) {
        char* d = pd[k];
        if (k >= NEARLY && lane < 32) st_wt128s(d, uoff + lane * 16, mv);
#pragma unroll
        for (int r = 0; r < 4; ++r)
            if ((u32)(lane * 16 + r * 1024) < nb) st_wt128s(d, uoff + 512 + r * 1024 + lane * 16, q[r]);
        if (k == 0) {
            u32x4 ot[4];
#pragma unroll
            for (int mt = 0; mt < 4; ++mt) {
                u32x4 ov;
#pragma unroll
                for (int j = 0; j < 4; ++j) {
                    const int e = mt * 16 + j * 4;
                    int w = __builtin_amdgcn_cvt_pk_fp8_f32(__uint_as_float(y[e]) * inv, __uint_as_float(y[e + 1]) * inv, 0, false);
                    ov[j] = (u32)__builtin_amdgcn_cvt_pk_fp8_f32(__uint_as_float(y[e + 2]) * inv, __uint_as_float(y[e + 3]) * inv, w, true);
                }
                ot[mt] = ov;
            }
            store_tile(p.out + (size_t)((rank >> 2) * 2048 + tr * 128 + wm * 64) * NN + (rank & 3) * 1024 + tc * 128 + wn * 64,
                       ot, lane);
        }
        WAIT_VM0();
        if (lane == 0) st_sys64(d + F2_OFF + (rank * 512 + uq) * FL_B, ((u64)call << 32) | h2);
    }
    STAMP(5);
}

extern "C" void kernel_launch(int rank, int nranks, ncclComm_t comm, hipStream_t stream,
                              void* const* d_in, const int* in_sizes, int n_in,
                              void* d_out, int out_size, const int* in_dtypes, int out_dtype,
                              void* const* d_out_all, void* const* d_ws_all, size_t ws_bytes) {
    (void)comm; (void)in_dtypes; (void)out_dtype; (void)d_out_all;
    if (nranks != NR || (unsigned)rank >= (unsigned)NR || n_in < 2 || ws_bytes < (size_t)SPARE_OFF + WS_USED ||
        in_sizes[0] != MM * KS || in_sizes[1] != KS * NN || out_size != MM * NN) {
        fprintf(stderr, "kernel_launch: unexpected problem shape\n");
        return;
    }
    HIP_CHECK(hipFuncSetAttribute((const void*)fused_kernel, hipFuncAttributeMaxDynamicSharedMemorySize, LDS_BYTES));
    Params p;
    p.x = (const char*)d_in[0];
    p.w = (const char*)d_in[1];
    p.out = (char*)d_out;
    for (int r = 0; r < NR; ++r) p.ws[r] = (char*)d_ws_all[r];
    for (int r = 0; r < NR; ++r) p.wsd[r] = p.ws[r] + (r == rank ? SPARE_OFF : 0u);
    for (int k = 0; k < 14; ++k) p.peer[k] = p.ws[(rank + 1 + k % 7) & 7];
    p.rank = rank;
    p.pad = 0;
    fused_kernel<<<dim3(256), dim3(768), LDS_BYTES, stream>>>(p);
    HIP_CHECK(hipGetLastError());
}
```

```cpp
#include <hip/hip_runtime.h>
#include <rccl/rccl.h>
#include <cstdio>
#include <cstdint>

#define HIP_CHECK(expr) do { hipError_t _e = (expr); if (_e != hipSuccess) { \
    fprintf(stderr, "HIP error %s at %s:%d\n", hipGetErrorString(_e), __FILE__, __LINE__); \
    return; } } while (0)

typedef unsigned u32;
typedef unsigned long long u64;
typedef u32 u32x2 __attribute__((ext_vector_type(2)));
typedef u32 u32x3 __attribute__((ext_vector_type(3)));
typedef u32 u32x4 __attribute__((ext_vector_type(4)));
typedef float f32x2 __attribute__((ext_vector_type(2)));
typedef float f32x4 __attribute__((ext_vector_type(4)));
typedef float f32x16 __attribute__((ext_vector_type(16)));
typedef unsigned short u16x2 __attribute__((ext_vector_type(2)));
typedef __bf16 bf16x8 __attribute__((ext_vector_type(8)));
#define GAS __attribute__((address_space(1)))
#define LAS __attribute__((address_space(3)))
#define DI __device__ __forceinline__

constexpr int NR = 8;
constexpr int MM = 4096, NN = 4096, KS = 512;
constexpr int NCH = 16;
constexpr int NSLOT = 48;
constexpr u32 NITEM = NCH * 128;
#define NEARLY 6
#define INDELAY 16
#define OWNDELAY 0
#define KW 4
static_assert(KW <= 6, "the ticket must be older than KW instructions");
#define NPUSH 96
constexpr int NSTEP = 64;
constexpr u32 UNIT_B = 3072, RAWU_B = 4096, FL_B = 128;
constexpr u32 CTR_OFF = 0;
constexpr u32 TMO_OFF = 32768;
constexpr u32 TKT_OFF = 33024;
constexpr u32 AMAX_OFF = 65536;
constexpr u32 F1_OFF = 131072;
constexpr u32 F2_OFF = 1u << 20;
constexpr u32 DBG_OFF = 1572864;
constexpr u32 P1_OFF = 2u << 20;
constexpr u32 RAW_OFF = 20u << 20;
constexpr u32 P2_OFF = 44u << 20;
constexpr u32 P2_B = 4608;
constexpr u32 WS_USED = 62u << 20;
constexpr u32 SPARE_OFF = 64u << 20;
static_assert(F1_OFF + NR * NCH * NSLOT * FL_B <= F2_OFF, "");
static_assert(F2_OFF + NR * 512 * FL_B <= DBG_OFF, "");
static_assert(P1_OFF + NR * NCH * NSLOT * UNIT_B <= RAW_OFF, "");
static_assert(RAW_OFF + NR * NCH * NSLOT * RAWU_B <= P2_OFF, "");
static_assert(P2_OFF + NR * 512 * P2_B <= WS_USED, "");
constexpr int CODE_BYTES = 38 * 1024;
constexpr u64 DEADLINE = 50000000ull;
constexpr int MAXEXC = 8;
constexpr u32 MAXL1 = 64;
constexpr u32 HDR_RAW = 0x1000;
constexpr int MAXE2 = 128;
constexpr u32 H2_RAW = 1u << 28;

constexpr u32 L_STAGE = 0;
constexpr u32 L_COMP = 0;
constexpr u32 L_MISC = 131072;
constexpr u32 L_DEC = 0;
constexpr u32 DEC_B = 4352;
constexpr u32 L_LUT = 53248;
constexpr u32 L_SCR = 57344;
constexpr u32 SCR_B = 5248;
constexpr u32 LDS_BYTES = 131072 + 512;

struct Params {
    const char* x;
    const char* w;
    char* me;
    int rank;
    int pad;
    char* wsd[NR];
    char* out;
    char* ws[NR];
    char* peer[14];
};

#define PAY_AUX 17
#define ACC(var) do {} while (0)
#define PUT(i, val) do {} while (0)
#define STAMP(i) do {} while (0)

#define PSTAMP() do {} while (0)
DI u32 perm(u32 hi, u32 lo, u32 sel) { return __builtin_amdgcn_perm(hi, lo, sel); }
DI u32 bfi(u32 m, u32 a, u32 b) { return (m & a) | (~m & b); }
DI u32 rotr(u32 v, u32 n) { return __builtin_amdgcn_alignbit(v, v, n); }
DI u32 rfl(u32 v) { return __builtin_amdgcn_readfirstlane(v); }
template <int CTRL, int RM> DI u32 dpp0(u32 v) { return __builtin_amdgcn_update_dpp(0u, v, CTRL, RM, 0xF, false); }
DI u32 wave_umax(u32 v) {
    v = max(v, dpp0<0x111, 0xF>(v)); v = max(v, dpp0<0x112, 0xF>(v));
    v = max(v, dpp0<0x114, 0xF>(v)); v = max(v, dpp0<0x118, 0xF>(v));
    v = max(v, dpp0<0x142, 0xA>(v)); v = max(v, dpp0<0x143, 0xC>(v));
    return __builtin_amdgcn_readlane(v, 63);
}
DI u32 wave_incl_scan(u32 v) {
    v += dpp0<0x111, 0xF>(v); v += dpp0<0x112, 0xF>(v);
    v += dpp0<0x114, 0xF>(v); v += dpp0<0x118, 0xF>(v);
    v += dpp0<0x142, 0xA>(v); v += dpp0<0x143, 0xC>(v);
    return v;
}
DI u64 ld_sys64(const char* p) {
    return __hip_atomic_load((GAS u64*)p, __ATOMIC_RELAXED, __HIP_MEMORY_SCOPE_SYSTEM);
}
DI void st_sys64(char* p, u64 v) {
    __hip_atomic_store((GAS u64*)p, v, __ATOMIC_RELAXED, __HIP_MEMORY_SCOPE_SYSTEM);
}
DI void st_wt128(char* p, u32x4 v) {
    asm volatile("global_store_dwordx4 %0, %1, off sc0 sc1\n\ts_nop 1" :: "v"(p), "v"(v) : "memory");
}
DI void st_wt128s(const char* sbase, u32 voff, u32x4 v) {
    asm volatile("global_store_dwordx4 %0, %1, %2 sc0 sc1\n\ts_nop 1" :: "v"(voff), "v"(v), "s"(sbase) : "memory");
}
#define WAIT_VM0() asm volatile("s_waitcnt vmcnt(0)" ::: "memory")
#define WAIT_LGKM0() asm volatile("s_waitcnt lgkmcnt(0)" ::: "memory")
#define BARRIER() do { WAIT_LGKM0(); __builtin_amdgcn_s_barrier(); asm volatile("" ::: "memory"); } while (0)

DI bool spin_expired(char* myws, u32 call, u64 t0, u32& it) {
    if ((++it & 15u) == 0) {
        GAS u32* tmo = (GAS u32*)(myws + TMO_OFF);
        u32 t = rfl(__hip_atomic_load(tmo, __ATOMIC_RELAXED, __HIP_MEMORY_SCOPE_AGENT));
        if (t == call || __builtin_amdgcn_s_memrealtime() - t0 > DEADLINE) {
            __hip_atomic_store(tmo, call, __ATOMIC_RELAXED, __HIP_MEMORY_SCOPE_AGENT);
            return true;
        }
    }
    __builtin_amdgcn_s_sleep(1);
    return false;
}
DI bool tmo_set(char* myws, u32 call) {
    return rfl(__hip_atomic_load((GAS u32*)(myws + TMO_OFF), __ATOMIC_RELAXED, __HIP_MEMORY_SCOPE_AGENT)) == call;
}

DI u32 twice(u32 v) {
    const u32 m = v & 0x7FFFu;
    return (v & 0x8000u) | (m < 0x80u ? m << 1 : m >= 0x7F00u ? max(m, 0x7F80u) : m + 0x80u);
}
DI u32 pack_unit(const u32x4 (&R)[4], u32x4& p0, u32x4& p1, u32x4& p2, u64& act, LAS char* scr, int lane,
                 char* d0, char* d1, char* d2, char* d3, char* fl, int nd, u32 unit) {
    u32 L[8], H[8];
    u16x2 mx = {0, 0};
#pragma unroll
    for (int f = 0; f < 4; ++f) {
        const u32x4 r = R[f] + 0x00800080u;
        L[2 * f] = perm(r[1], r[0], 0x06040200u); H[2 * f] = perm(r[1], r[0], 0x07050301u);
        L[2 * f + 1] = perm(r[3], r[2], 0x06040200u); H[2 * f + 1] = perm(r[3], r[2], 0x07050301u);
#pragma unroll
        for (int d = 0; d < 4; ++d)
            mx = __builtin_elementwise_max(mx, __builtin_bit_cast(u16x2, R[f][d] & 0x7FFF7FFFu));
    }
    const u32 big = wave_umax(max((u32)mx[0], (u32)mx[1]));
    u32 top = (big + 0x80u) >> 8;
    bool raw = big >= 0x7F00u || top < 10u;
    u32 ct = 0;
#pragma unroll
    for (int i = 0; i < 8; ++i) ct += __builtin_popcount(((H[i] | 0x80808080u) - top * 0x01010101u) & 0x80808080u);
    if (__builtin_amdgcn_readlane(wave_incl_scan(ct), 63) <= 3u) top -= 1u;
    const u32 base = (top - 3u) & 0x7Fu, base4 = base * 0x01010101u;
    u32 T[8], N[8], far = 0, m1 = 0;
#pragma unroll
    for (int i = 0; i < 8; ++i) {
        T[i] = (H[i] | 0x80808080u) - base4;
        far |= (T[i] + 0x04040404u) ^ 0x80808080u;
        m1 = bfi(0x80808080u, T[i], m1 >> 1);
        N[i] = bfi(0x80808080u, H[i], T[i]);
    }
    LAS char* stg = scr + 4096 + 64;
    *(LAS u32*)(stg + 768 + (lane & 31) * 4) = 0;
    {
        LAS u32* o = (LAS u32*)(stg + lane * 12);
        o[0] = bfi(0x60606060u, N[6] << 5, bfi(0x83838383u, N[0], rotr(N[1], 29)));
        o[1] = bfi(0x60606060u, N[7] << 5, bfi(0x83838383u, N[2], rotr(N[3], 29)));
        o[2] = bfi(0x60606060u, bfi(0x20202020u, N[6] >> 2, N[7] >> 1), bfi(0x83838383u, N[4], rotr(N[5], 29)));
    }
    m1 = ~m1;
    const u32 c1 = __builtin_popcount(m1), i1 = wave_incl_scan(c1);
    u32 n1 = __builtin_amdgcn_readlane(i1, 63);
    raw = raw || n1 > MAXL1;
    if (!raw) {
        LAS unsigned short* el = (LAS unsigned short*)(stg + 768) + (i1 - c1);
        while (m1 != 0) {
            const u32 bp = __builtin_ctz(m1);
            *el++ = (unsigned short)((bp & 6) * 512 + lane * 16 + (bp & 1) * 8 + (bp >> 3) * 2 + 1);
            m1 &= m1 - 1;
        }
    }
    p0 = u32x4{L[0], L[1], L[2], L[3]};
    p1 = u32x4{L[4], L[5], L[6], L[7]};
    p2 = *(LAS u32x4*)(stg + lane * 16);
    u32 hdr = base;
    if (raw || __builtin_amdgcn_ballot_w64((far & 0xF8F8F8F8u) != 0) != 0) {
        u32 cnt = 0;
        if (!raw) {
#pragma unroll
            for (int f = 0; f < 4; ++f) *(LAS u32x4*)(scr + f * 1024 + lane * 16) = R[f];
            u32 m = 0;
#pragma unroll
            for (int i = 0; i < 8; ++i) {
                const u32 z = ((T[i] + 0x04040404u) ^ 0x80808080u) & 0xF8F8F8F8u;
                m |= ((((z & 0x7F7F7F7Fu) + 0x7F7F7F7Fu) | z) & 0x80808080u) >> (7 - i);
            }
            for (;;) {
                u64 bal = __builtin_amdgcn_ballot_w64(m != 0);
                if (bal == 0 || cnt > MAXEXC) break;
                if (m != 0) {
                    u32 bp = __builtin_ctz(m);
                    u32 addr = (bp & 6) * 512 + lane * 16 + (bp & 1) * 8 + (bp >> 3) * 2;
                    u32 v = twice(*(LAS unsigned short*)(scr + addr));
                    u32 slot = cnt + __builtin_amdgcn_mbcnt_hi((u32)(bal >> 32), __builtin_amdgcn_mbcnt_lo((u32)bal, 0));
                    if (slot < MAXEXC) *(LAS u32*)(scr + 4096 + slot * 4) = (addr << 16) | v;
                    m &= m - 1;
                }
                cnt += __builtin_popcountll(bal);
            }
        }
        if (raw || cnt > MAXEXC) {
            hdr = HDR_RAW;
            n1 = 0;
#pragma unroll
            for (int f = 0; f < 4; ++f) {
                u32x4 r;
#pragma unroll
                for (int d = 0; d < 4; ++d) r[d] = twice(R[f][d] & 0xFFFFu) | (twice(R[f][d] >> 16) << 16);
                const u32 o = RAW_OFF + unit * RAWU_B + f * 1024 + lane * 16;
                st_wt128s(d0, o, r);
                st_wt128s(d1, o, r);
                if (nd > 2) { st_wt128s(d2, o, r); st_wt128s(d3, o, r); }
            }
        } else {
            hdr |= cnt << 8;
            if (lane < 8 && (lane & 3) < nd) {
                u32x4 rec = *(LAS u32x4*)(scr + 4096 + (lane & 4) * 4);
                st_wt128(fl + 64 + (lane & 4) * 4, rec);
            }
        }
    }
    act = (1ull << (48 + ((n1 + 31) >> 5) * 4)) - 1;
    return hdr | (n1 << 16);
}

#define A_TICKET(adr) "v_mov_b32 v167, 1\n\ts_mov_b64 exec, 1\n\tglobal_atomic_add v167, %" #adr ", v167, off sc0\n\ts_mov_b64 exec, -1\n\t"
#define A_END(n) "s_mov_b64 exec, -1\n\ts_nop 1\n\ts_waitcnt vmcnt(" #n ")"
#define A_ST(so, v, d, off) "global_store_dwordx4 %" #so ", %" #v ", %" #d " offset:" #off " sc0 sc1\n\t"
#define A_ST4(so, a, b, c, m, d0, d1, d2, d3) \
    A_ST(so, a, d0, 0) A_ST(so, a, d1, 0) A_ST(so, a, d2, 0) A_ST(so, a, d3, 0) \
    A_ST(so, b, d0, 1024) A_ST(so, b, d1, 1024) A_ST(so, b, d2, 1024) A_ST(so, b, d3, 1024) \
    "s_mov_b64 exec, %" #m "\n\t" \
    A_ST(so, c, d0, 2048) A_ST(so, c, d1, 2048) A_ST(so, c, d2, 2048) A_ST(so, c, d3, 2048)

#define A_ST2(so, a, b, c, m, d0, d1) \
    A_ST(so, a, d0, 0) A_ST(so, a, d1, 0) A_ST(so, b, d0, 1024) A_ST(so, b, d1, 1024) \
    "s_mov_b64 exec, %" #m "\n\t" A_ST(so, c, d0, 2048) A_ST(so, c, d1, 2048)
#define A_LDX "global_load_dwordx4 %0, %4, %6\n\tglobal_load_dwordx4 %1, %5, %6\n\t" \
              "global_load_dwordx4 %2, %4, %6 offset:32\n\tglobal_load_dwordx4 %3, %5, %6 offset:32\n\t"
#define A_WL(i) "global_load_dword %" #i ", %16, %17\n\tv_add_u32 %16, 0x2000, %16\n\t"
#define A_LDW A_WL(0) A_WL(1) A_WL(2) A_WL(3) A_WL(4) A_WL(5) A_WL(6) \
              "global_load_dword %7, %16, %17\n\tv_add_u32 %16, 0x12000, %16\n\t" \
              A_WL(8) A_WL(9) A_WL(10) A_WL(11) A_WL(12) A_WL(13) A_WL(14) "global_load_dword %15, %16, %17\n\t"
#define X_OUT "=&v"(R[0]), "=&v"(R[1]), "=&v"(R[2]), "=&v"(R[3])
#define X_IN "v"(lx0), "v"(lx1), "s"(src), "v"(tka), "v"(so), "v"(p0), "v"(p1), "v"(p2), "s"(act)
#define W_OUT "=&v"(G[0]), "=&v"(G[1]), "=&v"(G[2]), "=&v"(G[3]), "=&v"(G[4]), "=&v"(G[5]), "=&v"(G[6]), \
              "=&v"(G[7]), "=&v"(G[8]), "=&v"(G[9]), "=&v"(G[10]), "=&v"(G[11]), "=&v"(G[12]), "=&v"(G[13]), \
              "=&v"(G[14]), "=&v"(G[15]), "+v"(lw)
#define W_IN "s"(src), "v"(tka), "v"(so), "v"(p0), "v"(p1), "v"(p2), "s"(act)

DI void w_to_frags(const u32 (&G)[16], u32x4 (&R)[4]) {
#pragma unroll
    for (int f = 0; f < 4; ++f)
#pragma unroll
        for (int d = 0; d < 4; ++d)
            R[f][d] = perm(G[(f >> 1) * 8 + 2 * d + 1], G[(f >> 1) * 8 + 2 * d], (f & 1) ? 0x07060302u : 0x05040100u);
}

struct Item { bool isw; u32 unit, r0; const char* src; char *d0, *d1, *d2, *d3;
};
DI Item make_item(const Params& p, u32 i) {
    Item it;
    const u32 c = (i >> 7) & (NCH - 1), k = (i >> 1) & 63;
    it.isw = i & 1;
    it.r0 = it.isw ? (k & 3) : (k & 1) * 4;
    const u32 sl = it.isw ? k >> 2 : k >> 1;
    it.unit = (p.rank * NCH + c) * NSLOT + (it.isw ? 32 : 0) + sl;
    it.src = it.isw ? p.w + (size_t)c * (32 * NN * 2) + ((k & 3) * 16 + sl) * 128
                    : p.x + (size_t)((k & 1) * 32 + sl) * (64 * KS * 2) + c * 64;
    return it;
}
DI void item_dests(const Params& p, Item& it) {
    const u32 r0 = it.r0, rs = it.isw ? 4 : 1;
    it.d0 = p.wsd[r0]; it.d1 = p.wsd[r0 + rs]; it.d2 = p.wsd[(r0 + 2 * rs) & 7]; it.d3 = p.wsd[(r0 + 3 * rs) & 7];
}

DI u32 lane_x(int lane) { return (u32)(lane & 31) * (KS * 2) + (lane >> 5) * 16; }
DI u32 lane_w(int lane) {
    return (u32)(8 * (lane >> 5)) * (NN * 2) + ((lane & 19) | ((lane & 4) << 1) | ((lane & 8) >> 1)) * 4;
}

DI void push_inputs(const Params& p, u32 g, LAS char* scr, int lane) {
    const u32 lx0 = lane_x(lane), lx1 = lx0 + 32 * KS * 2, lw0 = lane_w(lane);
    u32 G[16];
    u32x4 R[4], p0, p1, p2;
    {
        const int r0 = (g & 1) * 4;
        asm volatile("" :: "s"(p.x), "s"(p.w), "s"(p.me), "s"(p.rank), "s"(p.wsd[r0]), "s"(p.wsd[r0 + 1]),
                     "s"(p.wsd[r0 + 2]), "s"(p.wsd[r0 + 3]));
    }
    Item cur = make_item(p, 2 * g), nxt = make_item(p, 2 * g + 1);
    item_dests(p, cur);
    u32 call;
    {
        const char* src = nxt.src;
        u32 lw = lw0;
        asm volatile("global_load_dwordx4 %19, %23, %25\n\tglobal_load_dwordx4 %20, %24, %25\n\t"
                     "global_load_dwordx4 %21, %23, %25 offset:32\n\tglobal_load_dwordx4 %22, %24, %25 offset:32\n\t"
                     "global_load_dword %18, %26, %27 sc1\n\t" A_LDW "s_waitcnt vmcnt(16)"
                     : "=&v"(G[0]), "=&v"(G[1]), "=&v"(G[2]), "=&v"(G[3]), "=&v"(G[4]), "=&v"(G[5]), "=&v"(G[6]),
                       "=&v"(G[7]), "=&v"(G[8]), "=&v"(G[9]), "=&v"(G[10]), "=&v"(G[11]), "=&v"(G[12]), "=&v"(G[13]),
                       "=&v"(G[14]), "=&v"(G[15]), "+v"(lw), "+s"(src), "=&v"(call),
                       "=&v"(R[0]), "=&v"(R[1]), "=&v"(R[2]), "=&v"(R[3])
                     : "v"(lx0), "v"(lx1), "s"(cur.src), "v"(CTR_OFF + g * 128), "s"(p.me) : "memory");
        call = rfl(call) + 1u;
    }
    char* const tka = p.me + TKT_OFF;
    const u32 tbase = (call - 1u) * NITEM - 2u * NPUSH;
    {
        u32x4 q[2];
#pragma unroll
        for (int f = 0; f < 4; ++f) {
            const u32x4 r = R[f] + 0x00800080u;
            q[f >> 1][(f & 1) * 2] = perm(r[1], r[0], 0x06040200u);
            q[f >> 1][(f & 1) * 2 + 1] = perm(r[3], r[2], 0x06040200u);
        }
        const u32 so = P1_OFF + cur.unit * UNIT_B + lane * 16;
        asm volatile("s_nop 4\n\t" A_TICKET(23)
                     A_ST(16, 17, 19, 0) A_ST(16, 17, 20, 0) A_ST(16, 17, 21, 0) A_ST(16, 17, 22, 0)
                     A_ST(16, 18, 19, 1024) A_ST(16, 18, 20, 1024) A_ST(16, 18, 21, 1024) A_ST(16, 18, 22, 1024)
                     "s_nop 1\n\ts_waitcnt vmcnt(9)"
                     : "+v"(G[0]), "+v"(G[1]), "+v"(G[2]), "+v"(G[3]), "+v"(G[4]), "+v"(G[5]), "+v"(G[6]), "+v"(G[7]),
                       "+v"(G[8]), "+v"(G[9]), "+v"(G[10]), "+v"(G[11]), "+v"(G[12]), "+v"(G[13]), "+v"(G[14]), "+v"(G[15])
                     : "v"(so), "v"(q[0]), "v"(q[1]), "s"(cur.d0), "s"(cur.d1), "s"(cur.d2), "s"(cur.d3), "v"(tka)
                     : "memory", "v167");
    }
#define ITEMDONE() do {} while (0)
    bool more = true;
    char* fp = nullptr;
    u32 fv = 0;
    int fn = 0;
    for (;;) {
        const int nd = cur.isw ? 2 : 4;
        char* const dl = (lane & 1) ? cur.d1 : cur.d0;
        char* const dh = (lane & 1) ? cur.d3 : cur.d2;
        char* const fl = ((lane & 2) ? dh : dl) + F1_OFF + cur.unit * FL_B;
        u64 act;
        const u32 hdr = pack_unit(R, p0, p1, p2, act, scr, lane, cur.d0, cur.d1, cur.d2, cur.d3, fl, nd, cur.unit);
        const u32 so = P1_OFF + cur.unit * UNIT_B + lane * 16;
        if (fn != 0) {
            u32 tk;
            asm volatile("s_waitcnt vmcnt(%6)\n\tv_readfirstlane_b32 %0, v167\n\ts_nop 1"
                         : "=s"(tk), "+v"(p0), "+v"(p1), "+v"(p2) : "v"(so), "s"(act), "n"(KW) : "memory");
            const u32 i = tk - tbase;
            more = i < NITEM;
            nxt = make_item(p, i);
        }
        const char* src = nxt.src;
        if (nxt.isw) {
            u32 lw = lw0;
            if (fn == 0)
                asm volatile("s_nop 4\n\ts_mov_b64 exec, %2\n\t" A_ST(0, 1, 3, 2048) A_ST(0, 1, 4, 2048)
                             A_ST(0, 1, 5, 2048) A_ST(0, 1, 6, 2048) "s_mov_b64 exec, -1\n\ts_nop 1"
                             :: "v"(so), "v"(p2), "s"(act), "s"(cur.d0), "s"(cur.d1), "s"(cur.d2), "s"(cur.d3) : "memory");
            else if (cur.isw)
                asm volatile("s_nop 4\n\t" A_LDW A_TICKET(18) A_ST2(19, 20, 21, 22, 23, 24, 25) A_END(7)
                             : W_OUT : W_IN, "s"(cur.d0), "s"(cur.d1) : "memory", "v167");
            else
                asm volatile("s_nop 4\n\t" A_LDW A_TICKET(18) A_ST4(19, 20, 21, 22, 23, 24, 25, 26, 27) A_END(13)
                             : W_OUT : W_IN, "s"(cur.d0), "s"(cur.d1), "s"(cur.d2), "s"(cur.d3) : "memory", "v167");
            w_to_frags(G, R);
        } else {
            if (cur.isw)
                asm volatile("s_nop 4\n\t" A_LDX A_TICKET(7) A_ST2(8, 9, 10, 11, 12, 13, 14) A_END(7)
                             : X_OUT : X_IN, "s"(cur.d0), "s"(cur.d1) : "memory", "v167");
            else
                asm volatile("s_nop 4\n\t" A_LDX A_TICKET(7) A_ST4(8, 9, 10, 11, 12, 13, 14, 15, 16) A_END(13)
                             : X_OUT : X_IN, "s"(cur.d0), "s"(cur.d1), "s"(cur.d2), "s"(cur.d3) : "memory", "v167");
        }
        if (lane < fn) st_sys64(fp, ((u64)call << 32) | fv);
        if (fn != 0) ITEMDONE();
        fp = fl;
        fv = hdr;
        fn = nd;
        if (!more) break;
        cur = nxt;
        item_dests(p, cur);
    }
    WAIT_VM0();
    if (lane < fn) st_sys64(fp, ((u64)call << 32) | fv);
    ITEMDONE();
}

DI void pack_own(const Params& p, u32 s, u32 call, LAS char* scr, int lane) {
    const u32 c = s / NSLOT, sl = s % NSLOT;
    u32x4 R[4], p0, p1, p2;
    if (sl >= 32) {
        const char* src = p.w + (size_t)c * (32 * NN * 2) + ((p.rank & 3) * 16 + sl - 32) * 128 + lane_w(lane);
        u32 G[16];
#pragma unroll
        for (int j = 0; j < 16; ++j) G[j] = *(const GAS u32*)(src + ((j >> 3) * 16 + (j & 7)) * (NN * 2));
        w_to_frags(G, R);
    } else {
        const char* src = p.x + (size_t)((p.rank >> 2) * 32 + sl) * (64 * KS * 2) + c * 64 + lane_x(lane);
#pragma unroll
        for (int f = 0; f < 4; ++f) R[f] = *(const GAS u32x4*)(src + (f & 1) * (32 * KS * 2) + (f >> 1) * 32);
    }
    char* const me = p.me;
    const u32 unit = p.rank * (NCH * NSLOT) + s;
    char* const fl = me + F1_OFF + unit * FL_B;
    u64 act;
    const u32 hdr = pack_unit(R, p0, p1, p2, act, scr, lane, me, me, me, me, fl, 1, unit);
    const u32 so = P1_OFF + unit * UNIT_B + lane * 16;
    st_wt128s(me, so, p0); st_wt128s(me, so + 1024, p1); st_wt128s(me, so + 2048, p2);
    WAIT_VM0();
    if (lane == 0) st_sys64(fl, ((u64)call << 32) | hdr);
}

typedef __amdgpu_buffer_rsrc_t rsrc_t;
struct PSet { u32x4 lo[4]; u32x3 hi[2]; u32 lst, rec; u32 hdr0, hdr1; u32 unit; };

DI u32x2 load_flags(rsrc_t rs, u32 voff, u32 unit) {
    return __builtin_amdgcn_raw_buffer_load_b64(rs, voff, F1_OFF + unit * FL_B, 17);
}
DI bool wait_flags(u32x2& F0, u32x2& Fn, rsrc_t rs, char* myws, u32 unit, u32 next, int lane, u32 call) {
    u64 t0 = __builtin_amdgcn_s_memrealtime();
    u32 it = 0;
    bool bad = false;
    const u32 voff = F1_OFF + (((lane & 2) ? next : unit) + (lane & 1)) * FL_B;
    u32x2 F;
    for (;;) {
        asm volatile("" ::: "memory");
        F = __builtin_amdgcn_raw_buffer_load_b64(rs, voff, 0, 17);
        if ((__builtin_amdgcn_ballot_w64(F[1] < call) & 3) == 0) break;
        if (spin_expired(myws, call, t0, it)) { bad = true; break; }
    }
    F0 = F;
    Fn[0] = __builtin_amdgcn_update_dpp(0u, F[0], 0xEE, 0xF, 0xF, false);
    Fn[1] = __builtin_amdgcn_update_dpp(0u, F[1], 0xEE, 0xF, 0xF, false);
    return bad;
}
DI void issue_set(PSet& S, u32x2 F, rsrc_t rs, u32 unit, int lane) {
    asm volatile("" ::: "memory");
    S.hdr0 = __builtin_amdgcn_readlane(F[0], 0);
    S.hdr1 = __builtin_amdgcn_readlane(F[0], 1);
    S.unit = unit;
#pragma unroll
    for (int u = 0; u < 2; ++u) {
        S.hi[u] = __builtin_amdgcn_raw_buffer_load_b96(rs, lane * 12 + u * UNIT_B + 2048, P1_OFF + unit * UNIT_B, 17);
        S.lo[2 * u] = __builtin_amdgcn_raw_buffer_load_b128(rs, lane * 16 + u * UNIT_B, P1_OFF + unit * UNIT_B, 17);
        S.lo[2 * u + 1] = __builtin_amdgcn_raw_buffer_load_b128(rs, lane * 16 + u * UNIT_B + 1024, P1_OFF + unit * UNIT_B, 17);
    }
    const u32 n1 = ((lane & 32) ? S.hdr1 : S.hdr0) >> 16;
    S.lst = __builtin_amdgcn_raw_buffer_load_b32(
        rs, (u32)(lane & 31) * 2 < n1 ? (lane >> 5) * UNIT_B + 2816 + (lane & 31) * 4 : 0x80000000u, P1_OFF + unit * UNIT_B, 17);
    S.rec = __builtin_amdgcn_raw_buffer_load_b32(rs, (lane & 7) * 4 + 64 + (lane & 8) * (FL_B / 8), F1_OFF + unit * FL_B, 17);
}
DI void unpack_set(const PSet& S, rsrc_t rs, LAS char* lds, u32 oo, int lane) {
    LAS char* o = lds + oo;
#pragma unroll
    for (int u = 0; u < 2; ++u) {
        const u32 base4 = ((u ? S.hdr1 : S.hdr0) & 0x7F) * 0x01010101u;
#pragma unroll
        for (int f = 0; f < 4; ++f) {
            u32 h0, h1;
            if (f < 3) {
                const u32 w = S.hi[u][f];
                h0 = (w & 0x83838383u) + base4;
                h1 = (rotr(w, 3) & 0x83838383u) + base4;
            } else {
                h0 = ((S.hi[u][2] << 2) & 0x80808080u) | (((S.hi[u][0] >> 5) & 0x03030303u) + base4);
                h1 = ((S.hi[u][2] << 1) & 0x80808080u) | (((S.hi[u][1] >> 5) & 0x03030303u) + base4);
            }
            u32 l0 = S.lo[2 * u + (f >> 1)][(f & 1) * 2], l1 = S.lo[2 * u + (f >> 1)][(f & 1) * 2 + 1];
            u32x4 r = {perm(h0, l0, 0x05010400u), perm(h0, l0, 0x07030602u),
                       perm(h1, l1, 0x05010400u), perm(h1, l1, 0x07030602u)};
            *(LAS u32x4*)(o + u * 4096 + f * 1024 + lane * 16) = r;
            __builtin_amdgcn_sched_barrier(0);
        }
    }
    {
        const u32 ob = oo + (lane >> 5) * 4096;
        const u32 e0 = S.lst & 0xFFFFu, e1 = S.lst >> 16;
        const u32 a0 = (e0 & 0xFFCu) | ob, a1 = (e1 & 0xFFCu) | ob;
        const u32 v0 = 4u << ((e0 << 3) & 31u), v1 = 4u << ((e1 << 3) & 31u);
        asm volatile("v_cmpx_ne_u32_e32 vcc, 0, %0\n\tds_sub_u32 %2, %4\n\t"
                     "v_cmpx_ne_u32_e32 vcc, 0, %1\n\tds_sub_u32 %3, %5\n\ts_mov_b64 exec, -1\n\ts_nop 3"
                     :: "v"(e0), "v"(e1), "v"(a0), "v"(a1), "v"(v0), "v"(v1) : "memory", "vcc");
    }
    if (__builtin_expect(((S.hdr0 | S.hdr1) & 0xFF00u) != 0, 0)) {
        asm volatile("" ::: "memory");
        const u32 hdr = (lane & 8) ? S.hdr1 : S.hdr0;
        if (lane < 16 && (u32)(lane & 7) < ((hdr >> 8) & 15u))
            *(LAS unsigned short*)(o + (lane & 8) * 512 + ((S.rec >> 16) & 0xFFE)) = (unsigned short)S.rec;
        if (__builtin_expect(((S.hdr0 | S.hdr1) & HDR_RAW) != 0, 0)) {
#pragma unroll
            for (int u = 0; u < 2; ++u)
                if ((u ? S.hdr1 : S.hdr0) & HDR_RAW) {
#pragma unroll
                    for (int f = 0; f < 4; ++f)
                        *(LAS u32x4*)(o + u * 4096 + f * 1024 + lane * 16) = __builtin_amdgcn_raw_buffer_load_b128(
                            rs, lane * 16 + f * 1024, RAW_OFF + (S.unit + u) * RAWU_B, 17);
                }
        }
    }
}

DI void store_tile(char* o, const u32x4 (&v)[4], int lane) {
    o += (size_t)(lane & 15) * NN + ((lane >> 4) & 1) * 32 + (lane >> 5) * 16;
#pragma unroll
    for (int mi = 0; mi < 2; ++mi) {
        u32x4 a, b;
#pragma unroll
        for (int j = 0; j < 4; ++j) {
            auto r = __builtin_amdgcn_permlane16_swap(v[2 * mi][j], v[2 * mi + 1][j], false, false);
            a[j] = r[0]; b[j] = r[1];
        }
        st_wt128(o + (size_t)(32 * mi) * NN, a);
        st_wt128(o + (size_t)(32 * mi + 16) * NN, b);
    }
}

extern "C" __global__ __launch_bounds__(768, 3) void fused_kernel(Params p) {
    extern __shared__ __attribute__((aligned(16))) char smem_[];
    LAS char* lds = (LAS char*)smem_;
    const int lane = threadIdx.x & 63;
    const int wave = rfl(threadIdx.x >> 6);
    const int b = blockIdx.x, xcd = b & 7, idx = b >> 3;
    const int rank = p.rank;
    char* myws = p.me;
    STAMP(0);
    if (__builtin_expect(wave == 0 && b < NPUSH, 1)) push_inputs(p, b, lds + L_SCR, lane);

    GAS u32* ctr = (GAS u32*)(myws + CTR_OFF + b * 128);
    const u32 ctr_old = __hip_atomic_load(ctr, __ATOMIC_RELAXED, __HIP_MEMORY_SCOPE_AGENT);

    if (idx < 16) {
        const int g = xcd * 16 + idx;
        if (threadIdx.x >= 256 && threadIdx.x < 512) {
            u32 m = threadIdx.x - 256, n = 0, s[2] = {0, 0};
#pragma unroll
            for (int e = 0; e < 8; ++e) {
                u32 bit = (m >> e) & 1;
                s[e >> 2] |= (bit ? n : 0x0Cu) << ((e & 3) * 8);
                n += bit;
            }
            *(LAS u32x2*)(lds + L_LUT + m * 8) = u32x2{s[0], s[1]};
        }
        if (idx >= 14 && wave >= 1) {
            const char* pc;
            int lo;
            asm volatile("s_getpc_b64 %0\n.Lhere%=:\n\ts_mov_b32 %1, fused_kernel-.Lhere%=" : "=s"(pc), "=s"(lo));
            const int o = (((idx - 14) * 11 + wave - 1) * 64 + lane) * 64;
            if (o < CODE_BYTES) {
                const u32 v = *(const GAS u32*)(pc + lo + o);
                asm volatile("" :: "v"(v));
            }
        }
        const u32 call = rfl(ctr_old) + 1u;
        STAMP(1);
        BARRIER();
        if (threadIdx.x == 64) __hip_atomic_store(ctr, call, __ATOMIC_RELAXED, __HIP_MEMORY_SCOPE_AGENT);
        const rsrc_t rs = __builtin_amdgcn_make_buffer_rsrc(myws, 0, WS_USED, 0x00020000);
        const int uq = g * 4 + (wave & 3);
        const int rot = uq % 7;
        const int tr = (g >> 5) * 4 + ((g >> 2) & 3), tc = ((g >> 4) & 1) * 4 + (g & 3);
        LAS char* buf = lds + L_DEC + wave * DEC_B;
        for (int k = wave >> 2; k < 7; k += 3) {
            const int src = (rank + 7 - (k + rot) % 7) & 7;
            const u32 un = src * 512 + uq;
            bool bad = false;
            u64 F = ld_sys64(myws + F2_OFF + un * FL_B);
            if (rfl((u32)(F >> 32)) < call) {
                u64 t0 = __builtin_amdgcn_s_memrealtime();
    u32 it = 0;
                for (;;) {
                    F = ld_sys64(myws + F2_OFF + un * FL_B);
                    if (rfl((u32)(F >> 32)) >= call) break;
                    if (spin_expired(myws, call, t0, it)) { bad = true; break; }
                }
            }
            asm volatile("" ::: "memory");
            const u32 h2 = rfl((u32)F);
            const u32 total = min(h2 & 0x1FFFu, 4096u), ng = (total + 15) >> 4;
            const u32 so = P2_OFF + un * P2_B;
            const u32x2 mk = __builtin_amdgcn_raw_buffer_load_b64(rs, (lane & 31) * 16 + (lane >> 5) * 8, so, 17);
            if (__builtin_expect((h2 & H2_RAW) != 0, 0)) {
                u32x4 pl[4];
#pragma unroll
                for (int r = 0; r < 4; ++r)
                    pl[r] = __builtin_amdgcn_raw_buffer_load_b128(
                        rs, (u32)(lane + r * 64) < ng ? (u32)(512 + lane * 16 + r * 1024) : 0x80000000u, so, 17);
                bad = bad || tmo_set(myws, call);
#pragma unroll
                for (int r = 0; r < 4; ++r) *(LAS u32x4*)(buf + r * 1024 + lane * 16) = pl[r];
            } else {
                const u32 ne = min((h2 >> 13) & 0xFFu, (u32)MAXE2), w4 = ((h2 >> 21) & 0x3Fu) * 0x01010101u;
                u32x3 pl[4];
#pragma unroll
                for (int r = 0; r < 4; ++r)
                    pl[r] = __builtin_amdgcn_raw_buffer_load_b96(
                        rs, (u32)(lane + r * 64) < ng ? (u32)(512 + lane * 12 + r * 768) : 0x80000000u, so, 17);
                u32 ex[2];
#pragma unroll
                for (int r = 0; r < 2; ++r)
                    ex[r] = __builtin_amdgcn_raw_buffer_load_b16(
                        rs, (u32)(lane + r * 64) < ne ? 512 + ng * 12 + lane * 2 + r * 128 : 0x80000000u, so, 17);
                bad = bad || tmo_set(myws, call);
#pragma unroll
                for (int r = 0; r < 4; ++r)
                    if (r < 2 || (u32)(r * 64) < ng) {
                        const u32 d3 = bfi(0x03030303u, pl[r][0] >> 6, bfi(0x0C0C0C0Cu, pl[r][1] >> 4, (pl[r][2] >> 2) & 0x30303030u));
                        *(LAS u32x4*)(buf + r * 1024 + lane * 16) =
                            u32x4{(pl[r][0] & 0x3F3F3F3Fu) + w4, (pl[r][1] & 0x3F3F3F3Fu) + w4, (pl[r][2] & 0x3F3F3F3Fu) + w4, d3 + w4};
                    }
#pragma unroll
                for (int r = 0; r < 2; ++r)
                    if ((u32)(lane + r * 64) < ne)
                        __hip_atomic_fetch_and((LAS u32*)(buf + (ex[r] & 0xFFCu)), ~(0x40u << ((ex[r] & 3u) * 8)),
                                               __ATOMIC_RELAXED, __HIP_MEMORY_SCOPE_WAVEFRONT);
                asm volatile("" ::: "memory");
            }
            u32 cnt = __builtin_popcount(mk[0]) + __builtin_popcount(mk[1]);
            u32 o = wave_incl_scan(cnt) - cnt;
            u32x4 ot[4];
#pragma unroll
            for (int mt = 0; mt < 4; ++mt) {
                u32x4 ov;
#pragma unroll
                for (int hh = 0; hh < 2; ++hh) {
                    u32 mb = (mk[mt >> 1] >> ((mt & 1) * 16 + hh * 8)) & 0xFF;
                    u32x2 sel = *(LAS u32x2*)(lds + L_LUT + mb * 8);
                    LAS u32* dp = (LAS u32*)(buf + (o & ~3u));
                    u32 d0 = dp[0], d1 = dp[1], d2 = dp[2];
                    u32 a0 = __builtin_amdgcn_alignbyte(d1, d0, o), a1 = __builtin_amdgcn_alignbyte(d2, d1, o);
                    ov[2 * hh] = perm(a1, a0, sel[0]);
                    ov[2 * hh + 1] = perm(a1, a0, sel[1]);
                    o += __builtin_popcount(mb);
                }
                if (bad) ov = u32x4{0x7F7F7F7Fu, 0x7F7F7F7Fu, 0x7F7F7F7Fu, 0x7F7F7F7Fu};
                ot[mt] = ov;
            }
            store_tile(p.out + (size_t)((src >> 2) * 2048 + tr * 128 + ((wave >> 1) & 1) * 64) * NN +
                       (src & 3) * 1024 + tc * 128 + (wave & 1) * 64, ot, lane);
            STAMP(2 + k / 3);
        }
        return;
    }

    const int g = xcd * 16 + (idx - 16);
    const int tr = (g >> 5) * 4 + ((g >> 2) & 3), tc = ((g >> 4) & 1) * 4 + (g & 3);
    const u32 call = rfl(ctr_old) + 1u;

    if (wave >= 4) {
        const int v = wave - 4, ph = v >> 1;
        const int slot = (v & 1) ? 32 + 2 * tc : 2 * tr;
        const rsrc_t rs = __builtin_amdgcn_make_buffer_rsrc(myws, 0, WS_USED, 0x00020000);
        const u32 oa = L_STAGE + ph * 16384 + (v & 1) * 8192;
        volatile LAS unsigned short* prog = (volatile LAS unsigned short*)(lds + L_MISC + 64 + v * 2);
        volatile LAS u32x4* done = (volatile LAS u32x4*)(lds + L_MISC + 96);
        const u32 fo = (lane & 1) * FL_B;
        u32 utab;
        {
            const u32 t = min(4 * lane + ph, 2 * NSTEP - 1), r = t - NCH;
            utab = (t < NCH ? rank * NCH + t : ((rank + 1 + r % 7) & 7) * NCH + r / 7) * NSLOT + slot;
        }
#define UNIT(i) ((u32)__builtin_amdgcn_readlane(utab, (i)))
        PSet A, B;
        bool bad = false;
        u32x2 F0, Fn;
#define PAYWAIT() do {} while (0)
#define ACC2(x) do {} while (0)
#define STAGE(S, o, t) do { \
            if (__builtin_expect((int)rfl(min(min(D[0], D[1]), min(D[2], D[3]))) < (t) - 7, 0)) { \
                do { __builtin_amdgcn_s_sleep(1); D = *done; } while ((int)rfl(min(min(D[0], D[1]), min(D[2], D[3]))) < (t) - 7); } \
            ACC2(a_bar); PAYWAIT(); \
            unpack_set(S, rs, lds, o, lane); \
            asm volatile("" ::: "memory"); \
            *prog = (unsigned short)(((t) >> 2) + 1); \
            D = *done; ACC2(a_unp); } while (0)
#define TRYFETCH(S, pend, un, nx, valid) do { \
            const bool up_ = (valid) && __builtin_amdgcn_ballot_w64(Fn[1] < call) == 0; \
            pend = (valid) && !up_; \
            F0 = Fn; \
            Fn = load_flags(rs, fo, nx); \
            issue_set(S, F0, __builtin_amdgcn_make_buffer_rsrc(myws, 0, up_ ? WS_USED : 0u, 0x00020000), un, lane); \
            ACC2(a_iss); } while (0)
#define REPAIR(S, pend, un, nx) do { \
            if (pend) { bad |= wait_flags(F0, Fn, rs, myws, un, nx, lane, call); issue_set(S, F0, rs, un, lane); LASTREP(un); } \
            ACC2(a_flag); } while (0)
#define LASTREP(un) do {} while (0)
        __builtin_amdgcn_s_sleep(OWNDELAY);
        if (v < 6) pack_own(p, v * 128 + g, call, lds + L_STAGE + 65536 + v * SCR_B, lane);
        bool pa = true, pb;
        u32 ua = UNIT(0), ub = UNIT(1);
        REPAIR(A, pa, ua, ub);
        STAMP(1);
        TRYFETCH(B, pb, ub, UNIT(2), true);
        BARRIER();
        u32x4 D = {0, 0, 0, 0};
        for (int i = 0; i < NSTEP / 2; i += 2) {
            const bool more = i < NSTEP / 2 - 2;
            const int t = 4 * i + ph;
            STAGE(A, oa, t);
            ua = UNIT(i + 2);
            REPAIR(B, pb, ub, ua);
            TRYFETCH(A, pa, ua, UNIT(i + 3), more);
            STAGE(B, oa + 65536, t + 4);
            ub = UNIT(i + 3);
            REPAIR(A, pa, ua, ub);
            TRYFETCH(B, pb, ub, UNIT(i + 4), more);
        }
        STAMP(2);
        PUT(6, lr_t); PUT(13, lr_u * 100); PUT(14, lr_n * 100);
        BARRIER();
        STAMP(3);
        u32 gm;
        {
            const char* ap = myws + AMAX_OFF + (v * 128 + lane) * 8;
            u64 t0 = __builtin_amdgcn_s_memrealtime();
    u32 it = 0;
            for (;;) {
                const u64 a0 = ld_sys64(ap), a1 = ld_sys64(ap + 512);
                const u32 tm = __hip_atomic_load((GAS u32*)(myws + TMO_OFF), __ATOMIC_RELAXED, __HIP_MEMORY_SCOPE_AGENT);
                gm = max((u32)a0, (u32)a1);
                const u32 tg = min((u32)(a0 >> 32), (u32)(a1 >> 32));
                if (rfl(tm) == call) { bad = true; break; }
                if (__builtin_amdgcn_ballot_w64(tg < call) == 0) break;
                if (spin_expired(myws, call, t0, it)) { bad = true; break; }
            }
        }
        gm = wave_umax(gm);
        if (bad) gm = 0xFFFFFFFFu;
        if (lane == 0) *(LAS u32*)(lds + L_MISC + 32 + v * 4) = gm;
        STAMP(4);
        BARRIER();
        return;
    }

    const int wm = wave >> 1, wn = wave & 1;
    char* amp = p.ws[0];
#pragma unroll
    for (int r = 1; r < NR; ++r) amp = (lane & 7) == r ? p.ws[r] : amp;
    amp += AMAX_OFF + (rank * 128 + g) * 8;
    asm volatile("" : "+v"(amp));
    const int uq = g * 4 + wave;
    const u32 uoff = P2_OFF + (rank * 512 + uq) * P2_B;
    char* pd[7];
#pragma unroll
    for (int k = 0; k < 7; ++k) pd[k] = p.peer[uq % 7 + k];
    asm volatile("" : "+s"(pd[0]), "+s"(pd[1]), "+s"(pd[2]), "+s"(pd[3]), "+s"(pd[4]), "+s"(pd[5]), "+s"(pd[6]));
    f32x16 acc[2][2];
#pragma unroll
    for (int i = 0; i < 4; ++i)
#pragma unroll
        for (int j = 0; j < 16; ++j) acc[i >> 1][i & 1][j] = 0.f;
    {
        u32 xb = L_STAGE + wm * 4096 + lane * 16, wb = L_STAGE + (2 + wn) * 4096 + lane * 16;
        volatile LAS u32* prog = (volatile LAS u32*)(lds + L_MISC + 64);
        volatile LAS u32* done = (volatile LAS u32*)(lds + L_MISC + 96 + wave * 4);
        bf16x8 x0[4], w0[4], x1[4], w1[4];
#define SB() __builtin_amdgcn_sched_barrier(0)
#define RD1(X, W, off, f) do { W[f] = *(LAS bf16x8*)(lds + wb + (off) + (f) * 1024); X[f] = *(LAS bf16x8*)(lds + xb + (off) + (f) * 1024); SB(); } while (0)
#define MM1(X, W, i) do { acc[((i) >> 1) & 1][(i) & 1] = __builtin_amdgcn_mfma_f32_32x32x16_bf16( \
            W[((i) >> 2) * 2 + ((i) & 1)], X[(i) >> 1], acc[((i) >> 1) & 1][(i) & 1], 0, 0, 0); \
            asm volatile("" : "+v"(acc[((i) >> 1) & 1][(i) & 1])); SB(); } while (0)
#define LACKS(need) (((rfl(P) - (u32)(need) * 0x10001u) & 0x80008000u) != 0)
#define AWAIT(j, need) do { \
            if (__builtin_expect(LACKS(need), 0)) { do { P = prog[(j) & 3]; } while (LACKS(need)); } \
            asm volatile("" ::: "memory"); SB(); } while (0)
#define RELEASE(j, t) do { asm volatile("" ::: "memory"); *done = (t) + 1; P = prog[((j) + 1) & 3]; SB(); } while (0)
#define STEP(X, W, XN, WN, j) do { constexpr u32 o_ = ((j) & 3) * 16384; \
            MM1(X, W, 0); AWAIT(j, (t4 >> 2) + ((j) >> 2) + 1); MM1(X, W, 1); \
            if ((j) == 4) { xb ^= 65536u; wb ^= 65536u; } \
            RD1(XN, WN, o_, 0); RD1(XN, WN, o_, 1); MM1(X, W, 2); \
            RD1(XN, WN, o_, 2); RD1(XN, WN, o_, 3); MM1(X, W, 3); RELEASE(j, t4 + (j)); \
            MM1(X, W, 4); MM1(X, W, 5); MM1(X, W, 6); MM1(X, W, 7); } while (0)
        {
            __builtin_amdgcn_s_sleep(INDELAY);
            const char* src = ((wave & 1) ? p.w : p.x) + ((g * 2 + (wave >> 1)) * 16384 + lane * 16);
            u32x4 t[16];
#pragma unroll
            for (int j = 0; j < 16; ++j) t[j] = *(const GAS u32x4*)(src + j * 1024);
#pragma unroll
            for (int j = 0; j < 16; ++j) asm volatile("" :: "v"(t[j]));
        }
        if (threadIdx.x < 12) *(LAS u32*)(lds + L_MISC + 64 + threadIdx.x * 4) = 0;
        BARRIER();
        if (threadIdx.x == 0 && call) __hip_atomic_store(ctr, call, __ATOMIC_RELAXED, __HIP_MEMORY_SCOPE_AGENT);
        u32 P = 0;
        AWAIT(0, 1);
        RD1(x0, w0, 0, 0); RD1(x0, w0, 0, 1); RD1(x0, w0, 0, 2); RD1(x0, w0, 0, 3);
        RELEASE(0, 0);
        int t4 = 0;
        for (; t4 < 2 * NSTEP - 4; t4 += 4) {
            STEP(x0, w0, x1, w1, 1); STEP(x1, w1, x0, w0, 2); STEP(x0, w0, x1, w1, 3); STEP(x1, w1, x0, w0, 4);
        }
        STEP(x0, w0, x1, w1, 1); STEP(x1, w1, x0, w0, 2); STEP(x0, w0, x1, w1, 3);
#pragma unroll
        for (int i = 0; i < 8; ++i) MM1(x1, w1, i);
    }
    STAMP(2);
#define Y(e) ((int)__float_as_uint(acc[(e) >> 5][(e) & 1][((e) >> 1) & 15]))
    asm volatile("s_nop 15\n\ts_nop 7"
                 : "+v"(acc[0][0]), "+v"(acc[0][1]), "+v"(acc[1][0]), "+v"(acc[1][1]));
    int mi = 0;
#pragma unroll
    for (int e = 0; e < 64; e += 2) asm("v_max3_i32 %0, %1, %2, %0" : "+v"(mi) : "v"(Y(e)), "v"(Y(e + 1)));
    asm volatile("s_nop 1" : "+v"(mi));
    const u32 mx = wave_umax((u32)mi);
    if (lane == 0) *(LAS u32*)(lds + L_MISC + 16 + wave * 4) = mx;
    BARRIER();
    if (wave == 0) {
        u32x4 wx = *(LAS u32x4*)(lds + L_MISC + 16);
        u32 tm = max(max(wx[0], wx[1]), max(wx[2], wx[3]));
        if (lane < NR) st_sys64(amp, ((u64)call << 32) | tm);
        STAMP(6);
    }
    u32 mk0 = 0, mk1 = 0;
#pragma unroll
    for (int e = 31; e >= 0; e -= 2) {
        u64 c0, c1, c2, c3;
        asm("v_cmp_lt_i32_e64 %2, 0, %6\n\tv_cmp_lt_i32_e64 %3, 0, %7\n\tv_cmp_lt_i32_e64 %4, 0, %8\n\tv_cmp_lt_i32_e64 %5, 0, %9\n\t"
            "v_addc_co_u32_e64 %0, vcc, %0, %0, %2\n\tv_addc_co_u32_e64 %1, vcc, %1, %1, %3\n\t"
            "v_addc_co_u32_e64 %0, vcc, %0, %0, %4\n\tv_addc_co_u32_e64 %1, vcc, %1, %1, %5"
            : "+v"(mk0), "+v"(mk1), "=&s"(c0), "=&s"(c1), "=&s"(c2), "=&s"(c3)
            : "v"(Y(e)), "v"(Y(e + 32)), "v"(Y(e - 1)), "v"(Y(e + 31)) : "vcc");
    }
    u32x4 mv;
    {
        asm volatile("s_nop 1" : "+v"(mk0), "+v"(mk1));
        auto s0 = __builtin_amdgcn_permlane32_swap(mk0, mk0, false, false);
        auto s1 = __builtin_amdgcn_permlane32_swap(mk1, mk1, false, false);
        mv = u32x4{s0[0], s1[0], s0[1], s1[1]};
    }
#define MA 3
#define MB 5
#define MASKS(k0, k1) do { if (lane < 32) { _Pragma("unroll") for (int k = (k0); k < (k1); ++k) st_wt128s(pd[k], uoff + lane * 16, mv); } } while (0)
    MASKS(0, MA);
    const u32 cnt = __builtin_popcount(mk0) + __builtin_popcount(mk1);
    const u32 incl = wave_incl_scan(cnt);
    const u32 total = __builtin_amdgcn_readlane(incl, 63);
    LAS char* cb = lds + L_COMP + wave * 16384;
    {
        u32 pos = L_COMP + wave * 16384 + (incl - cnt) * 4;
#define C1(n) "v_cmpx_lt_i32_e32 vcc, 0, %" #n "\n\tds_write_b32 %0, %" #n "\n\tv_add_u32 %0, 4, %0\n\ts_mov_b64 exec, -1\n\t"
#pragma unroll
        for (int e = 0; e < 64; e += 8) {
            if (e == 32 && MA < MB) { asm volatile("s_waitcnt vmcnt(1)" ::: "memory"); MASKS(MA, MB); }
            asm volatile(C1(1) C1(2) C1(3) C1(4) C1(5) C1(6) C1(7) C1(8)
                         : "+v"(pos)
                         : "v"(Y(e)), "v"(Y(e + 1)), "v"(Y(e + 2)), "v"(Y(e + 3)), "v"(Y(e + 4)), "v"(Y(e + 5)),
                           "v"(Y(e + 6)), "v"(Y(e + 7))
                         : "memory", "vcc");
        }
        asm volatile("s_nop 3");
        if (lane < 16 && total + lane < 4096u) *(LAS u32*)(cb + (total + lane) * 4) = mx;
    }
    const u32 ng = (total + 15) >> 4;
    f32x4 cv[4][4];
#pragma unroll
    for (int r = 0; r < 4; ++r)
#pragma unroll
        for (int j = 0; j < 4; ++j) cv[r][j] = *(LAS f32x4*)(cb + r * 4096 + lane * 64 + j * 16);
    STAMP(3);
    if (MB < NEARLY) { asm volatile("s_waitcnt vmcnt(1)" ::: "memory"); MASKS(MB, NEARLY); }
    BARRIER();
    MASKS(NEARLY, 7);
    float inv;
    {
        u32x4 gv = *(LAS u32x4*)(lds + L_MISC + 32), gw = *(LAS u32x4*)(lds + L_MISC + 48);
        float amax = __uint_as_float(max(max(max(gv[0], gv[1]), max(gv[2], gv[3])), max(max(gw[0], gw[1]), max(gw[2], gw[3]))));
        inv = 1.0f / (amax / 448.0f);
    }
    STAMP(4);
    u32x4 q[4];
#pragma unroll
    for (int r = 0; r < 4; ++r) {
        q[r] = u32x4{0x7F7F7F7Fu, 0x7F7F7F7Fu, 0x7F7F7F7Fu, 0x7F7F7F7Fu};
        if (r < 2 || (u32)(r * 64) < ng) {
#pragma unroll
            for (int j = 0; j < 4; ++j) {
                f32x4 t = cv[r][j] * inv;
                int w = __builtin_amdgcn_cvt_pk_fp8_f32(t[0], t[1], 0, false);
                q[r][j] = (u32)__builtin_amdgcn_cvt_pk_fp8_f32(t[2], t[3], w, true);
            }
        }
    }
    u32 h2, nb;
    {
        const u32 cmax = rfl((u32)__builtin_amdgcn_cvt_pk_fp8_f32(__uint_as_float(mx) * inv, 0.f, 0, false)) & 0xFFu;
        const u32 wl = max(cmax, 63u) - 63u, k4 = (0x80u - wl) * 0x01010101u;
        u32 m[2] = {0, 0};
#pragma unroll
        for (int r = 0; r < 4; ++r) {
            u32 t[4];
#pragma unroll
            for (int j = 0; j < 4; ++j) { t[j] = q[r][j] + k4; m[r >> 1] = bfi(0x80808080u, t[j], m[r >> 1] >> 1); }
            if (r < 2 || (u32)(r * 64) < ng) {
                LAS u32* o = (LAS u32*)(cb + lane * 12 + r * 768);
                o[0] = bfi(0x3F3F3F3Fu, t[0], t[3] << 6);
                o[1] = bfi(0x3F3F3F3Fu, t[1], t[3] << 4);
                o[2] = bfi(0x3F3F3F3Fu, t[2], t[3] << 2);
            }
        }
#pragma unroll
        for (int h = 0; h < 2; ++h)
            m[h] = ~m[h] & (((u32)(lane + 128 * h) < ng ? 0x0F0F0F0Fu : 0u) | ((u32)(lane + 128 * h + 64) < ng ? 0xF0F0F0F0u : 0u));
        const u32 ec = __builtin_popcount(m[0]) + __builtin_popcount(m[1]);
        const u32 ei = wave_incl_scan(ec);
        const u32 ne = __builtin_amdgcn_readlane(ei, 63);
        if (__builtin_expect(ne > MAXE2 || cmax > 126u, 0)) {
            h2 = total | H2_RAW;
            nb = ng * 16;
        } else {
            h2 = total | (ne << 13) | (wl << 21);
            nb = (ng * 12 + ne * 2 + 63) & ~63u;
            LAS unsigned short* el = (LAS unsigned short*)(cb + ng * 12) + (ei - ec);
            u64 mm = ((u64)m[1] << 32) | m[0];
            while (mm != 0) {
                const u32 bp = __builtin_ctzll(mm);
                *el++ = (unsigned short)(lane * 16 + (bp >> 5) * 2048 + (bp & 4) * 256 + (bp & 3) * 4 + ((bp >> 3) & 3));
                mm &= mm - 1;
            }
#pragma unroll
            for (int r = 0; r < 4; ++r) q[r] = *(LAS u32x4*)(cb + r * 1024 + lane * 16);
        }
    }
#pragma unroll
    for (int k = 0; k < 7; ++k) {
        char* d = pd[k];
#pragma unroll
        for (int r = 0; r < 4; ++r)
            if ((u32)(lane * 16 + r * 1024) < nb) st_wt128s(d, uoff + 512 + r * 1024 + lane * 16, q[r]);
        if (k == 0) {
            u32x4 ot[4];
#pragma unroll
            for (int mt = 0; mt < 4; ++mt) {
                u32x4 ov;
#pragma unroll
                for (int j = 0; j < 4; ++j) {
                    const int e = mt * 16 + j * 4;
#define YR(e) (__int_as_float(max(Y(e), 0)) * inv)
                    int w = __builtin_amdgcn_cvt_pk_fp8_f32(YR(e), YR(e + 1), 0, false);
                    ov[j] = (u32)__builtin_amdgcn_cvt_pk_fp8_f32(YR(e + 2), YR(e + 3), w, true);
                }
                ot[mt] = ov;
            }
            store_tile(p.out + (size_t)((rank >> 2) * 2048 + tr * 128 + wm * 64) * NN + (rank & 3) * 1024 + tc * 128 + wn * 64,
                       ot, lane);
        }
        WAIT_VM0();
        if (lane == 0) st_sys64(d + F2_OFF + (rank * 512 + uq) * FL_B, ((u64)call << 32) | h2);
    }
    STAMP(5);
}

extern "C" void kernel_launch(int rank, int nranks, ncclComm_t comm, hipStream_t stream,
                              void* const* d_in, const int* in_sizes, int n_in,
                              void* d_out, int out_size, const int* in_dtypes, int out_dtype,
                              void* const* d_out_all, void* const* d_ws_all, size_t ws_bytes) {
    (void)comm; (void)in_dtypes; (void)out_dtype; (void)d_out_all;
    if (nranks != NR || (unsigned)rank >= (unsigned)NR || n_in < 2 || ws_bytes < (size_t)SPARE_OFF + WS_USED ||
        in_sizes[0] != MM * KS || in_sizes[1] != KS * NN || out_size != MM * NN) {
        fprintf(stderr, "kernel_launch: unexpected problem shape\n");
        return;
    }
    HIP_CHECK(hipFuncSetAttribute((const void*)fused_kernel, hipFuncAttributeMaxDynamicSharedMemorySize, LDS_BYTES));
    Params p;
    p.x = (const char*)d_in[0];
    p.w = (const char*)d_in[1];
    p.out = (char*)d_out;
    for (int r = 0; r < NR; ++r) p.ws[r] = (char*)d_ws_all[r];
    for (int r = 0; r < NR; ++r) p.wsd[r] = p.ws[r] + (r == rank ? SPARE_OFF : 0u);
    for (int k = 0; k < 14; ++k) p.peer[k] = p.ws[(rank + 1 + k % 7) & 7];
    p.rank = rank;
    p.me = p.ws[rank];
    p.pad = 0;
    fused_kernel<<<dim3(256), dim3(768), LDS_BYTES, stream>>>(p);
    HIP_CHECK(hipGetLastError());
}
```

```cpp
#include <hip/hip_runtime.h>
#include <rccl/rccl.h>
#include <cstdio>
#include <cstdint>

#define HIP_CHECK(expr) do { hipError_t _e = (expr); if (_e != hipSuccess) { \
    fprintf(stderr, "HIP error %s at %s:%d\n", hipGetErrorString(_e), __FILE__, __LINE__); \
    return; } } while (0)

typedef unsigned u32;
typedef unsigned long long u64;
typedef u32 u32x2 __attribute__((ext_vector_type(2)));
typedef u32 u32x3 __attribute__((ext_vector_type(3)));
typedef u32 u32x4 __attribute__((ext_vector_type(4)));
typedef float f32x2 __attribute__((ext_vector_type(2)));
typedef float f32x4 __attribute__((ext_vector_type(4)));
typedef float f32x16 __attribute__((ext_vector_type(16)));
typedef unsigned short u16x2 __attribute__((ext_vector_type(2)));
typedef __bf16 bf16x8 __attribute__((ext_vector_type(8)));
#define GAS __attribute__((address_space(1)))
#define LAS __attribute__((address_space(3)))
#define DI __device__ __forceinline__

constexpr int NR = 8;
constexpr int MM = 4096, NN = 4096, KS = 512;
constexpr int NCH = 16;
constexpr int NSLOT = 48;
constexpr u32 NITEM = NCH * 128;
#define NEARLY 6
#define INDELAY 8
#define OWNDELAY 0
#define KW 4
static_assert(KW <= 6, "the ticket must be older than KW instructions");
#define CWB 8
#define PSL 1
#define NPUSH 88
constexpr int NSTEP = 64;
constexpr u32 UNIT_B = 3072, RAWU_B = 4096, FL_B = 128;
constexpr u32 CTR_OFF = 0;
constexpr u32 TMO_OFF = 32768;
constexpr u32 TKT_OFF = 33024;
constexpr u32 AMAX_OFF = 65536;
constexpr u32 F1_OFF = 131072;
constexpr u32 F2_OFF = 1u << 20;
constexpr u32 DBG_OFF = 1572864;
constexpr u32 P1_OFF = 2u << 20;
constexpr u32 RAW_OFF = 20u << 20;
constexpr u32 P2_OFF = 44u << 20;
constexpr u32 P2_B = 4608;
constexpr u32 WS_USED = 62u << 20;
constexpr u32 SPARE_OFF = 64u << 20;
static_assert(F1_OFF + NR * NCH * NSLOT * FL_B <= F2_OFF, "");
static_assert(F2_OFF + NR * 512 * FL_B <= DBG_OFF, "");
static_assert(P1_OFF + NR * NCH * NSLOT * UNIT_B <= RAW_OFF, "");
static_assert(RAW_OFF + NR * NCH * NSLOT * RAWU_B <= P2_OFF, "");
static_assert(P2_OFF + NR * 512 * P2_B <= WS_USED, "");
constexpr int CODE_BYTES = 42 * 1024;
constexpr u64 DEADLINE = 50000000ull;
constexpr int MAXEXC = 7;
constexpr u32 MAXL1 = 64;
constexpr u32 HDR_OVF = 0x800, HDR_RAW = 0x1000;
constexpr int MAXE2 = 128;
constexpr u32 H2_RAW = 1u << 28;
constexpr u32 INFLAG = 14;

constexpr u32 L_STAGE = 0;
constexpr u32 L_COMP = 0;
constexpr u32 L_MISC = 131072;
constexpr u32 L_DEC = 0;
constexpr u32 DEC_B = 4352;
constexpr u32 L_LUT = 53248;
constexpr u32 L_SCR = 57344;
constexpr u32 SCR_B = 5504;
constexpr u32 LDS_BYTES = 131072 + 512;

struct Params {
    const char* x;
    const char* w;
    char* me;
    int rank;
    int pad;
    char* wsd[NR];
    char* out;
    char* ws[NR];
    char* peer[14];
};

#define PAY_AUX 17
#define ACC(var) do {} while (0)
#define PUT(i, val) do {} while (0)
#define STAMP(i) do {} while (0)

#define PSTAMP() do {} while (0)
DI u32 perm(u32 hi, u32 lo, u32 sel) { return __builtin_amdgcn_perm(hi, lo, sel); }
DI u32 bfi(u32 m, u32 a, u32 b) { return (m & a) | (~m & b); }
DI u32 rotr(u32 v, u32 n) { return __builtin_amdgcn_alignbit(v, v, n); }
DI u32 rfl(u32 v) { return __builtin_amdgcn_readfirstlane(v); }
template <int CTRL, int RM> DI u32 dpp0(u32 v) { return __builtin_amdgcn_update_dpp(0u, v, CTRL, RM, 0xF, false); }
DI u32 wave_umax(u32 v) {
    v = max(v, dpp0<0x111, 0xF>(v)); v = max(v, dpp0<0x112, 0xF>(v));
    v = max(v, dpp0<0x114, 0xF>(v)); v = max(v, dpp0<0x118, 0xF>(v));
    v = max(v, dpp0<0x142, 0xA>(v)); v = max(v, dpp0<0x143, 0xC>(v));
    return __builtin_amdgcn_readlane(v, 63);
}
DI u32 wave_incl_scan(u32 v) {
    v += dpp0<0x111, 0xF>(v); v += dpp0<0x112, 0xF>(v);
    v += dpp0<0x114, 0xF>(v); v += dpp0<0x118, 0xF>(v);
    v += dpp0<0x142, 0xA>(v); v += dpp0<0x143, 0xC>(v);
    return v;
}
DI u64 ld_sys64(const char* p) {
    return __hip_atomic_load((GAS u64*)p, __ATOMIC_RELAXED, __HIP_MEMORY_SCOPE_SYSTEM);
}
DI void st_sys64(char* p, u64 v) {
    __hip_atomic_store((GAS u64*)p, v, __ATOMIC_RELAXED, __HIP_MEMORY_SCOPE_SYSTEM);
}
DI void st_wt128(char* p, u32x4 v) {
    asm volatile("global_store_dwordx4 %0, %1, off sc0 sc1\n\ts_nop 1" :: "v"(p), "v"(v) : "memory");
}
DI void st_wt128s(const char* sbase, u32 voff, u32x4 v) {
    asm volatile("global_store_dwordx4 %0, %1, %2 sc0 sc1\n\ts_nop 1" :: "v"(voff), "v"(v), "s"(sbase) : "memory");
}
#define WAIT_VM0() asm volatile("s_waitcnt vmcnt(0)" ::: "memory")
#define WAIT_LGKM0() asm volatile("s_waitcnt lgkmcnt(0)" ::: "memory")
#define BARRIER() do { WAIT_LGKM0(); __builtin_amdgcn_s_barrier(); asm volatile("" ::: "memory"); } while (0)

DI bool spin_expired(char* myws, u32 call, u64 t0, u32& it) {
    if ((++it & 15u) == 0) {
        GAS u32* tmo = (GAS u32*)(myws + TMO_OFF);
        u32 t = rfl(__hip_atomic_load(tmo, __ATOMIC_RELAXED, __HIP_MEMORY_SCOPE_AGENT));
        if (t == call || __builtin_amdgcn_s_memrealtime() - t0 > DEADLINE) {
            __hip_atomic_store(tmo, call, __ATOMIC_RELAXED, __HIP_MEMORY_SCOPE_AGENT);
            return true;
        }
    }
    __builtin_amdgcn_s_sleep(PSL);
    return false;
}
DI bool tmo_set(char* myws, u32 call) {
    return rfl(__hip_atomic_load((GAS u32*)(myws + TMO_OFF), __ATOMIC_RELAXED, __HIP_MEMORY_SCOPE_AGENT)) == call;
}

DI u32 twice(u32 v) {
    const u32 m = v & 0x7FFFu;
    return (v & 0x8000u) | (m < 0x80u ? m << 1 : m >= 0x7F00u ? max(m, 0x7F80u) : m + 0x80u);
}
DI u64 pack_unit(const u32x4 (&R)[4], u32x4& p0, u32x4& p1, u32x4& p2, u64& act, LAS char* scr, int lane,
                 char* d0, char* d1, char* d2, char* d3, int nd, u32 unit, u32 call) {
    u32 L[8], H[8];
    u16x2 mx = {0, 0};
#pragma unroll
    for (int f = 0; f < 4; ++f) {
        const u32x4 r = R[f] + 0x00800080u;
        L[2 * f] = perm(r[1], r[0], 0x06040200u); H[2 * f] = perm(r[1], r[0], 0x07050301u);
        L[2 * f + 1] = perm(r[3], r[2], 0x06040200u); H[2 * f + 1] = perm(r[3], r[2], 0x07050301u);
#pragma unroll
        for (int d = 0; d < 4; ++d)
            mx = __builtin_elementwise_max(mx, __builtin_bit_cast(u16x2, R[f][d] & 0x7FFF7FFFu));
    }
    const u32 mine = max((u32)mx[0], (u32)mx[1]), big = wave_umax(mine);
    u32 top = (big + 0x80u) >> 8;
    bool raw = big >= 0x7F00u || top < 10u;
    const bool cut = __builtin_popcountll(__builtin_amdgcn_ballot_w64(mine + 0x80u >= (top << 8))) <= 2;
    if (cut) top -= 1u;
    const u32 base = (top - 3u) & 0x7Fu, k4 = 0x80808080u - base * 0x01010101u;
    u32 T[8], N[8], near = 0x80808080u, m1 = 0;
#pragma unroll
    for (int i = 0; i < 8; ++i) {
        T[i] = (H[i] & 0x7F7F7F7Fu) + k4;
        near &= T[i] + 0x04040404u;
        m1 = bfi(0x80808080u, T[i], m1 >> 1);
        N[i] = bfi(0x80808080u, H[i], T[i]);
    }
    LAS char* stg = scr + 4096 + 64;
    {
        LAS u32* o = (LAS u32*)(stg + lane * 12);
        o[0] = bfi(0x60606060u, N[6] << 5, bfi(0x83838383u, N[0], rotr(N[1], 29)));
        o[1] = bfi(0x60606060u, N[7] << 5, bfi(0x83838383u, N[2], rotr(N[3], 29)));
        o[2] = bfi(0x60606060u, bfi(0x20202020u, N[6] >> 2, N[7] >> 1), bfi(0x83838383u, N[4], rotr(N[5], 29)));
    }
    LAS unsigned short* lst = (LAS unsigned short*)(stg + 1024);
    m1 = ~m1;
    const u32 c1 = __builtin_popcount(m1), i1 = wave_incl_scan(c1);
    u32 n1 = __builtin_amdgcn_readlane(i1, 63);
    raw = raw || n1 > MAXL1;
    if (!raw) {
        LAS unsigned short* el = lst + (i1 - c1);
        while (m1 != 0) {
            const u32 bp = __builtin_ctz(m1);
            *el++ = (unsigned short)((bp & 6) * 256 + lane * 8 + (bp & 1) * 4 + (bp >> 3));
            m1 &= m1 - 1;
        }
    }
    p0 = u32x4{L[0], L[1], L[2], L[3]};
    p1 = u32x4{L[4], L[5], L[6], L[7]};
    u32 hdr = base, rec = 0, n2 = 0;
    if (raw || cut || __builtin_amdgcn_ballot_w64(near != 0x80808080u) != 0) {
        u32 cnt = 0;
        if (!raw) {
#pragma unroll
            for (int f = 0; f < 4; ++f) *(LAS u32x4*)(scr + f * 1024 + lane * 16) = R[f];
            u32 m = 0;
#pragma unroll
            for (int i = 0; i < 8; ++i) {
                const u32 z = ((T[i] + 0x04040404u) ^ 0x80808080u) & 0xF8F8F8F8u;
                m |= ((((z & 0x7F7F7F7Fu) + 0x7F7F7F7Fu) | z) & 0x80808080u) >> (7 - i);
            }
            for (;;) {
                u64 bal = __builtin_amdgcn_ballot_w64(m != 0);
                if (bal == 0 || cnt > MAXEXC) break;
                if (m != 0) {
                    u32 bp = __builtin_ctz(m);
                    u32 addr = (bp & 6) * 512 + lane * 16 + (bp & 1) * 8 + (bp >> 3) * 2;
                    u32 v = twice(*(LAS unsigned short*)(scr + addr));
                    u32 slot = cnt + __builtin_amdgcn_mbcnt_hi((u32)(bal >> 32), __builtin_amdgcn_mbcnt_lo((u32)bal, 0));
                    if (slot < MAXEXC) *(LAS u32*)(scr + 4096 + 28 - slot * 4) = (addr << 16) | v;
                    m &= m - 1;
                }
                cnt += __builtin_popcountll(bal);
            }
        }
        if (raw || cnt > MAXEXC) {
            hdr = HDR_RAW;
            n1 = 0;
#pragma unroll
            for (int f = 0; f < 4; ++f) {
                u32x4 r;
#pragma unroll
                for (int d = 0; d < 4; ++d) r[d] = twice(R[f][d] & 0xFFFFu) | (twice(R[f][d] >> 16) << 16);
                const u32 o = RAW_OFF + unit * RAWU_B + f * 1024 + lane * 16;
                st_wt128s(d0, o, r);
                st_wt128s(d1, o, r);
                if (nd > 2) { st_wt128s(d2, o, r); st_wt128s(d3, o, r); }
            }
        } else {
            n2 = cnt;
            rec = *(LAS u32*)(scr + 4096 + (lane & 7) * 4);
        }
    }
    const u32 nin = min(n1, min(32u, 5u * (7u - n2))), nov = n1 - nin;
    if (__builtin_expect(nov != 0, 0)) {
        hdr |= HDR_OVF;
        *(LAS unsigned short*)(stg + 768 + lane * 2) = lst[nin + lane];
    }
    p2 = *(LAS u32x4*)(stg + lane * 16);
    act = (1ull << (48 + ((nov + 31) >> 5) * 4)) - 1;
    const u32 j = lane & 7;
    const LAS unsigned short* e = lst + j * 5 - 5;
    const u32 e2 = e[2] & 0x7FFu;
    u32 w0 = (e[0] & 0x7FFu) | ((e[1] & 0x7FFu) << 11) | (e2 << 22);
    u32 w1 = (e2 >> 10) | ((e[3] & 0x7FFu) << 1) | ((e[4] & 0x7FFu) << 12) | (call << 23);
    if (j + n2 > 7u) w0 = rec;
    if (j == 0) { w0 = hdr | (n2 << 8) | (nin << 16) | (nov << 24); w1 = call; }
    return ((u64)w1 << 32) | w0;
}

#define A_TICKET(adr) "v_mov_b32 v167, 1\n\ts_mov_b64 exec, 1\n\tglobal_atomic_add v167, %" #adr ", v167, off sc0\n\ts_mov_b64 exec, -1\n\t"
#define A_END(n) "s_mov_b64 exec, -1\n\ts_nop 1\n\ts_waitcnt vmcnt(" #n ")"
#define A_ST(so, v, d, off) "global_store_dwordx4 %" #so ", %" #v ", %" #d " offset:" #off " sc0 sc1\n\t"
#define A_ST4(so, a, b, c, m, d0, d1, d2, d3) \
    A_ST(so, a, d0, 0) A_ST(so, a, d1, 0) A_ST(so, a, d2, 0) A_ST(so, a, d3, 0) \
    A_ST(so, b, d0, 1024) A_ST(so, b, d1, 1024) A_ST(so, b, d2, 1024) A_ST(so, b, d3, 1024) \
    "s_mov_b64 exec, %" #m "\n\t" \
    A_ST(so, c, d0, 2048) A_ST(so, c, d1, 2048) A_ST(so, c, d2, 2048) A_ST(so, c, d3, 2048)

#define A_ST2(so, a, b, c, m, d0, d1) \
    A_ST(so, a, d0, 0) A_ST(so, a, d1, 0) A_ST(so, b, d0, 1024) A_ST(so, b, d1, 1024) \
    "s_mov_b64 exec, %" #m "\n\t" A_ST(so, c, d0, 2048) A_ST(so, c, d1, 2048)
#define A_LDX "global_load_dwordx4 %0, %4, %6\n\tglobal_load_dwordx4 %1, %5, %6\n\t" \
              "global_load_dwordx4 %2, %4, %6 offset:32\n\tglobal_load_dwordx4 %3, %5, %6 offset:32\n\t"
#define A_WL(i) "global_load_dword %" #i ", %16, %17\n\tv_add_u32 %16, 0x2000, %16\n\t"
#define A_LDW A_WL(0) A_WL(1) A_WL(2) A_WL(3) A_WL(4) A_WL(5) A_WL(6) \
              "global_load_dword %7, %16, %17\n\tv_add_u32 %16, 0x12000, %16\n\t" \
              A_WL(8) A_WL(9) A_WL(10) A_WL(11) A_WL(12) A_WL(13) A_WL(14) "global_load_dword %15, %16, %17\n\t"
#define X_OUT "=&v"(R[0]), "=&v"(R[1]), "=&v"(R[2]), "=&v"(R[3])
#define X_IN "v"(lx0), "v"(lx1), "s"(src), "v"(tka), "v"(so), "v"(p0), "v"(p1), "v"(p2), "s"(act)
#define W_OUT "=&v"(G[0]), "=&v"(G[1]), "=&v"(G[2]), "=&v"(G[3]), "=&v"(G[4]), "=&v"(G[5]), "=&v"(G[6]), \
              "=&v"(G[7]), "=&v"(G[8]), "=&v"(G[9]), "=&v"(G[10]), "=&v"(G[11]), "=&v"(G[12]), "=&v"(G[13]), \
              "=&v"(G[14]), "=&v"(G[15]), "+v"(lw)
#define W_IN "s"(src), "v"(tka), "v"(so), "v"(p0), "v"(p1), "v"(p2), "s"(act)

DI void w_to_frags(const u32 (&G)[16], u32x4 (&R)[4]) {
#pragma unroll
    for (int f = 0; f < 4; ++f)
#pragma unroll
        for (int d = 0; d < 4; ++d)
            R[f][d] = perm(G[(f >> 1) * 8 + 2 * d + 1], G[(f >> 1) * 8 + 2 * d], (f & 1) ? 0x07060302u : 0x05040100u);
}

struct Item { bool isw; u32 unit, r0; const char* src; char *d0, *d1, *d2, *d3;
};
DI Item make_item(const Params& p, u32 i) {
    Item it;
    const u32 c = (i >> 7) & (NCH - 1), k = (i >> 1) & 63;
    it.isw = i & 1;
    it.r0 = it.isw ? (k & 3) : (k & 1) * 4;
    const u32 sl = it.isw ? k >> 2 : k >> 1;
    it.unit = (p.rank * NCH + c) * NSLOT + (it.isw ? 32 : 0) + sl;
    it.src = it.isw ? p.w + (size_t)c * (32 * NN * 2) + ((k & 3) * 16 + sl) * 128
                    : p.x + (size_t)((k & 1) * 32 + sl) * (64 * KS * 2) + c * 64;
    return it;
}
DI void item_dests(const Params& p, Item& it) {
    const u32 r0 = it.r0, rs = it.isw ? 4 : 1;
    it.d0 = p.wsd[r0]; it.d1 = p.wsd[r0 + rs]; it.d2 = p.wsd[(r0 + 2 * rs) & 7]; it.d3 = p.wsd[(r0 + 3 * rs) & 7];
}

DI u32 lane_x(int lane) { return (u32)(lane & 31) * (KS * 2) + (lane >> 5) * 16; }
DI u32 lane_w(int lane) {
    return (u32)(8 * (lane >> 5)) * (NN * 2) + ((lane & 19) | ((lane & 4) << 1) | ((lane & 8) >> 1)) * 4;
}

DI void push_inputs(const Params& p, u32 g, LAS char* scr, int lane) {
    const u32 lx0 = lane_x(lane), lx1 = lx0 + 32 * KS * 2, lw0 = lane_w(lane);
    u32 G[16];
    u32x4 R[4], p0, p1, p2;
    {
        const int r0 = (g & 1) * 4;
        asm volatile("" :: "s"(p.x), "s"(p.w), "s"(p.me), "s"(p.rank), "s"(p.wsd[r0]), "s"(p.wsd[r0 + 1]),
                     "s"(p.wsd[r0 + 2]), "s"(p.wsd[r0 + 3]));
    }
    Item cur = make_item(p, 2 * g), nxt = make_item(p, 2 * g + 1);
    item_dests(p, cur);
    u32 call;
    {
        const char* src = nxt.src;
        u32 lw = lw0;
        asm volatile("global_load_dwordx4 %19, %23, %25\n\tglobal_load_dwordx4 %20, %24, %25\n\t"
                     "global_load_dwordx4 %21, %23, %25 offset:32\n\tglobal_load_dwordx4 %22, %24, %25 offset:32\n\t"
                     "global_load_dword %18, %26, %27 sc1\n\t" A_LDW "s_waitcnt vmcnt(16)"
                     : "=&v"(G[0]), "=&v"(G[1]), "=&v"(G[2]), "=&v"(G[3]), "=&v"(G[4]), "=&v"(G[5]), "=&v"(G[6]),
                       "=&v"(G[7]), "=&v"(G[8]), "=&v"(G[9]), "=&v"(G[10]), "=&v"(G[11]), "=&v"(G[12]), "=&v"(G[13]),
                       "=&v"(G[14]), "=&v"(G[15]), "+v"(lw), "+s"(src), "=&v"(call),
                       "=&v"(R[0]), "=&v"(R[1]), "=&v"(R[2]), "=&v"(R[3])
                     : "v"(lx0), "v"(lx1), "s"(cur.src), "v"(CTR_OFF + g * 128), "s"(p.me) : "memory");
        call = rfl(call) + 1u;
    }
    char* const tka = p.me + TKT_OFF;
    const u32 tbase = (call - 1u) * NITEM - 2u * NPUSH;
    {
        u32x4 q[2];
#pragma unroll
        for (int f = 0; f < 4; ++f) {
            const u32x4 r = R[f] + 0x00800080u;
            q[f >> 1][(f & 1) * 2] = perm(r[1], r[0], 0x06040200u);
            q[f >> 1][(f & 1) * 2 + 1] = perm(r[3], r[2], 0x06040200u);
        }
        const u32 so = P1_OFF + cur.unit * UNIT_B + lane * 16;
        asm volatile("s_nop 4\n\t" A_TICKET(23)
                     A_ST(16, 17, 19, 0) A_ST(16, 17, 20, 0) A_ST(16, 17, 21, 0) A_ST(16, 17, 22, 0)
                     A_ST(16, 18, 19, 1024) A_ST(16, 18, 20, 1024) A_ST(16, 18, 21, 1024) A_ST(16, 18, 22, 1024)
                     "s_nop 1\n\ts_waitcnt vmcnt(9)"
                     : "+v"(G[0]), "+v"(G[1]), "+v"(G[2]), "+v"(G[3]), "+v"(G[4]), "+v"(G[5]), "+v"(G[6]), "+v"(G[7]),
                       "+v"(G[8]), "+v"(G[9]), "+v"(G[10]), "+v"(G[11]), "+v"(G[12]), "+v"(G[13]), "+v"(G[14]), "+v"(G[15])
                     : "v"(so), "v"(q[0]), "v"(q[1]), "s"(cur.d0), "s"(cur.d1), "s"(cur.d2), "s"(cur.d3), "v"(tka)
                     : "memory", "v167");
    }
#define ITEMDONE() do {} while (0)
    bool more = true;
    char* fp = nullptr;
    u64 fv = 0;
    int fn = 0;
    for (;;) {
        const int nd = cur.isw ? 2 : 4;
        char* const dl = (lane & 8) ? cur.d1 : cur.d0;
        char* const dh = (lane & 8) ? cur.d3 : cur.d2;
        char* const fl = ((lane & 16) ? dh : dl) + F1_OFF + cur.unit * FL_B + (lane & 7) * 8;
        u64 act;
        const u64 hdr = pack_unit(R, p0, p1, p2, act, scr, lane, cur.d0, cur.d1, cur.d2, cur.d3, nd, cur.unit, call);
        const u32 so = P1_OFF + cur.unit * UNIT_B + lane * 16;
        if (fn != 0) {
            u32 tk;
            asm volatile("s_waitcnt vmcnt(%6)\n\tv_readfirstlane_b32 %0, v167\n\ts_nop 1"
                         : "=s"(tk), "+v"(p0), "+v"(p1), "+v"(p2) : "v"(so), "s"(act), "n"(KW) : "memory");
            const u32 i = tk - tbase;
            more = i < NITEM;
            nxt = make_item(p, i);
        }
        const char* src = nxt.src;
        if (nxt.isw) {
            u32 lw = lw0;
            if (fn == 0)
                asm volatile("s_nop 4\n\ts_mov_b64 exec, %2\n\t" A_ST(0, 1, 3, 2048) A_ST(0, 1, 4, 2048)
                             A_ST(0, 1, 5, 2048) A_ST(0, 1, 6, 2048) "s_mov_b64 exec, -1\n\ts_nop 1"
                             :: "v"(so), "v"(p2), "s"(act), "s"(cur.d0), "s"(cur.d1), "s"(cur.d2), "s"(cur.d3) : "memory");
            else if (cur.isw)
                asm volatile("s_nop 4\n\t" A_LDW A_TICKET(18) A_ST2(19, 20, 21, 22, 23, 24, 25) A_END(7)
                             : W_OUT : W_IN, "s"(cur.d0), "s"(cur.d1) : "memory", "v167");
            else
                asm volatile("s_nop 4\n\t" A_LDW A_TICKET(18) A_ST4(19, 20, 21, 22, 23, 24, 25, 26, 27) A_END(13)
                             : W_OUT : W_IN, "s"(cur.d0), "s"(cur.d1), "s"(cur.d2), "s"(cur.d3) : "memory", "v167");
            w_to_frags(G, R);
        } else {
            if (cur.isw)
                asm volatile("s_nop 4\n\t" A_LDX A_TICKET(7) A_ST2(8, 9, 10, 11, 12, 13, 14) A_END(7)
                             : X_OUT : X_IN, "s"(cur.d0), "s"(cur.d1) : "memory", "v167");
            else
                asm volatile("s_nop 4\n\t" A_LDX A_TICKET(7) A_ST4(8, 9, 10, 11, 12, 13, 14, 15, 16) A_END(13)
                             : X_OUT : X_IN, "s"(cur.d0), "s"(cur.d1), "s"(cur.d2), "s"(cur.d3) : "memory", "v167");
        }
        if (lane < fn) st_sys64(fp, fv);
        if (fn != 0) ITEMDONE();
        fp = fl;
        fv = hdr;
        fn = nd * 8;
        if (!more) break;
        cur = nxt;
        item_dests(p, cur);
    }
    WAIT_VM0();
    if (lane < fn) st_sys64(fp, fv);
    ITEMDONE();
}

struct OSet { u32x4 r[16]; };
DI void own_load(OSet& O, bool isw, const char* src, int lane) {
    if (isw) {
        src += lane_w(lane);
#pragma unroll
        for (int j = 0; j < 64; ++j)
            O.r[j >> 2][j & 3] = *(const GAS u32*)(src + ((j >> 4) & 1) * 128 + ((j >> 5) * 32 + ((j >> 3) & 1) * 16 + (j & 7)) * (NN * 2));
    } else {
        src += (lane >> 3) * (KS * 2) + (lane & 7) * 16;
#pragma unroll
        for (int j = 0; j < 16; ++j) O.r[j] = *(const GAS u32x4*)(src + j * 8 * (KS * 2));
    }
}
DI void own_stage(const OSet& O, bool isw, LAS char* o, int lane) {
    if (isw) {
#pragma unroll
        for (int j = 0; j < 16; ++j) {
            const u32x4 a = O.r[j & 14], b = O.r[j | 1];
            const u32 sel = (j & 1) ? 0x07060302u : 0x05040100u;
            *(LAS u32x4*)(o + (j >> 3) * 65536 + (j & 7) * 1024 + lane * 16) =
                u32x4{perm(a[1], a[0], sel), perm(a[3], a[2], sel), perm(b[1], b[0], sel), perm(b[3], b[2], sel)};
        }
    } else {
        o += ((lane >> 2) & 1) * 65536 + ((lane >> 1) & 1) * 2048 + (lane & 1) * 512 + (lane >> 3) * 16;
#pragma unroll
        for (int j = 0; j < 16; ++j) *(LAS u32x4*)(o + (j >> 3) * 4096 + ((j >> 2) & 1) * 1024 + (j & 3) * 128) = O.r[j];
    }
}

typedef __amdgpu_buffer_rsrc_t rsrc_t;
struct PSet { u32x4 lo[4]; u32x3 hi[2]; u32 pos, lst, rec; u32 hdr0, hdr1; u32 unit; };
struct FTag { u32 sh, ref; };
DI u64 stale(u32x2 F, FTag g) { return __builtin_amdgcn_ballot_w64((F[1] >> g.sh) != g.ref); }

DI u32x2 load_flags(rsrc_t rs, u32 voff, u32 unit) {
    return __builtin_amdgcn_raw_buffer_load_b64(rs, voff, F1_OFF + unit * FL_B, 17);
}
DI bool wait_flags(u32x2& F0, u32x2& Fn, rsrc_t rs, char* myws, u32 unit, u32 next, int lane, u32 call, FTag g) {
    u64 t0 = __builtin_amdgcn_s_memrealtime();
    u32 it = 0;
    bool bad = false;
    const u32 voff = F1_OFF + (((lane & 16) ? next : unit) + ((lane >> 3) & 1)) * FL_B + (lane & 7) * 8;
    u32x2 F;
    for (;;) {
        asm volatile("" ::: "memory");
        F = __builtin_amdgcn_raw_buffer_load_b64(rs, voff, 0, 17);
        if ((stale(F, g) & 0xFFFF) == 0) break;
        if (spin_expired(myws, call, t0, it)) { bad = true; break; }
    }
    F0 = F;
    Fn[0] = __builtin_amdgcn_permlane16_swap(F[0], F[0], false, false)[1];
    Fn[1] = __builtin_amdgcn_permlane16_swap(F[1], F[1], false, false)[1];
    return bad;
}
DI void issue_set(PSet& S, u32x2 F, rsrc_t rs, u32 unit, int lane) {
    asm volatile("" ::: "memory");
    S.hdr0 = __builtin_amdgcn_readlane(F[0], 0);
    S.hdr1 = __builtin_amdgcn_readlane(F[0], 8);
    S.rec = F[0];
    S.unit = unit;
#pragma unroll
    for (int u = 0; u < 2; ++u) {
        S.hi[u] = __builtin_amdgcn_raw_buffer_load_b96(rs, lane * 12 + u * UNIT_B + 2048, P1_OFF + unit * UNIT_B, 17);
        S.lo[2 * u] = __builtin_amdgcn_raw_buffer_load_b128(rs, lane * 16 + u * UNIT_B, P1_OFF + unit * UNIT_B, 17);
        S.lo[2 * u + 1] = __builtin_amdgcn_raw_buffer_load_b128(rs, lane * 16 + u * UNIT_B + 1024, P1_OFF + unit * UNIT_B, 17);
    }
    const u32 from = ((lane >> 5) * 8 + 1 + (lane & 31) / 5) * 4;
    const u64 gw = ((u64)(u32)__builtin_amdgcn_ds_bpermute(from, (int)F[1]) << 32) | (u32)__builtin_amdgcn_ds_bpermute(from, (int)F[0]);
    S.pos = (u32)(gw >> ((lane & 31) % 5 * 11));
    const u32 nov = ((lane & 32) ? S.hdr1 : S.hdr0) >> 24;
    S.lst = __builtin_amdgcn_raw_buffer_load_b32(
        rs, (u32)(lane & 31) * 2 < nov ? (lane >> 5) * UNIT_B + 2816 + (lane & 31) * 4 : 0x80000000u, P1_OFF + unit * UNIT_B, 17);
}
DI void lower_listed(u32 ob, u32 pos, u32 i, u32 n) {
    asm volatile("v_cmpx_lt_u32_e32 vcc, %0, %1\n\tds_sub_u32 %2, %3\n\ts_mov_b64 exec, -1\n\ts_nop 3"
                 :: "v"(i), "v"(n), "v"(((pos << 1) & 0xFFCu) | ob), "v"(0x400u << ((pos << 4) & 31u)) : "memory", "vcc");
}
DI void unpack_set(const PSet& S, rsrc_t rs, LAS char* lds, u32 oo, int lane) {
    LAS char* o = lds + oo;
#pragma unroll
    for (int u = 0; u < 2; ++u) {
        const u32 base4 = ((u ? S.hdr1 : S.hdr0) & 0x7F) * 0x01010101u;
#pragma unroll
        for (int f = 0; f < 4; ++f) {
            u32 h0, h1;
            if (f < 3) {
                const u32 w = S.hi[u][f];
                h0 = (w & 0x83838383u) + base4;
                h1 = (rotr(w, 3) & 0x83838383u) + base4;
            } else {
                h0 = ((S.hi[u][2] << 2) & 0x80808080u) | (((S.hi[u][0] >> 5) & 0x03030303u) + base4);
                h1 = ((S.hi[u][2] << 1) & 0x80808080u) | (((S.hi[u][1] >> 5) & 0x03030303u) + base4);
            }
            u32 l0 = S.lo[2 * u + (f >> 1)][(f & 1) * 2], l1 = S.lo[2 * u + (f >> 1)][(f & 1) * 2 + 1];
            u32x4 r = {perm(h0, l0, 0x05010400u), perm(h0, l0, 0x07030602u),
                       perm(h1, l1, 0x05010400u), perm(h1, l1, 0x07030602u)};
            *(LAS u32x4*)(o + u * 4096 + f * 1024 + lane * 16) = r;
            __builtin_amdgcn_sched_barrier(0);
        }
    }
    const u32 ob = oo + (lane >> 5) * 4096, hl = (lane & 32) ? S.hdr1 : S.hdr0;
    lower_listed(ob, S.pos, lane & 31, (hl >> 16) & 63u);
    if (__builtin_expect(((S.hdr0 | S.hdr1) & 0xFF00u) != 0, 0)) {
        if ((S.hdr0 | S.hdr1) & HDR_OVF) {
            lower_listed(ob, S.lst, (lane & 31) * 2, hl >> 24);
            lower_listed(ob, S.lst >> 16, (lane & 31) * 2 + 1, hl >> 24);
        }
        const u32 hdr = (lane & 8) ? S.hdr1 : S.hdr0;
        if (lane < 16 && (lane & 7) + ((hdr >> 8) & 7u) > 7u)
            *(LAS unsigned short*)(o + (lane & 8) * 512 + ((S.rec >> 16) & 0xFFE)) = (unsigned short)S.rec;
        if (__builtin_expect(((S.hdr0 | S.hdr1) & HDR_RAW) != 0, 0)) {
#pragma unroll
            for (int u = 0; u < 2; ++u)
                if ((u ? S.hdr1 : S.hdr0) & HDR_RAW) {
#pragma unroll
                    for (int f = 0; f < 4; ++f)
                        *(LAS u32x4*)(o + u * 4096 + f * 1024 + lane * 16) = __builtin_amdgcn_raw_buffer_load_b128(
                            rs, lane * 16 + f * 1024, RAW_OFF + (S.unit + u) * RAWU_B, 17);
                }
        }
    }
}

DI void store_tile(char* o, const u32x4 (&v)[4], int lane) {
    o += (size_t)(lane & 15) * NN + ((lane >> 4) & 1) * 32 + (lane >> 5) * 16;
#pragma unroll
    for (int mi = 0; mi < 2; ++mi) {
        u32x4 a, b;
#pragma unroll
        for (int j = 0; j < 4; ++j) {
            auto r = __builtin_amdgcn_permlane16_swap(v[2 * mi][j], v[2 * mi + 1][j], false, false);
            a[j] = r[0]; b[j] = r[1];
        }
        st_wt128(o + (size_t)(32 * mi) * NN, a);
        st_wt128(o + (size_t)(32 * mi + 16) * NN, b);
    }
}

extern "C" __global__ __launch_bounds__(768, 3) void fused_kernel(Params p) {
    extern __shared__ __attribute__((aligned(16))) char smem_[];
    LAS char* lds = (LAS char*)smem_;
    const int lane = threadIdx.x & 63;
    const int wave = rfl(threadIdx.x >> 6);
    const int b = blockIdx.x, xcd = b & 7, idx = b >> 3;
    const int rank = p.rank;
    char* myws = p.me;
    STAMP(0);
    if (b < CWB && wave > 0) {
        const char* pc;
        int lo;
        asm volatile("s_getpc_b64 %0\n.Lhere%=:\n\ts_mov_b32 %1, fused_kernel-.Lhere%=" : "=s"(pc), "=s"(lo));
        const int o = ((wave - 1) * 64 + lane) * 64;
        if (o < CODE_BYTES) {
            const u32 v = *(const GAS u32*)(pc + lo + o);
            asm volatile("" :: "v"(v));
        }
    }
    if (__builtin_expect(wave == 0 && b < NPUSH, 1)) push_inputs(p, b, lds + L_SCR, lane);

    GAS u32* ctr = (GAS u32*)(myws + CTR_OFF + b * 128);
    const u32 ctr_old = __hip_atomic_load(ctr, __ATOMIC_RELAXED, __HIP_MEMORY_SCOPE_AGENT);

    if (idx < 16) {
        const int g = xcd * 16 + idx;
        if (threadIdx.x >= 256 && threadIdx.x < 512) {
            u32 m = threadIdx.x - 256, n = 0, s[2] = {0, 0};
#pragma unroll
            for (int e = 0; e < 8; ++e) {
                u32 bit = (m >> e) & 1;
                s[e >> 2] |= (bit ? n : 0x0Cu) << ((e & 3) * 8);
                n += bit;
            }
            *(LAS u32x2*)(lds + L_LUT + m * 8) = u32x2{s[0], s[1]};
        }
        if (b >= NPUSH) {
            __builtin_amdgcn_s_sleep(INDELAY);
            constexpr int NW = (128 - NPUSH) * 12, NL = (4096 + NW - 1) / NW;
            const u32 q = (b - NPUSH) * 12 + wave;
            u32x4 t[2 * NL];
#pragma unroll
            for (int j = 0; j < NL; ++j) {
                const u32 u = min(j * NW + q, 4095u);
                t[2 * j] = *(const GAS u32x4*)(p.w + u * 1024 + lane * 16);
                t[2 * j + 1] = *(const GAS u32x4*)(p.x + ((u & 511) * 8 + (lane >> 3)) * (KS * 2) + (u >> 9) * 128 + (lane & 7) * 16);
            }
#pragma unroll
            for (int j = 0; j < 2 * NL; ++j) asm volatile("" :: "v"(t[j]));
        }
        const u32 call = rfl(ctr_old) + 1u;
        STAMP(1);
        BARRIER();
        if (threadIdx.x == 64) __hip_atomic_store(ctr, call, __ATOMIC_RELAXED, __HIP_MEMORY_SCOPE_AGENT);
        const rsrc_t rs = __builtin_amdgcn_make_buffer_rsrc(myws, 0, WS_USED, 0x00020000);
        const int uq = g * 4 + (wave & 3);
        const int rot = uq % 7;
        const int tr = (g >> 5) * 4 + ((g >> 2) & 3), tc = ((g >> 4) & 1) * 4 + (g & 3);
        LAS char* buf = lds + L_DEC + wave * DEC_B;
        for (int k = wave >> 2; k < 7; k += 3) {
            const int src = (rank + 7 - (k + rot) % 7) & 7;
            const u32 un = src * 512 + uq;
            bool bad = false;
            const char* fa = myws + F2_OFF + un * FL_B + (lane & 7) * 8;
            u64 F = ld_sys64(fa);
            if (__builtin_amdgcn_ballot_w64((u32)(F >> 32) < call) != 0) {
                u64 t0 = __builtin_amdgcn_s_memrealtime();
                u32 it = 0;
                for (;;) {
                    F = ld_sys64(fa);
                    if (__builtin_amdgcn_ballot_w64((u32)(F >> 32) < call) == 0) break;
                    if (spin_expired(myws, call, t0, it)) { bad = true; break; }
                }
            }
            asm volatile("" ::: "memory");
            const u32 h2 = rfl((u32)F);
            const u32 total = min(h2 & 0x1FFFu, 4096u), ng = (total + 15) >> 4;
            const u32 so = P2_OFF + un * P2_B;
            const u32x2 mk = __builtin_amdgcn_raw_buffer_load_b64(rs, (lane & 31) * 16 + (lane >> 5) * 8, so, 17);
            if (__builtin_expect((h2 & H2_RAW) != 0, 0)) {
                u32x4 pl[4];
#pragma unroll
                for (int r = 0; r < 4; ++r)
                    pl[r] = __builtin_amdgcn_raw_buffer_load_b128(
                        rs, (u32)(lane + r * 64) < ng ? (u32)(512 + lane * 16 + r * 1024) : 0x80000000u, so, 17);
                bad = bad || tmo_set(myws, call);
#pragma unroll
                for (int r = 0; r < 4; ++r) *(LAS u32x4*)(buf + r * 1024 + lane * 16) = pl[r];
            } else {
                const u32 ne = min((h2 >> 13) & 0xFFu, (u32)MAXE2), w4 = ((h2 >> 21) & 0x3Fu) * 0x01010101u;
                u32x3 pl[4];
#pragma unroll
                for (int r = 0; r < 4; ++r)
                    pl[r] = __builtin_amdgcn_raw_buffer_load_b96(
                        rs, (u32)(lane + r * 64) < ng ? (u32)(512 + lane * 12 + r * 768) : 0x80000000u, so, 17);
                u32 ex[2];
#pragma unroll
                for (int r = 0; r < 2; ++r)
                    ex[r] = __builtin_amdgcn_raw_buffer_load_b16(
                        rs, (u32)(lane + r * 64) < ne && (u32)(lane + r * 64) >= INFLAG ? 512 + ng * 12 + (lane + r * 64 - INFLAG) * 2 : 0x80000000u,
                        so, 17);
                {
                    const u32 wv = (u32)__builtin_amdgcn_ds_bpermute(((lane >> 1) + 1) * 4, (int)(u32)F);
                    if (lane < INFLAG) ex[0] = (lane & 1) ? wv >> 16 : wv & 0xFFFFu;
                }
                bad = bad || tmo_set(myws, call);
#pragma unroll
                for (int r = 0; r < 4; ++r)
                    if (r < 2 || (u32)(r * 64) < ng) {
                        const u32 d3 = bfi(0x03030303u, pl[r][0] >> 6, bfi(0x0C0C0C0Cu, pl[r][1] >> 4, (pl[r][2] >> 2) & 0x30303030u));
                        *(LAS u32x4*)(buf + r * 1024 + lane * 16) =
                            u32x4{(pl[r][0] & 0x3F3F3F3Fu) + w4, (pl[r][1] & 0x3F3F3F3Fu) + w4, (pl[r][2] & 0x3F3F3F3Fu) + w4, d3 + w4};
                    }
#pragma unroll
                for (int r = 0; r < 2; ++r)
                    if ((u32)(lane + r * 64) < ne)
                        __hip_atomic_fetch_and((LAS u32*)(buf + (ex[r] & 0xFFCu)), ~(0x40u << ((ex[r] & 3u) * 8)),
                                               __ATOMIC_RELAXED, __HIP_MEMORY_SCOPE_WAVEFRONT);
                asm volatile("" ::: "memory");
            }
            u32 cnt = __builtin_popcount(mk[0]) + __builtin_popcount(mk[1]);
            u32 o = wave_incl_scan(cnt) - cnt;
            u32x4 ot[4];
#pragma unroll
            for (int mt = 0; mt < 4; ++mt) {
                u32x4 ov;
#pragma unroll
                for (int hh = 0; hh < 2; ++hh) {
                    u32 mb = (mk[mt >> 1] >> ((mt & 1) * 16 + hh * 8)) & 0xFF;
                    u32x2 sel = *(LAS u32x2*)(lds + L_LUT + mb * 8);
                    LAS u32* dp = (LAS u32*)(buf + (o & ~3u));
                    u32 d0 = dp[0], d1 = dp[1], d2 = dp[2];
                    u32 a0 = __builtin_amdgcn_alignbyte(d1, d0, o), a1 = __builtin_amdgcn_alignbyte(d2, d1, o);
                    ov[2 * hh] = perm(a1, a0, sel[0]);
                    ov[2 * hh + 1] = perm(a1, a0, sel[1]);
                    o += __builtin_popcount(mb);
                }
                if (bad) ov = u32x4{0x7F7F7F7Fu, 0x7F7F7F7Fu, 0x7F7F7F7Fu, 0x7F7F7F7Fu};
                ot[mt] = ov;
            }
            store_tile(p.out + (size_t)((src >> 2) * 2048 + tr * 128 + ((wave >> 1) & 1) * 64) * NN +
                       (src & 3) * 1024 + tc * 128 + (wave & 1) * 64, ot, lane);
            STAMP(2 + k / 3);
        }
        return;
    }

    const int g = xcd * 16 + (idx - 16);
    const int tr = (g >> 5) * 4 + ((g >> 2) & 3), tc = ((g >> 4) & 1) * 4 + (g & 3);
    const u32 call = rfl(ctr_old) + 1u;

    if (wave >= 4) {
        const int v = wave - 4, ph = v >> 1;
        const int slot = (v & 1) ? 32 + 2 * tc : 2 * tr;
        const rsrc_t rs = __builtin_amdgcn_make_buffer_rsrc(myws, 0, WS_USED, 0x00020000);
        const u32 oa = L_STAGE + ph * 16384 + (v & 1) * 8192;
        volatile LAS unsigned short* prog = (volatile LAS unsigned short*)(lds + L_MISC + 64 + v * 2);
        volatile LAS u32x4* done = (volatile LAS u32x4*)(lds + L_MISC + 96);
        const u32 fo = ((lane >> 3) & 1) * FL_B + (lane & 7) * 8;
        const FTag tag = {(lane & 7) ? 23u : 0u, (lane & 7) ? call & 511u : call};
        u32 utab;
        {
            const u32 r = min(4 * lane + ph, 7 * NCH - 1);
            utab = (((rank + 1 + r % 7) & 7) * NCH + r / 7) * NSLOT + slot;
        }
#define UNIT(i) ((u32)__builtin_amdgcn_readlane(utab, (i)))
        PSet A, B;
        bool bad = false;
        u32x2 F0, Fn;
#define PAYWAIT() do {} while (0)
#define ACC2(x) do {} while (0)
#define FREED(t) do { \
            if (__builtin_expect((int)rfl(min(min(D[0], D[1]), min(D[2], D[3]))) < (t) - 7, 0)) { \
                do { __builtin_amdgcn_s_sleep(1); D = *done; } while ((int)rfl(min(min(D[0], D[1]), min(D[2], D[3]))) < (t) - 7); } \
            asm volatile("" ::: "memory"); } while (0)
#define STAGED(t) do { asm volatile("" ::: "memory"); *prog = (unsigned short)(((t) >> 2) + 1); D = *done; } while (0)
#define STAGE(S, o, t) do { FREED(t); ACC2(a_bar); PAYWAIT(); unpack_set(S, rs, lds, o, lane); STAGED(t); ACC2(a_unp); } while (0)
#define TRYFETCH(S, pend, un, nx, valid) do { \
            const bool up_ = (valid) && stale(Fn, tag) == 0; \
            pend = (valid) && !up_; \
            F0 = Fn; \
            Fn = load_flags(rs, fo, nx); \
            issue_set(S, F0, __builtin_amdgcn_make_buffer_rsrc(myws, 0, up_ ? WS_USED : 0u, 0x00020000), un, lane); \
            ACC2(a_iss); } while (0)
#define REPAIR(S, pend, un, nx) do { \
            if (pend) { bad |= wait_flags(F0, Fn, rs, myws, un, nx, lane, call, tag); issue_set(S, F0, rs, un, lane); LASTREP(un); } \
            ACC2(a_flag); } while (0)
#define LASTREP(un) do {} while (0)
        u32x4 D = {0, 0, 0, 0};
        {
            __builtin_amdgcn_s_sleep(OWNDELAY);
            const bool isw = v & 1;
            const char* src = isw ? p.w + (size_t)ph * (128 * NN * 2) + ((rank & 3) * 8 + tc) * 256
                                  : p.x + (size_t)((rank >> 2) * 16 + tr) * (128 * KS * 2) + ph * 256;
            OSet O;
            own_load(O, isw, src, lane);
            BARRIER();
            own_stage(O, isw, lds + oa, lane);
            STAGED(ph + 4);
            STAMP(1);
            own_load(O, isw, src + (isw ? 64 * NN * 2 : 128), lane);
            Fn = load_flags(rs, fo, UNIT(0));
            FREED(ph + 12);
            own_stage(O, isw, lds + oa, lane);
            STAGED(ph + 12);
        }
        bool pa, pb;
        u32 ua = UNIT(0), ub = UNIT(1);
        TRYFETCH(A, pa, ua, ub, true);
        REPAIR(A, pa, ua, ub);
        TRYFETCH(B, pb, ub, UNIT(2), true);
        for (int i = 0; i < 28; i += 2) {
            const bool more = i < 26;
            const int t = NCH + 4 * i + ph;
            STAGE(A, oa, t);
            ua = UNIT(i + 2);
            REPAIR(B, pb, ub, ua);
            TRYFETCH(A, pa, ua, UNIT(i + 3), more);
            STAGE(B, oa + 65536, t + 4);
            ub = UNIT(i + 3);
            REPAIR(A, pa, ua, ub);
            TRYFETCH(B, pb, ub, UNIT(i + 4), more);
        }
        STAMP(2);
        PUT(6, lr_t); PUT(13, lr_u * 100); PUT(14, lr_n * 100);
        BARRIER();
        STAMP(3);
        u32 gm;
        {
            const char* ap = myws + AMAX_OFF + (v * 128 + lane) * 8;
            u64 t0 = __builtin_amdgcn_s_memrealtime();
    u32 it = 0;
            for (;;) {
                const u64 a0 = ld_sys64(ap), a1 = ld_sys64(ap + 512);
                const u32 tm = __hip_atomic_load((GAS u32*)(myws + TMO_OFF), __ATOMIC_RELAXED, __HIP_MEMORY_SCOPE_AGENT);
                gm = max((u32)a0, (u32)a1);
                const u32 tg = min((u32)(a0 >> 32), (u32)(a1 >> 32));
                if (rfl(tm) == call) { bad = true; break; }
                if (__builtin_amdgcn_ballot_w64(tg < call) == 0) break;
                if (spin_expired(myws, call, t0, it)) { bad = true; break; }
            }
        }
        gm = wave_umax(gm);
        if (bad) gm = 0xFFFFFFFFu;
        if (lane == 0) *(LAS u32*)(lds + L_MISC + 32 + v * 4) = gm;
        STAMP(4);
        BARRIER();
        return;
    }

    const int wm = wave >> 1, wn = wave & 1;
    char* amp = p.ws[0];
#pragma unroll
    for (int r = 1; r < NR; ++r) amp = (lane & 7) == r ? p.ws[r] : amp;
    amp += AMAX_OFF + (rank * 128 + g) * 8;
    asm volatile("" : "+v"(amp));
    const int uq = g * 4 + wave;
    const u32 uoff = P2_OFF + (rank * 512 + uq) * P2_B;
    char* pd[7];
#pragma unroll
    for (int k = 0; k < 7; ++k) pd[k] = p.peer[uq % 7 + k];
    asm volatile("" : "+s"(pd[0]), "+s"(pd[1]), "+s"(pd[2]), "+s"(pd[3]), "+s"(pd[4]), "+s"(pd[5]), "+s"(pd[6]));
    f32x16 acc[2][2];
#pragma unroll
    for (int i = 0; i < 4; ++i)
#pragma unroll
        for (int j = 0; j < 16; ++j) acc[i >> 1][i & 1][j] = 0.f;
    {
        u32 xb = L_STAGE + wm * 4096 + lane * 16, wb = L_STAGE + (2 + wn) * 4096 + lane * 16;
        volatile LAS u32* prog = (volatile LAS u32*)(lds + L_MISC + 64);
        volatile LAS u32* done = (volatile LAS u32*)(lds + L_MISC + 96 + wave * 4);
        bf16x8 x0[4], w0[4], x1[4], w1[4];
#define MSLEEP() do {} while (0)
#define SB() __builtin_amdgcn_sched_barrier(0)
#define RD1(X, W, off, f) do { W[f] = *(LAS bf16x8*)(lds + wb + (off) + (f) * 1024); X[f] = *(LAS bf16x8*)(lds + xb + (off) + (f) * 1024); SB(); } while (0)
#define MM1(X, W, i) do { acc[((i) >> 1) & 1][(i) & 1] = __builtin_amdgcn_mfma_f32_32x32x16_bf16( \
            W[((i) >> 2) * 2 + ((i) & 1)], X[(i) >> 1], acc[((i) >> 1) & 1][(i) & 1], 0, 0, 0); \
            asm volatile("" : "+v"(acc[((i) >> 1) & 1][(i) & 1])); SB(); } while (0)
#define LACKS(need) (((rfl(P) - (u32)(need) * 0x10001u) & 0x80008000u) != 0)
#define AWAIT(j, need) do { \
            if (__builtin_expect(LACKS(need), 0)) { do { MSLEEP(); P = prog[(j) & 3]; } while (LACKS(need)); } \
            asm volatile("" ::: "memory"); SB(); } while (0)
#define RELEASE(j, t) do { asm volatile("" ::: "memory"); *done = (t) + 1; P = prog[((j) + 1) & 3]; SB(); } while (0)
#define STEP(X, W, XN, WN, j) do { constexpr u32 o_ = ((j) & 3) * 16384; \
            MM1(X, W, 0); AWAIT(j, (t4 >> 2) + ((j) >> 2) + 1); MM1(X, W, 1); \
            if ((j) == 4) { xb ^= 65536u; wb ^= 65536u; } \
            RD1(XN, WN, o_, 0); RD1(XN, WN, o_, 1); MM1(X, W, 2); \
            RD1(XN, WN, o_, 2); RD1(XN, WN, o_, 3); MM1(X, W, 3); RELEASE(j, t4 + (j)); \
            MM1(X, W, 4); MM1(X, W, 5); MM1(X, W, 6); MM1(X, W, 7); } while (0)
        if (threadIdx.x < 12) *(LAS u32*)(lds + L_MISC + 64 + threadIdx.x * 4) = 0;
        BARRIER();
        if (threadIdx.x == 0) __hip_atomic_store(ctr, call, __ATOMIC_RELAXED, __HIP_MEMORY_SCOPE_AGENT);
        u32 P = 0;
        AWAIT(0, 1);
        RD1(x0, w0, 0, 0); RD1(x0, w0, 0, 1); RD1(x0, w0, 0, 2); RD1(x0, w0, 0, 3);
        RELEASE(0, 0);
        int t4 = 0;
        for (; t4 < 2 * NSTEP - 4; t4 += 4) {
            STEP(x0, w0, x1, w1, 1); STEP(x1, w1, x0, w0, 2); STEP(x0, w0, x1, w1, 3); STEP(x1, w1, x0, w0, 4);
            if (t4 == NCH - 4) {
#pragma unroll
                for (int i = 0; i < 4; ++i) acc[i >> 1][i & 1] *= 4.0f;
                SB();
            }
        }
        STEP(x0, w0, x1, w1, 1); STEP(x1, w1, x0, w0, 2); STEP(x0, w0, x1, w1, 3);
#pragma unroll
        for (int i = 0; i < 8; ++i) MM1(x1, w1, i);
    }
    STAMP(2);
#define Y(e) ((int)__float_as_uint(acc[(e) >> 5][(e) & 1][((e) >> 1) & 15]))
    asm volatile("s_nop 15\n\ts_nop 7"
                 : "+v"(acc[0][0]), "+v"(acc[0][1]), "+v"(acc[1][0]), "+v"(acc[1][1]));
    int mi = 0;
#pragma unroll
    for (int e = 0; e < 64; e += 2) asm("v_max3_i32 %0, %1, %2, %0" : "+v"(mi) : "v"(Y(e)), "v"(Y(e + 1)));
    asm volatile("s_nop 1" : "+v"(mi));
    const u32 mx = wave_umax((u32)mi);
    if (lane == 0) *(LAS u32*)(lds + L_MISC + 16 + wave * 4) = mx;
    BARRIER();
    if (wave == 0) {
        u32x4 wx = *(LAS u32x4*)(lds + L_MISC + 16);
        u32 tm = max(max(wx[0], wx[1]), max(wx[2], wx[3]));
        if (lane < NR) st_sys64(amp, ((u64)call << 32) | tm);
        STAMP(6);
    }
    u32 mk0 = 0, mk1 = 0;
#pragma unroll
    for (int e = 31; e >= 0; e -= 2) {
        u64 c0, c1, c2, c3;
        asm("v_cmp_lt_i32_e64 %2, 0, %6\n\tv_cmp_lt_i32_e64 %3, 0, %7\n\tv_cmp_lt_i32_e64 %4, 0, %8\n\tv_cmp_lt_i32_e64 %5, 0, %9\n\t"
            "v_addc_co_u32_e64 %0, vcc, %0, %0, %2\n\tv_addc_co_u32_e64 %1, vcc, %1, %1, %3\n\t"
            "v_addc_co_u32_e64 %0, vcc, %0, %0, %4\n\tv_addc_co_u32_e64 %1, vcc, %1, %1, %5"
            : "+v"(mk0), "+v"(mk1), "=&s"(c0), "=&s"(c1), "=&s"(c2), "=&s"(c3)
            : "v"(Y(e)), "v"(Y(e + 32)), "v"(Y(e - 1)), "v"(Y(e + 31)) : "vcc");
    }
    u32x4 mv;
    {
        asm volatile("s_nop 1" : "+v"(mk0), "+v"(mk1));
        auto s0 = __builtin_amdgcn_permlane32_swap(mk0, mk0, false, false);
        auto s1 = __builtin_amdgcn_permlane32_swap(mk1, mk1, false, false);
        mv = u32x4{s0[0], s1[0], s0[1], s1[1]};
    }
#define MA 3
#define MB 5
#define MASKS(k0, k1) do { if (lane < 32) { _Pragma("unroll") for (int k = (k0); k < (k1); ++k) st_wt128s(pd[k], uoff + lane * 16, mv); } } while (0)
    MASKS(0, MA);
    const u32 cnt = __builtin_popcount(mk0) + __builtin_popcount(mk1);
    const u32 incl = wave_incl_scan(cnt);
    const u32 total = __builtin_amdgcn_readlane(incl, 63);
    LAS char* cb = lds + L_COMP + wave * 16384;
    {
        u32 pos = L_COMP + wave * 16384 + (incl - cnt) * 4;
#define C1(n) "v_cmpx_lt_i32_e32 vcc, 0, %" #n "\n\tds_write_b32 %0, %" #n "\n\tv_add_u32 %0, 4, %0\n\ts_mov_b64 exec, -1\n\t"
#pragma unroll
        for (int e = 0; e < 64; e += 8) {
            if (e == 32 && MA < MB) { asm volatile("s_waitcnt vmcnt(1)" ::: "memory"); MASKS(MA, MB); }
            asm volatile(C1(1) C1(2) C1(3) C1(4) C1(5) C1(6) C1(7) C1(8)
                         : "+v"(pos)
                         : "v"(Y(e)), "v"(Y(e + 1)), "v"(Y(e + 2)), "v"(Y(e + 3)), "v"(Y(e + 4)), "v"(Y(e + 5)),
                           "v"(Y(e + 6)), "v"(Y(e + 7))
                         : "memory", "vcc");
        }
        asm volatile("s_nop 3");
        if (lane < 16 && total + lane < 4096u) *(LAS u32*)(cb + (total + lane) * 4) = mx;
    }
    const u32 ng = (total + 15) >> 4;
    f32x4 cv[4][4];
#pragma unroll
    for (int r = 0; r < 4; ++r)
#pragma unroll
        for (int j = 0; j < 4; ++j) cv[r][j] = *(LAS f32x4*)(cb + r * 4096 + lane * 64 + j * 16);
    STAMP(3);
    if (MB < NEARLY) { asm volatile("s_waitcnt vmcnt(1)" ::: "memory"); MASKS(MB, NEARLY); }
    BARRIER();
    MASKS(NEARLY, 7);
    float inv;
    {
        u32x4 gv = *(LAS u32x4*)(lds + L_MISC + 32), gw = *(LAS u32x4*)(lds + L_MISC + 48);
        float amax = __uint_as_float(max(max(max(gv[0], gv[1]), max(gv[2], gv[3])), max(max(gw[0], gw[1]), max(gw[2], gw[3]))));
        inv = 1.0f / (amax / 448.0f);
    }
    STAMP(4);
    u32x4 q[4];
#pragma unroll
    for (int r = 0; r < 4; ++r) {
        q[r] = u32x4{0x7F7F7F7Fu, 0x7F7F7F7Fu, 0x7F7F7F7Fu, 0x7F7F7F7Fu};
        if (r < 2 || (u32)(r * 64) < ng) {
#pragma unroll
            for (int j = 0; j < 4; ++j) {
                f32x4 t = cv[r][j] * inv;
                int w = __builtin_amdgcn_cvt_pk_fp8_f32(t[0], t[1], 0, false);
                q[r][j] = (u32)__builtin_amdgcn_cvt_pk_fp8_f32(t[2], t[3], w, true);
            }
        }
    }
    u32 h2, nb, fw = 0;
    {
        const u32 cmax = rfl((u32)__builtin_amdgcn_cvt_pk_fp8_f32(__uint_as_float(mx) * inv, 0.f, 0, false)) & 0xFFu;
        const u32 wl = max(cmax, 63u) - 63u, k4 = (0x80u - wl) * 0x01010101u;
        u32 m[2] = {0, 0};
#pragma unroll
        for (int r = 0; r < 4; ++r) {
            u32 t[4];
#pragma unroll
            for (int j = 0; j < 4; ++j) { t[j] = q[r][j] + k4; m[r >> 1] = bfi(0x80808080u, t[j], m[r >> 1] >> 1); }
            if (r < 2 || (u32)(r * 64) < ng) {
                LAS u32* o = (LAS u32*)(cb + lane * 12 + r * 768);
                o[0] = bfi(0x3F3F3F3Fu, t[0], t[3] << 6);
                o[1] = bfi(0x3F3F3F3Fu, t[1], t[3] << 4);
                o[2] = bfi(0x3F3F3F3Fu, t[2], t[3] << 2);
            }
        }
#pragma unroll
        for (int h = 0; h < 2; ++h)
            m[h] = ~m[h] & (((u32)(lane + 128 * h) < ng ? 0x0F0F0F0Fu : 0u) | ((u32)(lane + 128 * h + 64) < ng ? 0xF0F0F0F0u : 0u));
        const u32 ec = __builtin_popcount(m[0]) + __builtin_popcount(m[1]);
        const u32 ei = wave_incl_scan(ec);
        const u32 ne = __builtin_amdgcn_readlane(ei, 63);
        if (__builtin_expect(ne > MAXE2 || cmax > 126u, 0)) {
            h2 = total | H2_RAW;
            nb = ng * 16;
        } else {
            h2 = total | (ne << 13) | (wl << 21);
            nb = (ng * 12 + (max(ne, INFLAG) - INFLAG) * 2 + 63) & ~63u;
            u32 ix = ei - ec;
            u64 mm = ((u64)m[1] << 32) | m[0];
            while (mm != 0) {
                const u32 bp = __builtin_ctzll(mm);
                *(LAS unsigned short*)(cb + ix * 2 + (ix < INFLAG ? 8192 + 4 : ng * 12 - INFLAG * 2)) =
                    (unsigned short)(lane * 16 + (bp >> 5) * 2048 + (bp & 4) * 256 + (bp & 3) * 4 + ((bp >> 3) & 3));
                ++ix;
                mm &= mm - 1;
            }
            fw = *(LAS u32*)(cb + 8192 + (lane & 7) * 4);
#pragma unroll
            for (int r = 0; r < 4; ++r) q[r] = *(LAS u32x4*)(cb + r * 1024 + lane * 16);
        }
    }
#pragma unroll
    for (int k = 0; k < 7; ++k) {
        char* d = pd[k];
#pragma unroll
        for (int r = 0; r < 4; ++r)
            if ((u32)(lane * 16 + r * 1024) < nb) st_wt128s(d, uoff + 512 + r * 1024 + lane * 16, q[r]);
        if (k == 0) {
            u32x4 ot[4];
#pragma unroll
            for (int mt = 0; mt < 4; ++mt) {
                u32x4 ov;
#pragma unroll
                for (int j = 0; j < 4; ++j) {
                    const int e = mt * 16 + j * 4;
#define YR(e) (__int_as_float(max(Y(e), 0)) * inv)
                    int w = __builtin_amdgcn_cvt_pk_fp8_f32(YR(e), YR(e + 1), 0, false);
                    ov[j] = (u32)__builtin_amdgcn_cvt_pk_fp8_f32(YR(e + 2), YR(e + 3), w, true);
                }
                ot[mt] = ov;
            }
            store_tile(p.out + (size_t)((rank >> 2) * 2048 + tr * 128 + wm * 64) * NN + (rank & 3) * 1024 + tc * 128 + wn * 64,
                       ot, lane);
        }
        WAIT_VM0();
        if (lane < 8) st_sys64(d + F2_OFF + (rank * 512 + uq) * FL_B + lane * 8, ((u64)call << 32) | (lane ? fw : h2));
    }
    STAMP(5);
}

extern "C" void kernel_launch(int rank, int nranks, ncclComm_t comm, hipStream_t stream,
                              void* const* d_in, const int* in_sizes, int n_in,
                              void* d_out, int out_size, const int* in_dtypes, int out_dtype,
                              void* const* d_out_all, void* const* d_ws_all, size_t ws_bytes) {
    (void)comm; (void)in_dtypes; (void)out_dtype; (void)d_out_all;
    if (nranks != NR || (unsigned)rank >= (unsigned)NR || n_in < 2 || ws_bytes < (size_t)SPARE_OFF + WS_USED ||
        in_sizes[0] != MM * KS || in_sizes[1] != KS * NN || out_size != MM * NN) {
        fprintf(stderr, "kernel_launch: unexpected problem shape\n");
        return;
    }
    HIP_CHECK(hipFuncSetAttribute((const void*)fused_kernel, hipFuncAttributeMaxDynamicSharedMemorySize, LDS_BYTES));
    Params p;
    p.x = (const char*)d_in[0];
    p.w = (const char*)d_in[1];
    p.out = (char*)d_out;
    for (int r = 0; r < NR; ++r) p.ws[r] = (char*)d_ws_all[r];
    for (int r = 0; r < NR; ++r) p.wsd[r] = p.ws[r] + (r == rank ? SPARE_OFF : 0u);
    for (int k = 0; k < 14; ++k) p.peer[k] = p.ws[(rank + 1 + k % 7) & 7];
    p.rank = rank;
    p.me = p.ws[rank];
    p.pad = 0;
    fused_kernel<<<dim3(256), dim3(768), LDS_BYTES, stream>>>(p);
    HIP_CHECK(hipGetLastError());
}
```

```cpp
#include <hip/hip_runtime.h>
#include <rccl/rccl.h>
#include <cstdio>
#include <cstdint>

#define HIP_CHECK(expr) do { hipError_t _e = (expr); if (_e != hipSuccess) { \
    fprintf(stderr, "HIP error %s at %s:%d\n", hipGetErrorString(_e), __FILE__, __LINE__); \
    return; } } while (0)

typedef unsigned u32;
typedef unsigned long long u64;
typedef u32 u32x2 __attribute__((ext_vector_type(2)));
typedef u32 u32x3 __attribute__((ext_vector_type(3)));
typedef u32 u32x4 __attribute__((ext_vector_type(4)));
typedef float f32x4 __attribute__((ext_vector_type(4)));
typedef float f32x16 __attribute__((ext_vector_type(16)));
typedef unsigned short u16x2 __attribute__((ext_vector_type(2)));
typedef __bf16 bf16x8 __attribute__((ext_vector_type(8)));
#define GAS __attribute__((address_space(1)))
#define LAS __attribute__((address_space(3)))
#define DI __device__ __forceinline__

constexpr int NR = 8;
constexpr int MM = 4096, NN = 4096, KS = 512;
constexpr int NCH = 16;
constexpr int NSLOT = 48;
constexpr u32 NITEM = NCH * 128;
constexpr int INDELAY = 8;
constexpr int KW = 2;
constexpr int CWB = 8;
constexpr int PSL = 1;
constexpr int OUTDELAY = 110;
constexpr int NPUSH = 82;
static_assert(KW <= 4, "the loads and the ticket must be older than KW instructions");
constexpr int NSTEP = 64;
constexpr u32 UNIT_B = 3072, RAWU_B = 4096, FL_B = 128;
constexpr u32 CTR_OFF = 0;
constexpr u32 TMO_OFF = 32768;
constexpr u32 TKT_OFF = 33024;
constexpr u32 TMAX_OFF = 65536;
constexpr u32 AMAX_OFF = 98304;
constexpr u32 F1_OFF = 131072;
constexpr u32 F2_OFF = 1u << 20;
constexpr u32 P1_OFF = 2u << 20;
constexpr u32 RAW_OFF = 20u << 20;
constexpr u32 P2_OFF = 44u << 20;
constexpr u32 P2_B = 4608;
constexpr u32 WS_USED = 62u << 20;
constexpr u32 SPARE_OFF = 64u << 20;
static_assert(F1_OFF + NR * NCH * NSLOT * FL_B <= F2_OFF, "");
static_assert(F2_OFF + NR * 512 * FL_B <= P1_OFF, "");
static_assert(P1_OFF + NR * NCH * NSLOT * UNIT_B <= RAW_OFF, "");
static_assert(RAW_OFF + NR * NCH * NSLOT * RAWU_B <= P2_OFF, "");
static_assert(P2_OFF + NR * 512 * P2_B <= WS_USED, "");
constexpr int CODE_BYTES = 44 * 1024;
constexpr u64 DEADLINE = 50000000ull;
constexpr int MAXEXC = 7;
constexpr u32 MAXL1 = 64;
constexpr u32 HDR_OVF = 0x800, HDR_RAW = 0x1000;
constexpr int MAXE2 = 128;
constexpr u32 H2_RAW = 1u << 28;
constexpr u32 INFLAG = 35;

constexpr u32 L_STAGE = 0;
constexpr u32 L_COMP = 0;
constexpr u32 L_MASK = 65536;
constexpr u32 L_MISC = 131072;
constexpr u32 L_DEC = 0;
constexpr u32 DEC_B = 4352;
constexpr u32 L_LUT = 53248;
constexpr u32 L_SCR = 57344;
constexpr u32 LDS_BYTES = 131072 + 256;

struct Params {
    const char* x;
    const char* w;
    char* me;
    int rank;
    int pad;
    char* wsd[NR];
    char* out;
    char* ws[NR];
    char* peer[14];
};

DI u32 perm(u32 hi, u32 lo, u32 sel) { return __builtin_amdgcn_perm(hi, lo, sel); }
DI u32 bfi(u32 m, u32 a, u32 b) { return (m & a) | (~m & b); }
DI u32 rotr(u32 v, u32 n) { return __builtin_amdgcn_alignbit(v, v, n); }
DI u32 rfl(u32 v) { return __builtin_amdgcn_readfirstlane(v); }
template <int CTRL, int RM> DI u32 dpp0(u32 v) { return __builtin_amdgcn_update_dpp(0u, v, CTRL, RM, 0xF, false); }
DI u32 wave_umax(u32 v) {
    v = max(v, dpp0<0x111, 0xF>(v)); v = max(v, dpp0<0x112, 0xF>(v));
    v = max(v, dpp0<0x114, 0xF>(v)); v = max(v, dpp0<0x118, 0xF>(v));
    v = max(v, dpp0<0x142, 0xA>(v)); v = max(v, dpp0<0x143, 0xC>(v));
    return __builtin_amdgcn_readlane(v, 63);
}
DI u32 wave_incl_scan(u32 v) {
    v += dpp0<0x111, 0xF>(v); v += dpp0<0x112, 0xF>(v);
    v += dpp0<0x114, 0xF>(v); v += dpp0<0x118, 0xF>(v);
    v += dpp0<0x142, 0xA>(v); v += dpp0<0x143, 0xC>(v);
    return v;
}
DI u64 ld_sys64(const char* p) {
    return __hip_atomic_load((GAS u64*)p, __ATOMIC_RELAXED, __HIP_MEMORY_SCOPE_SYSTEM);
}
DI void st_sys64(char* p, u64 v) {
    __hip_atomic_store((GAS u64*)p, v, __ATOMIC_RELAXED, __HIP_MEMORY_SCOPE_SYSTEM);
}
DI void st_wt128(char* p, u32x4 v) {
    asm volatile("global_store_dwordx4 %0, %1, off sc0 sc1\n\ts_nop 1" :: "v"(p), "v"(v) : "memory");
}
DI void st_wt128s(const char* sbase, u32 voff, u32x4 v) {
    asm volatile("global_store_dwordx4 %0, %1, %2 sc0 sc1\n\ts_nop 1" :: "v"(voff), "v"(v), "s"(sbase) : "memory");
}
#define WAIT_VM0() asm volatile("s_waitcnt vmcnt(0)" ::: "memory")
#define WAIT_LGKM0() asm volatile("s_waitcnt lgkmcnt(0)" ::: "memory")
#define BARRIER() do { WAIT_LGKM0(); __builtin_amdgcn_s_barrier(); asm volatile("" ::: "memory"); } while (0)

DI bool spin_expired(char* myws, u32 call, u64 t0, u32& it) {
    if ((++it & 15u) == 0) {
        GAS u32* tmo = (GAS u32*)(myws + TMO_OFF);
        u32 t = rfl(__hip_atomic_load(tmo, __ATOMIC_RELAXED, __HIP_MEMORY_SCOPE_AGENT));
        if (t == call || __builtin_amdgcn_s_memrealtime() - t0 > DEADLINE) {
            __hip_atomic_store(tmo, call, __ATOMIC_RELAXED, __HIP_MEMORY_SCOPE_AGENT);
            return true;
        }
    }
    __builtin_amdgcn_s_sleep(PSL);
    return false;
}
DI bool tmo_set(char* myws, u32 call) {
    return rfl(__hip_atomic_load((GAS u32*)(myws + TMO_OFF), __ATOMIC_RELAXED, __HIP_MEMORY_SCOPE_AGENT)) == call;
}

DI u32 twice(u32 v) {
    const u32 m = v & 0x7FFFu;
    return (v & 0x8000u) | (m < 0x80u ? m << 1 : m >= 0x7F00u ? max(m, 0x7F80u) : m + 0x80u);
}
DI u64 pack_unit(const u32x4 (&R)[4], u32x4& p0, u32x4& p1, u32x4& p2, u64& act, LAS char* scr, int lane,
                 char* d0, char* d1, char* d2, char* d3, int nd, u32 unit, u32 call) {
    u32 L[8], H[8];
    u16x2 mx = {0, 0};
#pragma unroll
    for (int f = 0; f < 4; ++f) {
        const u32x4 r = R[f] + 0x00800080u;
        L[2 * f] = perm(r[1], r[0], 0x06040200u); H[2 * f] = perm(r[1], r[0], 0x07050301u);
        L[2 * f + 1] = perm(r[3], r[2], 0x06040200u); H[2 * f + 1] = perm(r[3], r[2], 0x07050301u);
#pragma unroll
        for (int d = 0; d < 4; ++d)
            mx = __builtin_elementwise_max(mx, __builtin_bit_cast(u16x2, R[f][d] & 0x7FFF7FFFu));
    }
    const u32 mine = max((u32)mx[0], (u32)mx[1]), big = wave_umax(mine);
    u32 top = (big + 0x80u) >> 8;
    bool raw = big >= 0x7F00u || top < 10u;
    const bool cut = __builtin_popcountll(__builtin_amdgcn_ballot_w64(mine + 0x80u >= (top << 8))) <= 2;
    if (cut) top -= 1u;
    const u32 base = (top - 3u) & 0x7Fu, k4 = 0x80808080u - base * 0x01010101u;
    u32 T[8], N[8], near = 0x80808080u, m1 = 0;
#pragma unroll
    for (int i = 0; i < 8; ++i) {
        T[i] = (H[i] & 0x7F7F7F7Fu) + k4;
        near &= T[i] + 0x04040404u;
        m1 = bfi(0x80808080u, T[i], m1 >> 1);
        N[i] = bfi(0x80808080u, H[i], T[i]);
    }
    LAS char* stg = scr + 4096 + 64;
    {
        LAS u32* o = (LAS u32*)(stg + lane * 12);
        o[0] = bfi(0x60606060u, N[6] << 5, bfi(0x83838383u, N[0], rotr(N[1], 29)));
        o[1] = bfi(0x60606060u, N[7] << 5, bfi(0x83838383u, N[2], rotr(N[3], 29)));
        o[2] = bfi(0x60606060u, bfi(0x20202020u, N[6] >> 2, N[7] >> 1), bfi(0x83838383u, N[4], rotr(N[5], 29)));
    }
    LAS unsigned short* lst = (LAS unsigned short*)(stg + 1024);
    m1 = ~m1;
    const u32 c1 = __builtin_popcount(m1), i1 = wave_incl_scan(c1);
    u32 n1 = __builtin_amdgcn_readlane(i1, 63);
    raw = raw || n1 > MAXL1;
    if (!raw) {
        LAS unsigned short* el = lst + (i1 - c1);
        while (m1 != 0) {
            const u32 bp = __builtin_ctz(m1);
            *el++ = (unsigned short)((bp & 6) * 256 + lane * 8 + (bp & 1) * 4 + (bp >> 3));
            m1 &= m1 - 1;
        }
    }
    p0 = u32x4{L[0], L[1], L[2], L[3]};
    p1 = u32x4{L[4], L[5], L[6], L[7]};
    u32 hdr = base, rec = 0, n2 = 0;
    if (raw || cut || __builtin_amdgcn_ballot_w64(near != 0x80808080u) != 0) {
        u32 cnt = 0;
        if (!raw) {
#pragma unroll
            for (int f = 0; f < 4; ++f) *(LAS u32x4*)(scr + f * 1024 + lane * 16) = R[f];
            u32 m = 0;
#pragma unroll
            for (int i = 0; i < 8; ++i) {
                const u32 z = ((T[i] + 0x04040404u) ^ 0x80808080u) & 0xF8F8F8F8u;
                m |= ((((z & 0x7F7F7F7Fu) + 0x7F7F7F7Fu) | z) & 0x80808080u) >> (7 - i);
            }
            for (;;) {
                u64 bal = __builtin_amdgcn_ballot_w64(m != 0);
                if (bal == 0 || cnt > MAXEXC) break;
                if (m != 0) {
                    u32 bp = __builtin_ctz(m);
                    u32 addr = (bp & 6) * 512 + lane * 16 + (bp & 1) * 8 + (bp >> 3) * 2;
                    u32 v = twice(*(LAS unsigned short*)(scr + addr));
                    u32 slot = cnt + __builtin_amdgcn_mbcnt_hi((u32)(bal >> 32), __builtin_amdgcn_mbcnt_lo((u32)bal, 0));
                    if (slot < MAXEXC) *(LAS u32*)(scr + 4096 + 28 - slot * 4) = (addr << 16) | v;
                    m &= m - 1;
                }
                cnt += __builtin_popcountll(bal);
            }
        }
        if (raw || cnt > MAXEXC) {
            hdr = HDR_RAW;
            n1 = 0;
#pragma unroll
            for (int f = 0; f < 4; ++f) {
                u32x4 r;
#pragma unroll
                for (int d = 0; d < 4; ++d) r[d] = twice(R[f][d] & 0xFFFFu) | (twice(R[f][d] >> 16) << 16);
                const u32 o = RAW_OFF + unit * RAWU_B + f * 1024 + lane * 16;
                st_wt128s(d0, o, r);
                st_wt128s(d1, o, r);
                if (nd > 2) { st_wt128s(d2, o, r); st_wt128s(d3, o, r); }
            }
        } else {
            n2 = cnt;
            rec = *(LAS u32*)(scr + 4096 + (lane & 7) * 4);
        }
    }
    const u32 nin = min(n1, min(32u, 5u * (7u - n2))), nov = n1 - nin;
    if (__builtin_expect(nov != 0, 0)) {
        hdr |= HDR_OVF;
        *(LAS unsigned short*)(stg + 768 + lane * 2) = lst[nin + lane];
    }
    p2 = *(LAS u32x4*)(stg + lane * 16);
    act = (1ull << (48 + ((nov + 31) >> 5) * 4)) - 1;
    const u32 j = lane & 7;
    const LAS unsigned short* e = lst + j * 5 - 5;
    const u32 e2 = e[2] & 0x7FFu;
    u32 w0 = (e[0] & 0x7FFu) | ((e[1] & 0x7FFu) << 11) | (e2 << 22);
    u32 w1 = (e2 >> 10) | ((e[3] & 0x7FFu) << 1) | ((e[4] & 0x7FFu) << 12) | (call << 23);
    if (j + n2 > 7u) w0 = rec;
    if (j == 0) { w0 = hdr | (n2 << 8) | (nin << 16) | (nov << 24); w1 = call; }
    return ((u64)w1 << 32) | w0;
}

#define A_TICKET(adr) "v_mov_b32 v167, 1\n\ts_mov_b64 exec, 1\n\tglobal_atomic_add v167, %" #adr ", v167, off sc0\n\ts_mov_b64 exec, -1\n\t"
#define A_END(n) "s_mov_b64 exec, -1\n\ts_nop 1\n\ts_waitcnt vmcnt(" #n ")"
#define A_ST(so, v, d, off) "global_store_dwordx4 %" #so ", %" #v ", %" #d " offset:" #off " sc0 sc1\n\t"
#define A_ST4C "s_mov_b64 exec, %8\n\t" A_ST(4, 7, 9, 2048) A_ST(4, 7, 10, 2048) A_ST(4, 7, 11, 2048) A_ST(4, 7, 12, 2048)
#define A_ST4 A_ST(4, 5, 9, 0) A_ST(4, 5, 10, 0) A_ST(4, 5, 11, 0) A_ST(4, 5, 12, 0) \
    A_ST(4, 6, 9, 1024) A_ST(4, 6, 10, 1024) A_ST(4, 6, 11, 1024) A_ST(4, 6, 12, 1024) A_ST4C
#define A_ST2 A_ST(4, 5, 9, 0) A_ST(4, 5, 10, 0) A_ST(4, 6, 9, 1024) A_ST(4, 6, 10, 1024) \
    "s_mov_b64 exec, %8\n\t" A_ST(4, 7, 9, 2048) A_ST(4, 7, 10, 2048)
#define A_LDX "s_nop 4\n\tglobal_load_dwordx4 v[150:153], %0, %2\n\tglobal_load_dwordx4 v[154:157], %1, %2\n\t" \
              "global_load_dwordx4 v[158:161], %0, %2 offset:32\n\tglobal_load_dwordx4 v[162:165], %1, %2 offset:32\n\t"
#define A_WL(i) "global_load_dword v" #i ", %0, %2\n\tv_add_u32 %0, 0x2000, %0\n\t"
#define A_LDW "s_nop 4\n\t" A_WL(150) A_WL(151) A_WL(152) A_WL(153) A_WL(154) A_WL(155) A_WL(156) \
              "global_load_dword v157, %0, %2\n\tv_add_u32 %0, 0x12000, %0\n\t" \
              A_WL(158) A_WL(159) A_WL(160) A_WL(161) A_WL(162) A_WL(163) A_WL(164) "global_load_dword v165, %0, %2\n\t"
#define A_CLOB "memory", "v150", "v151", "v152", "v153", "v154", "v155", "v156", "v157", "v158", "v159", "v160", \
               "v161", "v162", "v163", "v164", "v165", "v167"
#define A_STEP(text) asm volatile(text : "+v"(l0) : "v"(lx1), "s"(src), "v"(tka), "v"(so), "v"(p0), "v"(p1), "v"(p2), \
                                  "s"(act), "s"(cur.d0), "s"(cur.d1), "s"(cur.d2), "s"(cur.d3) : A_CLOB)
#define A_MV(i, r) "v_mov_b32 %" #i ", v" #r "\n\t"
#define A_TAKE A_MV(0, 150) A_MV(1, 151) A_MV(2, 152) A_MV(3, 153) A_MV(4, 154) A_MV(5, 155) A_MV(6, 156) A_MV(7, 157) \
               A_MV(8, 158) A_MV(9, 159) A_MV(10, 160) A_MV(11, 161) A_MV(12, 162) A_MV(13, 163) A_MV(14, 164) A_MV(15, 165)
#define G_OUT "=v"(G[0]), "=v"(G[1]), "=v"(G[2]), "=v"(G[3]), "=v"(G[4]), "=v"(G[5]), "=v"(G[6]), "=v"(G[7]), \
              "=v"(G[8]), "=v"(G[9]), "=v"(G[10]), "=v"(G[11]), "=v"(G[12]), "=v"(G[13]), "=v"(G[14]), "=v"(G[15])

DI void w_to_frags(const u32 (&G)[16], u32x4 (&R)[4]) {
#pragma unroll
    for (int f = 0; f < 4; ++f)
#pragma unroll
        for (int d = 0; d < 4; ++d)
            R[f][d] = perm(G[(f >> 1) * 8 + 2 * d + 1], G[(f >> 1) * 8 + 2 * d], (f & 1) ? 0x07060302u : 0x05040100u);
}

struct Item { bool isw; u32 unit, r0; const char* src; char *d0, *d1, *d2, *d3; };
DI Item make_item(const Params& p, u32 i) {
    Item it;
    const u32 c = (i >> 7) & (NCH - 1), k = (i >> 1) & 63;
    it.isw = i & 1;
    it.r0 = it.isw ? (k & 3) : (k & 1) * 4;
    const u32 sl = it.isw ? k >> 2 : k >> 1;
    it.unit = (p.rank * NCH + c) * NSLOT + (it.isw ? 32 : 0) + sl;
    const u32 ow = c * (32 * NN * 2) + ((k & 3) * 16 + (k >> 2)) * 128, ox = ((k & 1) * 32 + (k >> 1)) * (64 * KS * 2) + c * 64;
    it.src = (it.isw ? p.w : p.x) + (it.isw ? ow : ox);
    return it;
}
DI void item_dests(const Params& p, Item& it) {
    const u32 r0 = it.r0, rs = it.isw ? 4 : 1;
    it.d0 = p.wsd[r0]; it.d1 = p.wsd[r0 + rs]; it.d2 = p.wsd[(r0 + 2 * rs) & 7]; it.d3 = p.wsd[(r0 + 3 * rs) & 7];
}

DI u32 lane_x(int lane) { return (u32)(lane & 31) * (KS * 2) + (lane >> 5) * 16; }
DI u32 lane_w(int lane) {
    return (u32)(8 * (lane >> 5)) * (NN * 2) + ((lane & 19) | ((lane & 4) << 1) | ((lane & 8) >> 1)) * 4;
}

DI void push_inputs(const Params& p, u32 g, LAS char* scr, int lane) {
    const u32 lx0 = lane_x(lane), lx1 = lx0 + 32 * KS * 2, lw0 = lane_w(lane);
    u32 G[16];
    u32x4 R[4], p0, p1, p2;
    {
        const int r0 = (g & 1) * 4;
        asm volatile("" :: "s"(p.x), "s"(p.w), "s"(p.me), "s"(p.rank), "s"(p.wsd[r0]), "s"(p.wsd[r0 + 1]),
                     "s"(p.wsd[r0 + 2]), "s"(p.wsd[r0 + 3]));
    }
    Item cur = make_item(p, 2 * g), nxt = make_item(p, 2 * g + 1);
    item_dests(p, cur);
    u32 call;
    {
        u32 lw = lw0;
        asm volatile("global_load_dwordx4 %1, %6, %8\n\tglobal_load_dwordx4 %3, %7, %8\n\t"
                     "global_load_dwordx4 %4, %6, %8 offset:32\n\tglobal_load_dwordx4 %5, %7, %8 offset:32\n\t"
                     "global_load_dword v149, %9, %10 sc1\n\t" A_LDW
                     "s_waitcnt vmcnt(17)"
                     : "+v"(lw), "=&v"(R[0]), "+s"(nxt.src), "=&v"(R[1]), "=&v"(R[2]), "=&v"(R[3])
                     : "v"(lx0), "v"(lx1), "s"(cur.src), "v"(CTR_OFF + g * 128), "s"(p.me) : A_CLOB, "v149");
    }
    char* tka = p.me + TKT_OFF;
    {
        u32x4 q[2];
#pragma unroll
        for (int f = 0; f < 4; ++f) {
            const u32x4 r = R[f] + 0x00800080u;
            q[f >> 1][(f & 1) * 2] = perm(r[1], r[0], 0x06040200u);
            q[f >> 1][(f & 1) * 2 + 1] = perm(r[3], r[2], 0x06040200u);
        }
        u32 so = P1_OFF + cur.unit * UNIT_B + lane * 16;
        asm volatile("s_nop 4\n\t" A_TICKET(23)
                     A_ST(16, 17, 19, 0) A_ST(16, 17, 20, 0) A_ST(16, 17, 21, 0) A_ST(16, 17, 22, 0)
                     A_ST(16, 18, 19, 1024) A_ST(16, 18, 20, 1024) A_ST(16, 18, 21, 1024) A_ST(16, 18, 22, 1024)
                     "s_waitcnt vmcnt(9)\n\t"
                     "v_readfirstlane_b32 %24, v149\n\t" A_TAKE
                     : "=&v"(G[0]), "=&v"(G[1]), "=&v"(G[2]), "=&v"(G[3]), "=&v"(G[4]), "=&v"(G[5]), "=&v"(G[6]), "=&v"(G[7]),
                       "=&v"(G[8]), "=&v"(G[9]), "=&v"(G[10]), "=&v"(G[11]), "=&v"(G[12]), "=&v"(G[13]), "=&v"(G[14]), "=&v"(G[15]),
                       "+v"(so), "+v"(q[0]), "+v"(q[1]), "+s"(cur.d0), "+s"(cur.d1), "+s"(cur.d2), "+s"(cur.d3), "+v"(tka),
                       "=&s"(call)
                     :: "memory", "v167");
        call += 1u;
    }
    const u32 tbase = (call - 1u) * NITEM - 3u * NPUSH;
    asm volatile("" : "+s"(g));
    Item aft = make_item(p, 2 * NPUSH + g);
    bool more = true, more2 = true;
    char* fp = nullptr;
    u64 fv = 0;
    int fn = 0;
    for (;;) {
        const int nd = cur.isw ? 2 : 4;
        u64 act;
        const u64 hdr = pack_unit(R, p0, p1, p2, act, scr, lane, cur.d0, cur.d1, cur.d2, cur.d3, nd, cur.unit, call);
        const u32 so = P1_OFF + cur.unit * UNIT_B + lane * 16;
        if (fn != 0) {
            u32 tk;
            asm volatile("s_waitcnt vmcnt(%22)\n\tv_readfirstlane_b32 %16, v167\n\t" A_TAKE
                         : G_OUT, "=s"(tk), "+v"(p0), "+v"(p1), "+v"(p2) : "v"(so), "s"(act), "n"(KW) : "memory");
            const u32 i = tk - tbase;
            more2 = i < NITEM;
            aft = make_item(p, i);
        }
        const char* src = aft.src;
        u32 l0 = aft.isw ? lw0 : lx0;
        if (fn == 0) {
            if (aft.isw) A_STEP(A_LDW A_ST4C "s_mov_b64 exec, -1\n\ts_nop 1");
            else A_STEP(A_LDX A_ST4C "s_mov_b64 exec, -1\n\ts_nop 1");
        } else if (aft.isw) {
            if (cur.isw) A_STEP(A_LDW A_TICKET(3) A_ST2 A_END(23));
            else A_STEP(A_LDW A_TICKET(3) A_ST4 A_END(29));
        } else {
            if (cur.isw) A_STEP(A_LDX A_TICKET(3) A_ST2 A_END(11));
            else A_STEP(A_LDX A_TICKET(3) A_ST4 A_END(17));
        }
        if (lane < fn) st_sys64(fp, fv);
        char* const dl = (lane & 8) ? cur.d1 : cur.d0;
        char* const dh = (lane & 8) ? cur.d3 : cur.d2;
        fp = ((lane & 16) ? dh : dl) + F1_OFF + cur.unit * FL_B + (lane & 7) * 8;
        fv = hdr;
        fn = nd * 8;
        if (!more) break;
        cur = nxt; nxt = aft; more = more2;
        item_dests(p, cur);
        if (cur.isw) w_to_frags(G, R);
        else {
#pragma unroll
            for (int f = 0; f < 4; ++f) R[f] = u32x4{G[4 * f], G[4 * f + 1], G[4 * f + 2], G[4 * f + 3]};
        }
    }
    WAIT_VM0();
    if (lane < fn) st_sys64(fp, fv);
}

struct OSet { u32x4 r[16]; };
DI void own_load(OSet& O, bool isw, const char* src, int lane) {
    if (isw) {
        src += lane_w(lane);
#pragma unroll
        for (int j = 0; j < 64; ++j)
            O.r[j >> 2][j & 3] = *(const GAS u32*)(src + ((j >> 4) & 1) * 128 + ((j >> 5) * 32 + ((j >> 3) & 1) * 16 + (j & 7)) * (NN * 2));
    } else {
        src += (lane >> 3) * (KS * 2) + (lane & 7) * 16;
#pragma unroll
        for (int j = 0; j < 16; ++j) O.r[j] = *(const GAS u32x4*)(src + j * 8 * (KS * 2));
    }
}
DI void own_stage(const OSet& O, bool isw, LAS char* o, int lane) {
    if (isw) {
#pragma unroll
        for (int j = 0; j < 16; ++j) {
            const u32x4 a = O.r[j & 14], b = O.r[j | 1];
            const u32 sel = (j & 1) ? 0x07060302u : 0x05040100u;
            *(LAS u32x4*)(o + (j >> 3) * 65536 + (j & 7) * 1024 + lane * 16) =
                u32x4{perm(a[1], a[0], sel), perm(a[3], a[2], sel), perm(b[1], b[0], sel), perm(b[3], b[2], sel)};
        }
    } else {
        o += ((lane >> 2) & 1) * 65536 + ((lane >> 1) & 1) * 2048 + (lane & 1) * 512 + (lane >> 3) * 16;
#pragma unroll
        for (int j = 0; j < 16; ++j) *(LAS u32x4*)(o + (j >> 3) * 4096 + ((j >> 2) & 1) * 1024 + (j & 3) * 128) = O.r[j];
    }
}

typedef __amdgpu_buffer_rsrc_t rsrc_t;
struct PSet { u32x4 lo[4]; u32x3 hi[2]; u32 pos, lst, rec; u32 hdr0, hdr1; u32 unit; };
struct FTag { u32 sh, ref; };
DI u64 stale(u32x2 F, FTag g) { return __builtin_amdgcn_ballot_w64((F[1] >> g.sh) != g.ref); }

DI u32x2 load_flags(rsrc_t rs, u32 voff, u32 unit) {
    return __builtin_amdgcn_raw_buffer_load_b64(rs, voff, F1_OFF + unit * FL_B, 17);
}
DI bool wait_flags(u32x2& F0, u32x2& Fn, rsrc_t rs, char* myws, u32 unit, u32 next, int lane, u32 call, FTag g) {
    u64 t0 = __builtin_amdgcn_s_memrealtime();
    u32 it = 0;
    bool bad = false;
    const u32 voff = F1_OFF + (((lane & 16) ? next : unit) + ((lane >> 3) & 1)) * FL_B + (lane & 7) * 8;
    u32x2 F;
    for (;;) {
        asm volatile("" ::: "memory");
        F = __builtin_amdgcn_raw_buffer_load_b64(rs, voff, 0, 17);
        if ((stale(F, g) & 0xFFFF) == 0) break;
        if (spin_expired(myws, call, t0, it)) { bad = true; break; }
    }
    F0 = F;
    Fn[0] = __builtin_amdgcn_permlane16_swap(F[0], F[0], false, false)[1];
    Fn[1] = __builtin_amdgcn_permlane16_swap(F[1], F[1], false, false)[1];
    return bad;
}
DI void issue_set(PSet& S, u32x2 F, rsrc_t rs, u32 unit, int lane) {
    asm volatile("" ::: "memory");
    S.hdr0 = __builtin_amdgcn_readlane(F[0], 0);
    S.hdr1 = __builtin_amdgcn_readlane(F[0], 8);
    S.rec = F[0];
    S.unit = unit;
#pragma unroll
    for (int u = 0; u < 2; ++u) {
        S.hi[u] = __builtin_amdgcn_raw_buffer_load_b96(rs, lane * 12 + u * UNIT_B + 2048, P1_OFF + unit * UNIT_B, 17);
        S.lo[2 * u] = __builtin_amdgcn_raw_buffer_load_b128(rs, lane * 16 + u * UNIT_B, P1_OFF + unit * UNIT_B, 17);
        S.lo[2 * u + 1] = __builtin_amdgcn_raw_buffer_load_b128(rs, lane * 16 + u * UNIT_B + 1024, P1_OFF + unit * UNIT_B, 17);
    }
    const u32 from = ((lane >> 5) * 8 + 1 + (lane & 31) / 5) * 4;
    const u64 gw = ((u64)(u32)__builtin_amdgcn_ds_bpermute(from, (int)F[1]) << 32) | (u32)__builtin_amdgcn_ds_bpermute(from, (int)F[0]);
    S.pos = (u32)(gw >> ((lane & 31) % 5 * 11));
    const u32 nov = ((lane & 32) ? S.hdr1 : S.hdr0) >> 24;
    S.lst = __builtin_amdgcn_raw_buffer_load_b32(
        rs, (u32)(lane & 31) * 2 < nov ? (lane >> 5) * UNIT_B + 2816 + (lane & 31) * 4 : 0x80000000u, P1_OFF + unit * UNIT_B, 17);
}
DI void lower_listed(u32 ob, u32 pos, u32 i, u32 n) {
    asm volatile("v_cmpx_lt_u32_e32 vcc, %0, %1\n\tds_sub_u32 %2, %3\n\ts_mov_b64 exec, -1\n\ts_nop 3"
                 :: "v"(i), "v"(n), "v"(((pos << 1) & 0xFFCu) | ob), "v"(0x400u << ((pos << 4) & 31u)) : "memory", "vcc");
}
DI void unpack_set(const PSet& S, rsrc_t rs, LAS char* lds, u32 oo, int lane) {
    LAS char* o = lds + oo;
#pragma unroll
    for (int u = 0; u < 2; ++u) {
        const u32 base4 = ((u ? S.hdr1 : S.hdr0) & 0x7F) * 0x01010101u;
#pragma unroll
        for (int f = 0; f < 4; ++f) {
            u32 h0, h1;
            if (f < 3) {
                const u32 w = S.hi[u][f];
                h0 = (w & 0x83838383u) + base4;
                h1 = (rotr(w, 3) & 0x83838383u) + base4;
            } else {
                h0 = ((S.hi[u][2] << 2) & 0x80808080u) | (((S.hi[u][0] >> 5) & 0x03030303u) + base4);
                h1 = ((S.hi[u][2] << 1) & 0x80808080u) | (((S.hi[u][1] >> 5) & 0x03030303u) + base4);
            }
            u32 l0 = S.lo[2 * u + (f >> 1)][(f & 1) * 2], l1 = S.lo[2 * u + (f >> 1)][(f & 1) * 2 + 1];
            u32x4 r = {perm(h0, l0, 0x05010400u), perm(h0, l0, 0x07030602u),
                       perm(h1, l1, 0x05010400u), perm(h1, l1, 0x07030602u)};
            *(LAS u32x4*)(o + u * 4096 + f * 1024 + lane * 16) = r;
            __builtin_amdgcn_sched_barrier(0);
        }
    }
    const u32 ob = oo + (lane >> 5) * 4096, hl = (lane & 32) ? S.hdr1 : S.hdr0;
    lower_listed(ob, S.pos, lane & 31, (hl >> 16) & 63u);
    if (__builtin_expect(((S.hdr0 | S.hdr1) & 0xFF00u) != 0, 0)) {
        if ((S.hdr0 | S.hdr1) & HDR_OVF) {
            lower_listed(ob, S.lst, (lane & 31) * 2, hl >> 24);
            lower_listed(ob, S.lst >> 16, (lane & 31) * 2 + 1, hl >> 24);
        }
        const u32 hdr = (lane & 8) ? S.hdr1 : S.hdr0;
        if (lane < 16 && (lane & 7) + ((hdr >> 8) & 7u) > 7u)
            *(LAS unsigned short*)(o + (lane & 8) * 512 + ((S.rec >> 16) & 0xFFE)) = (unsigned short)S.rec;
        if (__builtin_expect(((S.hdr0 | S.hdr1) & HDR_RAW) != 0, 0)) {
#pragma unroll
            for (int u = 0; u < 2; ++u)
                if ((u ? S.hdr1 : S.hdr0) & HDR_RAW) {
#pragma unroll
                    for (int f = 0; f < 4; ++f)
                        *(LAS u32x4*)(o + u * 4096 + f * 1024 + lane * 16) = __builtin_amdgcn_raw_buffer_load_b128(
                            rs, lane * 16 + f * 1024, RAW_OFF + (S.unit + u) * RAWU_B, 17);
                }
        }
    }
}

DI void store_tile(char* o, const u32x4 (&v)[4], int lane) {
    o += (size_t)(lane & 15) * NN + ((lane >> 4) & 1) * 32 + (lane >> 5) * 16;
#pragma unroll
    for (int mi = 0; mi < 2; ++mi) {
        u32x4 a, b;
#pragma unroll
        for (int j = 0; j < 4; ++j) {
            auto r = __builtin_amdgcn_permlane16_swap(v[2 * mi][j], v[2 * mi + 1][j], false, false);
            a[j] = r[0]; b[j] = r[1];
        }
        st_wt128(o + (size_t)(32 * mi) * NN, a);
        st_wt128(o + (size_t)(32 * mi + 16) * NN, b);
    }
}

extern "C" __global__ __launch_bounds__(768, 3) void fused_kernel(Params p) {
    extern __shared__ __attribute__((aligned(16))) char smem_[];
    LAS char* lds = (LAS char*)smem_;
    const int lane = threadIdx.x & 63;
    const int wave = rfl(threadIdx.x >> 6);
    const int b = blockIdx.x, xcd = b & 7, idx = b >> 3;
    const int rank = p.rank;
    char* myws = p.me;
    if (b < CWB && wave > 0) {
        const char* pc;
        int lo;
        asm volatile("s_getpc_b64 %0\n.Lhere%=:\n\ts_mov_b32 %1, fused_kernel-.Lhere%=" : "=s"(pc), "=s"(lo));
        const int o = ((wave - 1) * 64 + lane) * 64;
        if (o < CODE_BYTES) {
            const u32 v = *(const GAS u32*)(pc + lo + o);
            asm volatile("" :: "v"(v));
        }
    }
    if (__builtin_expect(wave == 0 && b < NPUSH, 1)) push_inputs(p, b, lds + L_SCR, lane);

    GAS u32* ctr = (GAS u32*)(myws + CTR_OFF + b * 128);
    const u32 ctr_old = __hip_atomic_load(ctr, __ATOMIC_RELAXED, __HIP_MEMORY_SCOPE_AGENT);

    if (idx < 16) {
        const int g = xcd * 16 + idx;
        const int tr = (g >> 5) * 4 + ((g >> 2) & 3), tc = ((g >> 4) & 1) * 4 + (g & 3);
        if (threadIdx.x >= 256 && threadIdx.x < 512) {
            u32 m = threadIdx.x - 256, n = 0, s[2] = {0, 0};
#pragma unroll
            for (int e = 0; e < 8; ++e) {
                u32 bit = (m >> e) & 1;
                s[e >> 2] |= (bit ? n : 0x0Cu) << ((e & 3) * 8);
                n += bit;
            }
            *(LAS u32x2*)(lds + L_LUT + m * 8) = u32x2{s[0], s[1]};
        }
        if (b >= NPUSH) {
            __builtin_amdgcn_s_sleep(INDELAY);
            constexpr int NW = (128 - NPUSH) * 12, NL = (4096 + NW - 1) / NW;
            const u32 q = (b - NPUSH) * 12 + wave;
            u32x4 t[2 * NL];
#pragma unroll
            for (int j = 0; j < NL; ++j) {
                const u32 u = min(j * NW + q, 4095u);
                t[2 * j] = *(const GAS u32x4*)(p.w + u * 1024 + lane * 16);
                t[2 * j + 1] = *(const GAS u32x4*)(p.x + ((u & 511) * 8 + (lane >> 3)) * (KS * 2) + (u >> 9) * 128 + (lane & 7) * 16);
            }
#pragma unroll
            for (int j = 0; j < 2 * NL; ++j) asm volatile("" :: "v"(t[j]));
        }
        const u32 call = rfl(ctr_old) + 1u;
        BARRIER();
        if (threadIdx.x == 64) __hip_atomic_store(ctr, call, __ATOMIC_RELAXED, __HIP_MEMORY_SCOPE_AGENT);
        if (b == 127) {
            if (wave == 3) {
                char* dp = p.ws[0];
#pragma unroll
                for (int r = 1; r < NR; ++r) dp = (lane >> 3) == r ? p.ws[r] : dp;
                const char* ap = myws + TMAX_OFF + lane * 128;
                const u64 t0 = __builtin_amdgcn_s_memrealtime();
                u32 it = 0, gm;
                for (;;) {
                    const u64 a0 = ld_sys64(ap), a1 = ld_sys64(ap + 8192);
                    gm = max((u32)a0, (u32)a1);
                    if (__builtin_amdgcn_ballot_w64(min((u32)(a0 >> 32), (u32)(a1 >> 32)) < call) == 0) break;
                    if (spin_expired(myws, call, t0, it)) { gm = 0x7FFFFFFFu; break; }
                }
                st_sys64(dp + AMAX_OFF + rank * 128 + (lane & 7) * 8, ((u64)call << 32) | wave_umax(gm));
            }
        } else if (wave >= 4 && wave - 4 != rank) {
            __builtin_amdgcn_s_sleep(OUTDELAY);
            const char* a = p.out + (size_t)(((wave - 4) >> 2) * 2048 + tr * 128 + (lane >> 3)) * NN + (wave & 3) * 1024 +
                            tc * 128 + (lane & 7) * 16;
            u32x4 t[16];
#pragma unroll
            for (int j = 0; j < 16; ++j) t[j] = *(const GAS u32x4*)(a + j * 8 * NN);
#pragma unroll
            for (int j = 0; j < 16; ++j) asm volatile("" :: "v"(t[j]));
        }
        const rsrc_t rs = __builtin_amdgcn_make_buffer_rsrc(myws, 0, WS_USED, 0x00020000);
        const int uq = g * 4 + (wave & 3);
        const int rot = uq % 7;
        LAS char* buf = lds + L_DEC + wave * DEC_B;
        const u32 tsh = (lane & 7) ? 28u : 0u, tref = (lane & 7) ? call & 15u : call;
        for (int k = wave >> 2; k < 7; k += 3) {
            const int src = (rank + 7 - (k + rot) % 7) & 7;
            const u32 un = src * 512 + uq;
            bool bad = false;
            const u32 so = P2_OFF + un * P2_B;
            u32x2 m0 = {0, 0}, sel[8];
            u32 at[8];
            auto plan = [&]() {
                const u32 cnt = __builtin_popcount(m0[0]) + __builtin_popcount(m0[1]);
                u32 o = wave_incl_scan(cnt) - cnt;
#pragma unroll
                for (int i = 0; i < 8; ++i) {
                    const u32 mb = (m0[i >> 2] >> ((i & 3) * 8)) & 0xFF;
                    sel[i] = *(LAS u32x2*)(lds + L_LUT + mb * 8);
                    at[i] = o;
                    o += __builtin_popcount(mb);
                }
            };
            if (k >= 3) m0 = __builtin_amdgcn_raw_buffer_load_b64(rs, (lane & 31) * 16 + (lane >> 5) * 8, so, 17);
            plan();
            const char* fa = myws + F2_OFF + un * FL_B + (lane & 7) * 8;
            u64 F = ld_sys64(fa);
            if (__builtin_amdgcn_ballot_w64(((u32)(F >> 32) >> tsh) != tref) != 0) {
                u64 t0 = __builtin_amdgcn_s_memrealtime();
                u32 it = 0;
                for (;;) {
                    F = ld_sys64(fa);
                    if (__builtin_amdgcn_ballot_w64(((u32)(F >> 32) >> tsh) != tref) == 0) break;
                    if (spin_expired(myws, call, t0, it)) { bad = true; break; }
                }
            }
            asm volatile("" ::: "memory");
            const u32 h2 = rfl((u32)F);
            const u32 total = min(h2 & 0x1FFFu, 4096u), ng = (total + 15) >> 4;
            const u32x2 mk = __builtin_amdgcn_raw_buffer_load_b64(rs, (lane & 31) * 16 + (lane >> 5) * 8, so, 17);
            if (__builtin_expect((h2 & H2_RAW) != 0, 0)) {
                u32x4 pl[4];
#pragma unroll
                for (int r = 0; r < 4; ++r)
                    pl[r] = __builtin_amdgcn_raw_buffer_load_b128(
                        rs, (u32)(lane + r * 64) < ng ? (u32)(512 + lane * 16 + r * 1024) : 0x80000000u, so, 17);
                bad = bad || tmo_set(myws, call);
#pragma unroll
                for (int r = 0; r < 4; ++r) *(LAS u32x4*)(buf + r * 1024 + lane * 16) = pl[r];
            } else {
                const u32 ne = min((h2 >> 13) & 0xFFu, (u32)MAXE2), w4 = ((h2 >> 21) & 0x3Fu) * 0x01010101u;
                u32x3 pl[4];
#pragma unroll
                for (int r = 0; r < 4; ++r)
                    pl[r] = __builtin_amdgcn_raw_buffer_load_b96(
                        rs, (u32)(lane + r * 64) < ng ? (u32)(512 + lane * 12 + r * 768) : 0x80000000u, so, 17);
                u32 ex[2];
#pragma unroll
                for (int r = 0; r < 2; ++r)
                    ex[r] = __builtin_amdgcn_raw_buffer_load_b16(
                        rs, (u32)(lane + r * 64) < ne && (u32)(lane + r * 64) >= INFLAG ? 512 + ng * 12 + (lane + r * 64 - INFLAG) * 2 : 0x80000000u,
                        so, 17);
                {
                    const int from = (lane / 5 + 1) * 4;
                    const u64 gw = ((u64)(u32)__builtin_amdgcn_ds_bpermute(from, (int)(u32)(F >> 32)) << 32) |
                                   (u32)__builtin_amdgcn_ds_bpermute(from, (int)(u32)F);
                    if (lane < INFLAG) ex[0] = (u32)(gw >> (lane % 5 * 12));
                }
                bad = bad || tmo_set(myws, call);
#pragma unroll
                for (int r = 0; r < 4; ++r)
                    if (r < 2 || (u32)(r * 64) < ng) {
                        const u32 d3 = bfi(0x03030303u, pl[r][0] >> 6, bfi(0x0C0C0C0Cu, pl[r][1] >> 4, (pl[r][2] >> 2) & 0x30303030u));
                        *(LAS u32x4*)(buf + r * 1024 + lane * 16) =
                            u32x4{(pl[r][0] & 0x3F3F3F3Fu) + w4, (pl[r][1] & 0x3F3F3F3Fu) + w4, (pl[r][2] & 0x3F3F3F3Fu) + w4, d3 + w4};
                    }
#pragma unroll
                for (int r = 0; r < 2; ++r)
                    if ((u32)(lane + r * 64) < ne)
                        __hip_atomic_fetch_and((LAS u32*)(buf + (ex[r] & 0xFFCu)), ~(0x40u << ((ex[r] & 3u) * 8)),
                                               __ATOMIC_RELAXED, __HIP_MEMORY_SCOPE_WAVEFRONT);
                asm volatile("" ::: "memory");
            }
            if (__builtin_amdgcn_ballot_w64(mk[0] != m0[0] || mk[1] != m0[1]) != 0) { m0 = mk; plan(); }
            u32x4 ot[4];
#pragma unroll
            for (int i = 0; i < 8; ++i) {
                const LAS u32* dp = (const LAS u32*)(buf + (at[i] & ~3u));
                const u32 d0 = dp[0], d1 = dp[1], d2 = dp[2];
                const u32 a0 = __builtin_amdgcn_alignbyte(d1, d0, at[i]), a1 = __builtin_amdgcn_alignbyte(d2, d1, at[i]);
                ot[i >> 1][(i & 1) * 2] = perm(a1, a0, sel[i][0]);
                ot[i >> 1][(i & 1) * 2 + 1] = perm(a1, a0, sel[i][1]);
            }
            if (bad) {
#pragma unroll
                for (int i = 0; i < 4; ++i) ot[i] = u32x4{0x7F7F7F7Fu, 0x7F7F7F7Fu, 0x7F7F7F7Fu, 0x7F7F7F7Fu};
            }
            store_tile(p.out + (size_t)((src >> 2) * 2048 + tr * 128 + ((wave >> 1) & 1) * 64) * NN +
                       (src & 3) * 1024 + tc * 128 + (wave & 1) * 64, ot, lane);
        }
        return;
    }

    const int g = xcd * 16 + (idx - 16);
    const int tr = (g >> 5) * 4 + ((g >> 2) & 3), tc = ((g >> 4) & 1) * 4 + (g & 3);
    const u32 call = rfl(ctr_old) + 1u;

    if (wave >= 4) {
        const int v = wave - 4, ph = v >> 1;
        const int slot = (v & 1) ? 32 + 2 * tc : 2 * tr;
        const rsrc_t rs = __builtin_amdgcn_make_buffer_rsrc(myws, 0, WS_USED, 0x00020000);
        const u32 oa = L_STAGE + ph * 16384 + (v & 1) * 8192;
        volatile LAS unsigned short* prog = (volatile LAS unsigned short*)(lds + L_MISC + 64 + v * 2);
        volatile LAS u32x4* done = (volatile LAS u32x4*)(lds + L_MISC + 96);
        const u32 fo = ((lane >> 3) & 1) * FL_B + (lane & 7) * 8;
        const FTag tag = {(lane & 7) ? 23u : 0u, (lane & 7) ? call & 511u : call};
        u32 utab;
        {
            const u32 r = min(4 * lane + ph, 7 * NCH - 1);
            utab = (((rank + 1 + r % 7) & 7) * NCH + r / 7) * NSLOT + slot;
        }
#define UNIT(i) ((u32)__builtin_amdgcn_readlane(utab, (i)))
        PSet A, B;
        bool bad = false;
        __builtin_amdgcn_s_setprio(1);
        u32x2 F0, Fn;
#define FREED(t) do { \
            if (__builtin_expect((int)rfl(min(min(D[0], D[1]), min(D[2], D[3]))) < (t) - 7, 0)) { \
                do { __builtin_amdgcn_s_sleep(1); D = *done; } while ((int)rfl(min(min(D[0], D[1]), min(D[2], D[3]))) < (t) - 7); } \
            asm volatile("" ::: "memory"); } while (0)
#define STAGED(t) do { asm volatile("" ::: "memory"); *prog = (unsigned short)(((t) >> 2) + 1); D = *done; } while (0)
#define STAGE(S, o, t) do { FREED(t); unpack_set(S, rs, lds, o, lane); STAGED(t); } while (0)
#define TRYFETCH(S, pend, un, nx, valid) do { \
            const bool up_ = (valid) && stale(Fn, tag) == 0; \
            pend = (valid) && !up_; \
            F0 = Fn; \
            Fn = load_flags(rs, fo, nx); \
            issue_set(S, F0, __builtin_amdgcn_make_buffer_rsrc(myws, 0, up_ ? WS_USED : 0u, 0x00020000), un, lane); } while (0)
#define REPAIR(S, pend, un, nx) do { \
            if (pend) { bad |= wait_flags(F0, Fn, rs, myws, un, nx, lane, call, tag); issue_set(S, F0, rs, un, lane); } } while (0)
        u32x4 D = {0, 0, 0, 0};
        {
            const bool isw = v & 1;
            const char* src = isw ? p.w + (size_t)ph * (128 * NN * 2) + ((rank & 3) * 8 + tc) * 256
                                  : p.x + (size_t)((rank >> 2) * 16 + tr) * (128 * KS * 2) + ph * 256;
            OSet O;
            own_load(O, isw, src, lane);
            BARRIER();
            own_stage(O, isw, lds + oa, lane);
            STAGED(ph + 4);
            own_load(O, isw, src + (isw ? 64 * NN * 2 : 128), lane);
            Fn = load_flags(rs, fo, UNIT(0));
            FREED(ph + 12);
            own_stage(O, isw, lds + oa, lane);
            STAGED(ph + 12);
        }
        bool pa, pb;
        u32 ua = UNIT(0), ub = UNIT(1);
        TRYFETCH(A, pa, ua, ub, true);
        REPAIR(A, pa, ua, ub);
        TRYFETCH(B, pb, ub, UNIT(2), true);
        for (int i = 0; i < 28; i += 2) {
            const bool more = i < 26;
            const int t = NCH + 4 * i + ph;
            STAGE(A, oa, t);
            ua = UNIT(i + 2);
            REPAIR(B, pb, ub, ua);
            TRYFETCH(A, pa, ua, UNIT(i + 3), more);
            STAGE(B, oa + 65536, t + 4);
            ub = UNIT(i + 3);
            REPAIR(A, pa, ua, ub);
            TRYFETCH(B, pb, ub, UNIT(i + 4), more);
        }
        if (bad) __hip_atomic_store((GAS u32*)(myws + TMO_OFF), call, __ATOMIC_RELAXED, __HIP_MEMORY_SCOPE_AGENT);
        WAIT_VM0();
        BARRIER();
        if (v < 4) {
            const int uq = g * 4 + v;
            char* pd[7];
#pragma unroll
            for (int k = 0; k < 7; ++k) pd[k] = p.peer[uq % 7 + k];
            while (rfl(*(volatile LAS u32*)(lds + L_MISC + 112 + v * 4)) == 0) __builtin_amdgcn_s_sleep(1);
            asm volatile("" ::: "memory");
            const u32x4 mv = *(LAS u32x4*)(lds + L_MASK + v * 512 + (lane & 31) * 16);
            volatile LAS u32* landed = (volatile LAS u32*)(lds + L_MISC + 128 + v * 4);
            if (lane < 32) {
#pragma unroll
                for (int k = 0; k < 7; ++k) {
                    asm volatile("global_store_dwordx4 %0, %1, %2 sc0 sc1\n\ts_nop 1\n\ts_waitcnt vmcnt(1)"
                                 :: "v"(P2_OFF + (rank * 512 + uq) * P2_B + lane * 16), "v"(mv), "s"(pd[k]) : "memory");
                    *landed = k;
                }
            }
            WAIT_VM0();
            *landed = 7;
            return;
        }
        if (v != 4) return;
        u32 gm;
        {
            const char* ap = myws + AMAX_OFF + (lane & 7) * 128;
            u64 t0 = __builtin_amdgcn_s_memrealtime();
            u32 it = 0;
            for (;;) {
                const u64 a0 = ld_sys64(ap);
                const u32 tm = __hip_atomic_load((GAS u32*)(myws + TMO_OFF), __ATOMIC_RELAXED, __HIP_MEMORY_SCOPE_AGENT);
                gm = (u32)a0;
                if (rfl(tm) == call) { bad = true; break; }
                if (__builtin_amdgcn_ballot_w64((u32)(a0 >> 32) < call) == 0) break;
                if (spin_expired(myws, call, t0, it)) { bad = true; break; }
            }
        }
        gm = wave_umax(gm);
        if (bad) gm = 0x7FFFFFFFu;
        *(volatile LAS u32*)(lds + L_MISC + 32) = gm | 0x80000000u;
        return;
    }

    const int wm = wave >> 1, wn = wave & 1;
    const int uq = g * 4 + wave;
    const u32 uoff = P2_OFF + (rank * 512 + uq) * P2_B;
    char* pd[7];
#pragma unroll
    for (int k = 0; k < 7; ++k) pd[k] = p.peer[uq % 7 + k];
    asm volatile("" : "+s"(pd[0]), "+s"(pd[1]), "+s"(pd[2]), "+s"(pd[3]), "+s"(pd[4]), "+s"(pd[5]), "+s"(pd[6]));
    f32x16 acc[2][2];
#pragma unroll
    for (int i = 0; i < 4; ++i)
#pragma unroll
        for (int j = 0; j < 16; ++j) acc[i >> 1][i & 1][j] = 0.f;
    {
        u32 xb = L_STAGE + wm * 4096 + lane * 16, wb = L_STAGE + (2 + wn) * 4096 + lane * 16;
        volatile LAS u32* prog = (volatile LAS u32*)(lds + L_MISC + 64);
        volatile LAS u32* done = (volatile LAS u32*)(lds + L_MISC + 96 + wave * 4);
        bf16x8 x0[4], w0[4], x1[4], w1[4];
#define SB() __builtin_amdgcn_sched_barrier(0)
#define RD1(X, W, off, f) do { W[f] = *(LAS bf16x8*)(lds + wb + (off) + (f) * 1024); X[f] = *(LAS bf16x8*)(lds + xb + (off) + (f) * 1024); SB(); } while (0)
#define MM1(X, W, i) do { acc[((i) >> 1) & 1][(i) & 1] = __builtin_amdgcn_mfma_f32_32x32x16_bf16( \
            W[((i) >> 2) * 2 + ((i) & 1)], X[(i) >> 1], acc[((i) >> 1) & 1][(i) & 1], 0, 0, 0); \
            asm volatile("" : "+v"(acc[((i) >> 1) & 1][(i) & 1])); SB(); } while (0)
#define LACKS(need) (((rfl(P) - (u32)(need) * 0x10001u) & 0x80008000u) != 0)
#define AWAIT(j, need) do { \
            if (__builtin_expect(LACKS(need), 0)) { do { P = prog[(j) & 3]; } while (LACKS(need)); } \
            asm volatile("" ::: "memory"); SB(); } while (0)
#define RELEASE(j, t) do { asm volatile("" ::: "memory"); *done = (t) + 1; P = prog[((j) + 1) & 3]; SB(); } while (0)
#define STEP(X, W, XN, WN, j) do { constexpr u32 o_ = ((j) & 3) * 16384; \
            MM1(X, W, 0); AWAIT(j, (t4 >> 2) + ((j) >> 2) + 1); MM1(X, W, 1); \
            if ((j) == 4) { xb ^= 65536u; wb ^= 65536u; } \
            RD1(XN, WN, o_, 0); RD1(XN, WN, o_, 1); MM1(X, W, 2); \
            RD1(XN, WN, o_, 2); RD1(XN, WN, o_, 3); MM1(X, W, 3); RELEASE(j, t4 + (j)); \
            MM1(X, W, 4); MM1(X, W, 5); MM1(X, W, 6); MM1(X, W, 7); } while (0)
        if (threadIdx.x < 28) *(LAS u32*)(lds + L_MISC + 32 + threadIdx.x * 4) = 0;
        BARRIER();
        if (threadIdx.x == 0) __hip_atomic_store(ctr, call, __ATOMIC_RELAXED, __HIP_MEMORY_SCOPE_AGENT);
        u32 P = 0;
        AWAIT(0, 1);
        RD1(x0, w0, 0, 0); RD1(x0, w0, 0, 1); RD1(x0, w0, 0, 2); RD1(x0, w0, 0, 3);
        RELEASE(0, 0);
        int t4 = 0;
        for (; t4 < 2 * NSTEP - 4; t4 += 4) {
            STEP(x0, w0, x1, w1, 1); STEP(x1, w1, x0, w0, 2); STEP(x0, w0, x1, w1, 3); STEP(x1, w1, x0, w0, 4);
            if (t4 == NCH - 4) {
#pragma unroll
                for (int i = 0; i < 4; ++i) acc[i >> 1][i & 1] *= 4.0f;
                SB();
            }
        }
        STEP(x0, w0, x1, w1, 1); STEP(x1, w1, x0, w0, 2); STEP(x0, w0, x1, w1, 3);
#pragma unroll
        for (int i = 0; i < 8; ++i) MM1(x1, w1, i);
    }
#define Y(e) ((int)__float_as_uint(acc[(e) >> 5][(e) & 1][((e) >> 1) & 15]))
    asm volatile("s_nop 15\n\ts_nop 7"
                 : "+v"(acc[0][0]), "+v"(acc[0][1]), "+v"(acc[1][0]), "+v"(acc[1][1]));
    int mi = 0;
#pragma unroll
    for (int e = 0; e < 64; e += 2) asm("v_max3_i32 %0, %1, %2, %0" : "+v"(mi) : "v"(Y(e)), "v"(Y(e + 1)));
    asm volatile("s_nop 1" : "+v"(mi));
    const u32 mx = wave_umax((u32)mi);
    if (lane == 0) *(LAS u32*)(lds + L_MISC + 16 + wave * 4) = mx;
    BARRIER();
    if (wave == 0) {
        u32x4 wx = *(LAS u32x4*)(lds + L_MISC + 16);
        u32 tm = max(max(wx[0], wx[1]), max(wx[2], wx[3]));
        if (lane < 8) st_sys64(myws + TMAX_OFF + g * 128 + lane * 8, ((u64)call << 32) | tm);
    }
    u32 mk0 = 0, mk1 = 0;
#pragma unroll
    for (int e = 31; e >= 0; e -= 2) {
        u64 c0, c1, c2, c3;
        asm("v_cmp_lt_i32_e64 %2, 0, %6\n\tv_cmp_lt_i32_e64 %3, 0, %7\n\tv_cmp_lt_i32_e64 %4, 0, %8\n\tv_cmp_lt_i32_e64 %5, 0, %9\n\t"
            "v_addc_co_u32_e64 %0, vcc, %0, %0, %2\n\tv_addc_co_u32_e64 %1, vcc, %1, %1, %3\n\t"
            "v_addc_co_u32_e64 %0, vcc, %0, %0, %4\n\tv_addc_co_u32_e64 %1, vcc, %1, %1, %5"
            : "+v"(mk0), "+v"(mk1), "=&s"(c0), "=&s"(c1), "=&s"(c2), "=&s"(c3)
            : "v"(Y(e)), "v"(Y(e + 32)), "v"(Y(e - 1)), "v"(Y(e + 31)) : "vcc");
    }
    *(LAS u32x2*)(lds + L_MASK + wave * 512 + (lane & 31) * 16 + (lane >> 5) * 8) = u32x2{mk0, mk1};
    asm volatile("" ::: "memory");
    *(volatile LAS u32*)(lds + L_MISC + 112 + wave * 4) = 1;
    const u32 cnt = __builtin_popcount(mk0) + __builtin_popcount(mk1);
    const u32 incl = wave_incl_scan(cnt);
    const u32 total = __builtin_amdgcn_readlane(incl, 63);
    LAS char* cb = lds + L_COMP + wave * 16384;
    {
        u32 pos = L_COMP + wave * 16384 + (incl - cnt) * 4;
#define C1(n) "v_cmpx_lt_i32_e32 vcc, 0, %" #n "\n\tds_write_b32 %0, %" #n "\n\tv_add_u32 %0, 4, %0\n\ts_mov_b64 exec, -1\n\t"
#pragma unroll
        for (int e = 0; e < 64; e += 8) {
            asm volatile(C1(1) C1(2) C1(3) C1(4) C1(5) C1(6) C1(7) C1(8)
                         : "+v"(pos)
                         : "v"(Y(e)), "v"(Y(e + 1)), "v"(Y(e + 2)), "v"(Y(e + 3)), "v"(Y(e + 4)), "v"(Y(e + 5)),
                           "v"(Y(e + 6)), "v"(Y(e + 7))
                         : "memory", "vcc");
        }
        asm volatile("s_nop 3");
        if (lane < 16 && total + lane < 4096u) *(LAS u32*)(cb + (total + lane) * 4) = mx;
    }
    const u32 ng = (total + 15) >> 4;
    f32x4 cv[4][4];
#pragma unroll
    for (int r = 0; r < 4; ++r)
#pragma unroll
        for (int j = 0; j < 4; ++j) cv[r][j] = *(LAS f32x4*)(cb + r * 4096 + lane * 64 + j * 16);
#pragma unroll
    for (int r = 0; r < 4; ++r) asm volatile("" : "+v"(cv[r][0]), "+v"(cv[r][1]), "+v"(cv[r][2]), "+v"(cv[r][3]));
    float inv;
    {
        u32 gv;
        do { gv = rfl(*(volatile LAS u32*)(lds + L_MISC + 32)); } while ((int)gv >= 0);
        const float amax = __uint_as_float(gv & 0x7FFFFFFFu);
        inv = 448.0f * __builtin_amdgcn_rcpf(amax);
    }
    const u32 cmax = rfl((u32)__builtin_amdgcn_cvt_pk_fp8_f32(__uint_as_float(mx) * inv, 0.f, 0, false)) & 0xFFu;
    const u32 wl = max(cmax, 63u) - 63u, k4 = (0x80u - wl) * 0x01010101u;
    u32x4 q[4];
    u32 m[2] = {0, 0};
#pragma unroll
    for (int r = 0; r < 4; ++r) {
        q[r] = u32x4{0, 0, 0, 0};
        if (r < 2 || (u32)(r * 64) < ng) {
            u32 t[4], in = 0;
#pragma unroll
            for (int j = 0; j < 4; ++j) {
                const f32x4 z = cv[r][j] * inv;
                const int w = __builtin_amdgcn_cvt_pk_fp8_f32(z[0], z[1], 0, false);
                q[r][j] = (u32)__builtin_amdgcn_cvt_pk_fp8_f32(z[2], z[3], w, true);
                t[j] = q[r][j] + k4;
                in = bfi(0x80808080u, t[j], in >> 1);
            }
            LAS u32* o = (LAS u32*)(cb + lane * 12 + r * 768);
            o[0] = bfi(0x3F3F3F3Fu, t[0], t[3] << 6);
            o[1] = bfi(0x3F3F3F3Fu, t[1], t[3] << 4);
            o[2] = bfi(0x3F3F3F3Fu, t[2], t[3] << 2);
            if ((u32)(lane + r * 64) < ng) m[r >> 1] |= (~in & 0xF0F0F0F0u) >> ((r & 1) ? 0 : 4);
        }
    }
    const u32 ec = __builtin_popcount(m[0]) + __builtin_popcount(m[1]);
    const u32 ei = wave_incl_scan(ec);
    const u32 ne = __builtin_amdgcn_readlane(ei, 63);
    const bool raw = ne > MAXE2 || cmax > 126u;
    auto list = [&]() {
        u32 ix = ei - ec;
        u64 mm = ((u64)m[1] << 32) | m[0];
        while (mm != 0) {
            const u32 bp = __builtin_ctzll(mm);
            *(LAS unsigned short*)(cb + ix * 2 + (ix < INFLAG ? 8192 + 10 : ng * 12 - INFLAG * 2)) =
                (unsigned short)(lane * 16 + (bp >> 5) * 2048 + (bp & 4) * 256 + (bp & 3) * 4 + ((bp >> 3) & 3));
            ++ix;
            mm &= mm - 1;
        }
    };
    u32 nb = ng * 16;
    if (__builtin_expect(!raw, 1)) {
        nb = (ng * 12 + (max(ne, INFLAG) - INFLAG) * 2 + 63) & ~63u;
        if (__builtin_expect(ne > INFLAG, 0)) list();
#pragma unroll
        for (int r = 0; r < 4; ++r) q[r] = *(LAS u32x4*)(cb + r * 1024 + lane * 16);
    }
    u64 fw = 0;
    const u32 so = uoff + 512 + lane * 16;
#define CODES(d, r0) do { _Pragma("unroll") for (int r = (r0); r < 4; ++r) \
        if ((u32)(lane * 16 + r * 1024) < nb) st_wt128s(d, so + r * 1024, q[r]); } while (0)
#define FLAG2(d) do { if (lane < 8) st_sys64((d) + F2_OFF + (rank * 512 + uq) * FL_B + lane * 8, fw); } while (0)
    CODES(pd[0], 0);
    {
        if (!raw) {
            if (ne <= INFLAG) list();
            const LAS unsigned short* e = (LAS unsigned short*)(cb + 8192) + (lane & 7) * 5;
            const u32 e2 = e[2] & 0xFFFu;
            fw = ((u64)((e2 >> 8) | ((e[3] & 0xFFFu) << 4) | ((e[4] & 0xFFFu) << 16)) << 32) |
                 ((e[0] & 0xFFFu) | ((e[1] & 0xFFFu) << 12) | (e2 << 24));
        }
        const u32 h2 = raw ? total | H2_RAW : total | (ne << 13) | (wl << 21);
        fw = lane == 0 ? ((u64)call << 32) | h2 : fw | ((u64)call << 60);
        u32x4 ot[4];
#pragma unroll
        for (int mt = 0; mt < 4; ++mt) {
            u32x4 ov;
#pragma unroll
            for (int j = 0; j < 4; ++j) {
                const int e = mt * 16 + j * 4;
#define YR(e) (__int_as_float(max(Y(e), 0)) * inv)
                int w = __builtin_amdgcn_cvt_pk_fp8_f32(YR(e), YR(e + 1), 0, false);
                ov[j] = (u32)__builtin_amdgcn_cvt_pk_fp8_f32(YR(e + 2), YR(e + 3), w, true);
            }
            ot[mt] = ov;
        }
        store_tile(p.out + (size_t)((rank >> 2) * 2048 + tr * 128 + wm * 64) * NN + (rank & 3) * 1024 + tc * 128 + wn * 64,
                   ot, lane);
    }
#define LANDED(k) do { while (rfl(*(volatile LAS u32*)(lds + L_MISC + 128 + wave * 4)) <= (k)) __builtin_amdgcn_s_sleep(1); \
        asm volatile("" ::: "memory"); } while (0)
#pragma unroll
    for (int k = 0; k < 6; ++k) {
        WAIT_VM0();
        LANDED(k);
        FLAG2(pd[k]);
        CODES(pd[k + 1], 0);
    }
    WAIT_VM0();
    LANDED(6);
    FLAG2(pd[6]);
}

extern "C" void kernel_launch(int rank, int nranks, ncclComm_t comm, hipStream_t stream,
                              void* const* d_in, const int* in_sizes, int n_in,
                              void* d_out, int out_size, const int* in_dtypes, int out_dtype,
                              void* const* d_out_all, void* const* d_ws_all, size_t ws_bytes) {
    (void)comm; (void)in_dtypes; (void)out_dtype; (void)d_out_all;
    if (nranks != NR || (unsigned)rank >= (unsigned)NR || n_in < 2 || ws_bytes < (size_t)SPARE_OFF + WS_USED ||
        in_sizes[0] != MM * KS || in_sizes[1] != KS * NN || out_size != MM * NN) {
        fprintf(stderr, "kernel_launch: unexpected problem shape\n");
        return;
    }
    HIP_CHECK(hipFuncSetAttribute((const void*)fused_kernel, hipFuncAttributeMaxDynamicSharedMemorySize, LDS_BYTES));
    Params p;
    p.x = (const char*)d_in[0];
    p.w = (const char*)d_in[1];
    p.out = (char*)d_out;
    for (int r = 0; r < NR; ++r) p.ws[r] = (char*)d_ws_all[r];
    for (int r = 0; r < NR; ++r) p.wsd[r] = p.ws[r] + (r == rank ? SPARE_OFF : 0u);
    for (int k = 0; k < 14; ++k) p.peer[k] = p.ws[(rank + 1 + k % 7) & 7];
    p.rank = rank;
    p.me = p.ws[rank];
    p.pad = 0;
    fused_kernel<<<dim3(256), dim3(768), LDS_BYTES, stream>>>(p);
    HIP_CHECK(hipGetLastError());
}
```
